```python
import numpy as np

kernel_name = "gpu_ag_gemm_m4096_k4096_n4096_e4m3e4m3_dequant"


ABSMAX_RTOL = 0.03

COMPUTE_DTYPE = "float32"
EPILOGUE = "dequant"
IN_QUANT = None
OUT_QUANT = None
IN_FP8 = "e4m3e4m3"
IN_DTYPES = {"x": "e4m3", "w_mat": "e4m3",
             "scale_x": "float32", "scale_w": "float32"}
WORLD_SIZE = 8

def _snap_fp8(x, mant_bits, max_abs, min_normal):
    a = np.abs(x.astype(np.float64))
    _m, _e = np.frexp(a)
    q = np.where(a >= min_normal, np.ldexp(1.0, _e - 1 - mant_bits),
                 min_normal * 2.0 ** -mant_bits)
    snapped = np.minimum(np.round(a / q) * q, max_abs)
    return (np.sign(x) * snapped).astype(np.float32)

def setup_inputs():

    x = _snap_fp8(np.random.randn(4096, 4096).astype(np.float32), 3, 448.0, 2.0 ** -6)
    w_mat = _snap_fp8(np.random.randn(4096, 4096).astype(np.float32), 3, 448.0, 2.0 ** -6)
    scale_x = (0.02 * (1.0 + np.random.rand(1))).astype(np.float32)
    scale_w = (0.02 * (1.0 + np.random.rand(1))).astype(np.float32)
    return {"x": x, "w_mat": w_mat, "scale_x": scale_x, "scale_w": scale_w}

def reference(x, w_mat, scale_x, scale_w):
    acc = (x @ w_mat).astype(np.float32)
    y = (acc * (scale_x * scale_w)).astype(np.float32)
    return y
```

```cpp
#include <hip/hip_runtime.h>
#include <rccl/rccl.h>
#include <cstdio>

#define HIP_CHECK(expr) do { hipError_t _e = (expr); if (_e != hipSuccess) { \
    fprintf(stderr, "HIP error %s at %s:%d\n", hipGetErrorString(_e), __FILE__, __LINE__); \
    return; } } while (0)

typedef unsigned u32x4 __attribute__((ext_vector_type(4)));
typedef int i32x2 __attribute__((ext_vector_type(2)));
typedef int i32x8 __attribute__((ext_vector_type(8)));
typedef float f32x4 __attribute__((ext_vector_type(4)));
#define GAS __attribute__((address_space(1)))
#define LAS __attribute__((address_space(3)))

constexpr int NR = 8;
constexpr int KD = 4096;
constexpr int MS = 512;
constexpr int NL = 512;
constexpr int NB16 = MS / 16;
constexpr int CHK = 256;
constexpr int NCH = KD / CHK;
constexpr int NPUSH = 56;
constexpr int NGEMM = 256;
constexpr int THREADS = 256;
constexpr int LDS_BYTES = 2 * CHK * 128;

constexpr size_t WS_FLAG = 0;
constexpr size_t WS_TMO = 64 << 10;
constexpr size_t WS_CNT = 128 << 10;
constexpr size_t WS_TRACE = 512 << 10;
constexpr size_t WS_SLOT = 2 << 20;
constexpr size_t SLOT_BYTES = (size_t)MS * KD;
constexpr size_t WS_NEED = WS_SLOT + 2 * NR * SLOT_BYTES;
constexpr unsigned long long DEADLINE_TICKS = 100000000ull;

#define PUSH_LAG 1

struct Params {
    const unsigned char* x;
    const unsigned char* w;
    const float* sx;
    const float* sw;
    float* out;
    unsigned char* ws[NR];
    int rank;
    int pad;
};

#define STAMP(i) do {} while (0)

__device__ __forceinline__ void gload(u32x4& d, const unsigned char* a) {
    asm volatile("global_load_dwordx4 %0, %1, off" : "=v"(d) : "v"(a) : "memory");
}
__device__ __forceinline__ void gstore_sys(unsigned char* a, const u32x4& d) {
    asm volatile("global_store_dwordx4 %0, %1, off sc0 sc1\n\ts_nop 1" :: "v"(a), "v"(d) : "memory");
}
__device__ __forceinline__ void gflag_sys(unsigned* a, unsigned v) {
    asm volatile("global_store_dword %0, %1, off sc0 sc1\n\ts_nop 1" :: "v"(a), "v"(v) : "memory");
}
template <int N> __device__ __forceinline__ void wait_vm() {
    asm volatile("s_waitcnt vmcnt(%0)" :: "i"(N) : "memory");
}
template <int N> __device__ __forceinline__ void wait_vm_tie(u32x4 (&r)[4]) {
    asm volatile("s_waitcnt vmcnt(%4)" : "+v"(r[0]), "+v"(r[1]), "+v"(r[2]), "+v"(r[3]) : "i"(N) : "memory");
}

template <int C> struct PushStep {
    static __device__ __forceinline__ void run(u32x4 (&r)[3][4], const unsigned char* src, unsigned char* dst,
                                               unsigned* flag, unsigned base, bool lane0) {
        constexpr int c = C;
        if constexpr (c + 2 < NCH) {
#pragma unroll
            for (int j = 0; j < 4; ++j) gload(r[(c + 2) % 3][j], src + (c + 2) * CHK + j * 64);
        }
        constexpr int younger_loads = (c + 1 < NCH ? 4 : 0) + (c + 2 < NCH ? 4 : 0);
        constexpr int younger_stores = (c >= 1 ? 4 : 0) + (c >= 2 ? 4 : 0);
        wait_vm_tie<younger_loads + younger_stores>(r[c % 3]);
#pragma unroll
        for (int j = 0; j < 4; ++j) gstore_sys(dst + (size_t)c * (NB16 * 4096) + j * 1024, r[c % 3][j]);
        if constexpr (c >= PUSH_LAG) {
            constexpr int lo = c - PUSH_LAG + 1;
            constexpr int nl = (NCH - 2 - lo) < 0 ? 0 : ((NCH - 2 - lo) < PUSH_LAG ? (NCH - 2 - lo) : PUSH_LAG);
            wait_vm<4 * PUSH_LAG + 4 * nl>();
            if (lane0) gflag_sys(flag, base + (c - PUSH_LAG) + 1);
        }
        if constexpr (c + 1 < NCH) PushStep<C + 1>::run(r, src, dst, flag, base, lane0);
    }
};

__global__ void __launch_bounds__(THREADS, 2) ag_gemm(Params p) {
    extern __shared__ __attribute__((aligned(16))) unsigned char smem[];
    const int lane = threadIdx.x & 63;
    const int wave = __builtin_amdgcn_readfirstlane(threadIdx.x >> 6);
    unsigned char* const myws = p.ws[p.rank];
    STAMP(0);

    unsigned* cnt = (unsigned*)(myws + WS_CNT + (size_t)(blockIdx.x * 4 + wave) * 64);

    if (blockIdx.x < NPUSH) {
        const unsigned j = blockIdx.x % (NR - 1);
        const unsigned peer = (p.rank + 1 + j) % NR;
        const int b = (blockIdx.x / (NR - 1)) * 4 + wave;
        const unsigned char* src = p.x + (size_t)(b * 16 + (lane & 15)) * KD + (lane >> 4) * 16;
        u32x4 r[3][4];
#pragma unroll
        for (int c = 0; c < 2; ++c)
#pragma unroll
            for (int jj = 0; jj < 4; ++jj) gload(r[c][jj], src + c * CHK + jj * 64);
        unsigned old = 0;
        if (lane == 0) old = __hip_atomic_fetch_add(cnt, 1u, __ATOMIC_RELAXED, __HIP_MEMORY_SCOPE_AGENT);
        old = __builtin_amdgcn_readfirstlane(old);
        if (peer >= (unsigned)NR) return;
        unsigned char* pws = p.ws[peer];
        unsigned char* dst = pws + WS_SLOT + (size_t)((old & 1) * NR + p.rank) * SLOT_BYTES + (size_t)b * 4096 + lane * 16;
        unsigned* flag = (unsigned*)(pws + WS_FLAG + (size_t)(p.rank * NB16 + b) * 128);
        const unsigned base = old * NCH;
        STAMP(1);
        PushStep<0>::run(r, src, dst, flag, base, lane == 0);
        wait_vm<0>();
        if (lane == 0) gflag_sys(flag, base + NCH);
        STAMP(2);
        return;
    }

    const int g = blockIdx.x - NPUSH;
    const int rb = g & 7;
    const int cb = (g >> 3) & 3;
    const unsigned srcj = (unsigned)g >> 5;
    const unsigned s = (p.rank + srcj) % NR;
    const int b = rb * 4 + wave;
    const bool local = (srcj == 0);

    const int wrow = wave * 64 + (lane >> 3);
    auto stage_w = [&](int c) {
#pragma unroll
        for (int u = 0; u < 8; ++u) {
            const int row = wrow + u * 8;
            const int f = ((row >> 1) & 3) | (((row >> 4) & 1) << 2);
            const unsigned char* gp = p.w + (size_t)(c * CHK + row) * NL + cb * 128 + (((lane & 7) ^ f) << 4);
            __builtin_amdgcn_global_load_lds((const GAS void*)gp,
                                             (LAS void*)(smem + (c & 1) * (CHK * 128) + (wave * 8 + u) * 1024), 16, 0, 0);
        }
    };
    stage_w(0);

    unsigned old = 0;
    if (lane == 0) old = __hip_atomic_fetch_add(cnt, 1u, __ATOMIC_RELAXED, __HIP_MEMORY_SCOPE_AGENT);
    old = __builtin_amdgcn_readfirstlane(old);
    const unsigned call = old + 1;
    const unsigned base = old * NCH;
    const float scale = p.sx[0] * p.sw[0];

    const unsigned char* abase = local ? p.x : myws + WS_SLOT + (size_t)((old & 1) * NR + s) * SLOT_BYTES;
    const __amdgpu_buffer_rsrc_t arsrc = __builtin_amdgcn_make_buffer_rsrc((void*)abase, 0, (int)SLOT_BYTES, 0x00020000);
    const int avoff = local ? (b * 16 + (lane & 15)) * KD + (lane >> 4) * 16 : b * 4096 + lane * 16;
    const int a_cs = local ? CHK : NB16 * 4096;
    const int a_js = local ? 64 : 1024;
    unsigned* flag = (unsigned*)(myws + WS_FLAG + (size_t)(s * NB16 + b) * 128);
    unsigned* tmo = (unsigned*)(myws + WS_TMO);

    unsigned have = local ? NCH : 0;
    bool dead = false;
    auto wait_chunk = [&](int c) {
        if (have > (unsigned)c) return;
        const unsigned long long t0 = __builtin_amdgcn_s_memrealtime();
        unsigned it = 0;
        while (true) {
            unsigned v = __hip_atomic_load(flag, __ATOMIC_RELAXED, __HIP_MEMORY_SCOPE_SYSTEM);
            v = __builtin_amdgcn_readfirstlane(v);
            const int d = (int)(v - base);
            if (d > c) { have = d > NCH ? NCH : d; break; }
            if ((++it & 255) == 0) {
                unsigned t = __hip_atomic_load(tmo, __ATOMIC_RELAXED, __HIP_MEMORY_SCOPE_AGENT);
                t = __builtin_amdgcn_readfirstlane(t);
                if (t == call || __builtin_amdgcn_s_memrealtime() - t0 > DEADLINE_TICKS) {
                    __hip_atomic_store(tmo, call, __ATOMIC_RELAXED, __HIP_MEMORY_SCOPE_AGENT);
                    dead = true; have = NCH;
                    break;
                }
            }
            __builtin_amdgcn_s_sleep(2);
        }
        asm volatile("" ::: "memory");
    };
    auto load_a = [&](u32x4 (&a)[4], int c) {
#pragma unroll
        for (int j = 0; j < 4; ++j)
            a[j] = __builtin_amdgcn_raw_buffer_load_b128(arsrc, avoff, c * a_cs + j * a_js, 17);
    };

    const int G = lane >> 4, i16 = lane & 15;
    const int fl = ((i16 >> 2) & 3) | ((G & 1) << 2);
    const int ldsb = (16 * G + (i16 >> 1)) * 128 + fl * 16 + 8 * (i16 & 1);

    f32x4 acc[8];
#pragma unroll
    for (int n = 0; n < 8; ++n) acc[n] = f32x4{0.f, 0.f, 0.f, 0.f};

    auto compute = [&](const u32x4 (&a)[4], int buf) {
#pragma unroll
        for (int ks = 0; ks < 2; ++ks) {
            const i32x8 xa = {(int)a[2 * ks][0], (int)a[2 * ks][1], (int)a[2 * ks][2], (int)a[2 * ks][3],
                              (int)a[2 * ks + 1][0], (int)a[2 * ks + 1][1], (int)a[2 * ks + 1][2], (int)a[2 * ks + 1][3]};
#pragma unroll
            for (int n = 0; n < 8; ++n) {
                const unsigned char* lp = smem + buf * (CHK * 128) + ks * 16384 + (ldsb ^ (n << 4));
                i32x2 t0 = __builtin_amdgcn_ds_read_tr8_b64_v2i32((LAS i32x2*)(lp));
                i32x2 t1 = __builtin_amdgcn_ds_read_tr8_b64_v2i32((LAS i32x2*)(lp + 1024));
                i32x2 t2 = __builtin_amdgcn_ds_read_tr8_b64_v2i32((LAS i32x2*)(lp + 8192));
                i32x2 t3 = __builtin_amdgcn_ds_read_tr8_b64_v2i32((LAS i32x2*)(lp + 8192 + 1024));
                const i32x8 wf = {t0[0], t0[1], t1[0], t1[1], t2[0], t2[1], t3[0], t3[1]};
                acc[n] = __builtin_amdgcn_mfma_scale_f32_16x16x128_f8f6f4(wf, xa, acc[n], 0, 0, 0, 0, 0, 0);
            }
        }
    };

    STAMP(1);
    u32x4 a0[4], a1[4];
    bool pre = false;
    unsigned tm = 0;
#pragma unroll 1
    for (int c = 0; c < NCH; c += 2) {
        if (!pre) { wait_chunk(c); load_a(a0, c); }
        asm volatile("s_waitcnt vmcnt(0)" ::: "memory");
        __builtin_amdgcn_s_barrier();
        stage_w(c + 1);
        pre = have > (unsigned)(c + 1);
        if (pre) load_a(a1, c + 1);
        compute(a0, 0);
        STAMP(2 + c);
        if (!pre) { wait_chunk(c + 1); load_a(a1, c + 1); }
        if (c + 2 >= NCH) tm = __hip_atomic_load(tmo, __ATOMIC_RELAXED, __HIP_MEMORY_SCOPE_AGENT);
        asm volatile("s_waitcnt vmcnt(0)" ::: "memory");
        __builtin_amdgcn_s_barrier();
        pre = false;
        if (c + 2 < NCH) {
            stage_w(c + 2);
            pre = have > (unsigned)(c + 2);
            if (pre) load_a(a0, c + 2);
        }
        compute(a1, 1);
        STAMP(3 + c);
    }

    const bool bad = dead || (tm == call);
    float* op = p.out + (size_t)(s * MS + b * 16 + i16) * NL + cb * 128 + G * 4;
#pragma unroll
    for (int n = 0; n < 8; ++n) {
        f32x4 v = acc[n] * scale;
        if (bad) v = f32x4{__builtin_nanf(""), __builtin_nanf(""), __builtin_nanf(""), __builtin_nanf("")};
        *(f32x4*)(op + n * 16) = v;
    }
}

extern "C" void kernel_launch(int rank, int nranks, ncclComm_t comm,
                              hipStream_t stream,
                              void* const* d_in, const int* in_sizes, int n_in,
                              void* d_out, int out_size,
                              const int* in_dtypes, int out_dtype,
                              void* const* d_out_all,
                              void* const* d_ws_all, size_t ws_bytes) {
    (void)comm; (void)in_dtypes; (void)out_dtype; (void)d_out_all;
    if (nranks != NR || (unsigned)rank >= (unsigned)NR || n_in != 4 || in_sizes[0] != MS * KD ||
        in_sizes[1] != KD * NL || out_size != NR * MS * NL || ws_bytes < WS_NEED) {
        fprintf(stderr, "unexpected problem shape\n");
        return;
    }
    Params p;
    p.x = (const unsigned char*)d_in[0];
    p.w = (const unsigned char*)d_in[1];
    p.sx = (const float*)d_in[2];
    p.sw = (const float*)d_in[3];
    p.out = (float*)d_out;
    for (int r = 0; r < NR; ++r) p.ws[r] = (unsigned char*)d_ws_all[r];
    p.rank = rank;
    p.pad = 0;
    HIP_CHECK(hipFuncSetAttribute((const void*)ag_gemm, hipFuncAttributeMaxDynamicSharedMemorySize, LDS_BYTES));
    hipLaunchKernelGGL(ag_gemm, dim3(NPUSH + NGEMM), dim3(THREADS), LDS_BYTES, stream, p);
    HIP_CHECK(hipGetLastError());
}
```

```cpp
#include <hip/hip_runtime.h>
#include <rccl/rccl.h>
#include <cstdio>
#include <type_traits>

#define HIP_CHECK(expr) do { hipError_t _e = (expr); if (_e != hipSuccess) { \
    fprintf(stderr, "HIP error %s at %s:%d\n", hipGetErrorString(_e), __FILE__, __LINE__); \
    return; } } while (0)

typedef unsigned u32x4 __attribute__((ext_vector_type(4)));
typedef int i32x2 __attribute__((ext_vector_type(2)));
typedef int i32x8 __attribute__((ext_vector_type(8)));
typedef float f32x4 __attribute__((ext_vector_type(4)));
#define LAS __attribute__((address_space(3)))

#define PWV 2
#define WPP 16
#define WIN 3
#define OUT_AUX 17
#define POLL_SLEEP 2

constexpr int NR = 8;
constexpr int KD = 4096;
constexpr int MS = 512;
constexpr int NL = 512;
constexpr int NB16 = MS / 16;
constexpr int CHK = 256;
constexpr int NCH = KD / CHK;
constexpr int NU = NB16 * NCH;
constexpr int NPUSH = 7 * WPP / PWV;
constexpr int NGEMM = 128;
constexpr int GRIDX = NPUSH + NGEMM;
constexpr int THREADS = 512;
constexpr int STAGE = CHK * 128;
constexpr int LDS_A = 2 * STAGE;
constexpr int LDS_BYTES = LDS_A + 8 * 2 * 4096;
static_assert(GRIDX <= 256 && NPUSH % 8 == 0, "resident grid");

constexpr size_t WS_FLAG = 0;
constexpr size_t WS_TMO = 64 << 10;
constexpr size_t WS_TICKET = 96 << 10;
constexpr size_t WS_CNT = 128 << 10;
constexpr size_t WS_TRACE = 512 << 10;
constexpr size_t WS_SLOT = 2 << 20;
constexpr size_t SLOT_BYTES = (size_t)MS * KD;
constexpr size_t WS_NEED = WS_SLOT + 2 * NR * SLOT_BYTES;
constexpr unsigned long long DEADLINE_TICKS = 100000000ull;

struct Params {
    unsigned char* ws[NR];
    const unsigned char* x;
    const unsigned char* w;
    const float* sx;
    const float* sw;
    float* out;
    int rank;
    int pad;
};

#define STAMP(i) do {} while (0)
#define STAMP_OUT() do {} while (0)

template <int N> __device__ __forceinline__ void wait_vm() {
    asm volatile("s_waitcnt vmcnt(%0)" :: "i"(N) : "memory");
}
__device__ __forceinline__ void glds16(const unsigned char* sbase, unsigned voff, unsigned lds) {
    unsigned keep;
    asm volatile("s_mov_b32 %0, m0\n\ts_mov_b32 m0, %3\n\ts_nop 0\n\tglobal_load_lds_dwordx4 %1, %2\n\ts_mov_b32 m0, %0"
                 : "=&s"(keep) : "v"(voff), "s"(sbase), "s"(lds) : "memory");
}
__device__ __forceinline__ void glds16_sys(const unsigned char* sbase, unsigned voff, unsigned lds) {
    unsigned keep;
    asm volatile("s_mov_b32 %0, m0\n\ts_mov_b32 m0, %3\n\ts_nop 0\n\tglobal_load_lds_dwordx4 %1, %2 sc0 sc1\n\ts_mov_b32 m0, %0"
                 : "=&s"(keep) : "v"(voff), "s"(sbase), "s"(lds) : "memory");
}
template <int OFF> __device__ __forceinline__ void store_sys(const u32x4& d, unsigned char* sbase, unsigned voff) {
    asm volatile("global_store_dwordx4 %1, %0, %2 offset:%3 sc0 sc1\n\ts_nop 1" :: "v"(d), "v"(voff), "s"(sbase), "i"(OFF) : "memory");
}
__device__ __forceinline__ void put_flag(unsigned char* sbase, unsigned zero, unsigned val) {
    asm volatile("s_mov_b64 exec, 1\n\tglobal_store_dword %0, %1, %2 sc0 sc1\n\ts_mov_b64 exec, -1\n\ts_nop 0"
                 :: "v"(zero), "v"(val), "s"(sbase) : "memory");
}
__device__ __forceinline__ void draw(unsigned& t, unsigned* ctr, unsigned zero, unsigned one) {
    asm volatile("s_mov_b64 exec, 1\n\tglobal_atomic_add %0, %1, %2, %3 sc0\n\ts_mov_b64 exec, -1"
                 : "+v"(t) : "v"(zero), "v"(one), "s"(ctr) : "memory");
}
template <int N> __device__ __forceinline__ unsigned take(unsigned& t) {
    unsigned r;
    asm volatile("s_waitcnt vmcnt(%2)\n\tv_readfirstlane_b32 %0, %1\n\ts_nop 4" : "=s"(r) : "v"(t), "i"(N) : "memory");
    return r;
}
constexpr int window_n(int j, bool last) {
    const int back = WIN - j;
    const int cross = back <= 0 ? 0 : (back + 3) / 4;
    return WIN - 1 + 5 * cross - ((last && cross > 0) ? 5 : 0);
}
constexpr int FJ = (3 + WIN) % 4;
constexpr int FB = (3 + WIN) / 4;
static_assert(FB >= 1 && FB <= 2, "window");

__global__ void __launch_bounds__(THREADS, 2) ag_gemm(Params p) {
    extern __shared__ __attribute__((aligned(16))) unsigned char smem[];
    const int lane = threadIdx.x & 63;
    const int wave = __builtin_amdgcn_readfirstlane(threadIdx.x >> 6);
    unsigned char* const myws = p.ws[0];
    const unsigned lds0 = (unsigned)(size_t)(LAS unsigned char*)smem;
    STAMP(0);

    if (blockIdx.x < NPUSH) {
        if (wave >= PWV) return;
        const unsigned j = 1 + blockIdx.x % (NR - 1);
        if (j >= (unsigned)NR) return;
        unsigned char* pws = p.ws[j];
        unsigned* ctr = (unsigned*)(myws + WS_TICKET + j * 128);
        const unsigned zero = 0, one = 1;
        const unsigned lv = (lane & 15) * KD + (lane >> 4) * 16, sv = lane * 16;
        const unsigned ldw = lds0 + wave * 8192;
        auto fetch = [&](unsigned u, unsigned slot) {
            const unsigned char* sb = p.x + (size_t)(u % NB16) * 16 * KD + (u / NB16) * CHK;
#pragma unroll
            for (int q = 0; q < 4; ++q) glds16(sb + q * 64, lv, ldw + slot + q * 1024);
        };
        unsigned tk = 0;
        draw(tk, ctr, zero, one);
        unsigned u0 = (blockIdx.x / (NR - 1)) * PWV + wave;
        fetch(u0, 0);
        const unsigned t = take<4>(tk);
        const unsigned old = t / NU;
        const unsigned vflag = old + 1;
        unsigned char* dst = pws + WS_SLOT + (size_t)((old & 1) * NR + p.rank) * SLOT_BYTES;
        unsigned char* fl = pws + WS_FLAG + (size_t)((old & 1) * NR + p.rank) * (NB16 * 128);
        unsigned u1 = WPP + t - old * NU;
        STAMP(1);
        unsigned h1 = ~0u, h2 = ~0u, slot = 0;
        bool first = true;
        auto flag = [&](unsigned u) { if (u != ~0u) put_flag(fl + (u % NB16) * 128 + (u / NB16) * 4, zero, vflag); };
        auto body = [&](auto lastc) {
            constexpr bool last = decltype(lastc)::value;
            if constexpr (!last) {
                draw(tk, ctr, zero, one);
                fetch(u1, slot ^ 4096);
            }
            if (first) wait_vm<last ? 0 : 5>(); else wait_vm<last ? 4 : 9>();
            u32x4 r[4];
#pragma unroll
            for (int q = 0; q < 4; ++q) r[q] = *(const u32x4*)(smem + wave * 8192 + slot + q * 1024 + lane * 16);
            unsigned char* d = dst + (size_t)u0 * 4096;
            const unsigned fu = FB == 1 ? h1 : h2;
            wait_vm<window_n(0, last)>(); if (FJ == 0) flag(fu); store_sys<0>(r[0], d, sv);
            wait_vm<window_n(1, last)>(); if (FJ == 1) flag(fu); store_sys<1024>(r[1], d, sv);
            wait_vm<window_n(2, last)>(); if (FJ == 2) flag(fu); store_sys<2048>(r[2], d, sv);
            wait_vm<window_n(3, last)>(); if (FJ == 3) flag(fu); store_sys<3072>(r[3], d, sv);
            h2 = h1; h1 = u0; slot ^= 4096; first = false;
        };
        while (u1 < (unsigned)NU) {
            body(std::integral_constant<bool, false>{});
            u0 = u1;
            u1 = WPP + take<8>(tk) - old * NU;
        }
        body(std::integral_constant<bool, true>{});
        if (FB >= 2) { wait_vm<4>(); flag(h2); }
        STAMP(3);
        wait_vm<0>(); flag(h1);
        STAMP(2);
        STAMP_OUT();
        return;
    }

    const int g = blockIdx.x - NPUSH;
    const int cb = (g >> 3) & 3;
    const int rg = (g & 7) + ((g >> 5) << 3);
    const unsigned s = (p.rank + (rg >> 2)) % NR;
    const int b = (rg & 3) * 8 + wave;
    const bool local = (rg >> 2) == 0;

    const unsigned wv0 = (wave * 32 + (lane >> 3)) * NL + cb * 128 + (((lane & 7) ^ ((lane >> 4) & 3)) << 4);
    const unsigned wv1 = wv0 ^ 64;
    auto stage_w = [&](int c) {
        const unsigned char* sb = p.w + (size_t)c * (CHK * NL);
        const unsigned ld = lds0 + (c & 1) * STAGE + wave * 4096;
        glds16(sb, wv0, ld);
        glds16(sb + 8 * NL, wv0, ld + 1024);
        glds16(sb + 16 * NL, wv1, ld + 2048);
        glds16(sb + 24 * NL, wv1, ld + 3072);
    };
    unsigned* cnt = (unsigned*)(myws + WS_CNT + (size_t)(blockIdx.x * 8 + wave) * 64);
    unsigned old = 0;
    if (lane == 0) old = __hip_atomic_fetch_add(cnt, 1u, __ATOMIC_RELAXED, __HIP_MEMORY_SCOPE_AGENT);
    old = __builtin_amdgcn_readfirstlane(old);
    const unsigned call = old + 1;
    const float scale = p.sx[0] * p.sw[0];
    float* outp = p.out;
    asm volatile("" : "+s"(outp) : "v"(scale) : "memory");
    stage_w(0);

    const unsigned char* abase = local ? p.x : myws + WS_SLOT + (size_t)((old & 1) * NR + s) * SLOT_BYTES;
    const unsigned avoff = local ? (b * 16 + (lane & 15)) * KD + (lane >> 4) * 16 : b * 4096 + lane * 16;
    const int a_cs = local ? CHK : NB16 * 4096;
    const int a_js = local ? 64 : 1024;
    unsigned* flag = (unsigned*)(myws + WS_FLAG + (size_t)(((old & 1) * NR + s) * NB16 + b) * 128) + (lane & 15);
    unsigned* tmo = (unsigned*)(myws + WS_TMO);

    unsigned have = local ? 0xffffu : 0u;
    bool dead = false;
    auto wait_chunk = [&](int c) {
        if ((have >> c) & 1) return;
        const unsigned long long t0 = __builtin_amdgcn_s_memrealtime();
        unsigned it = 0;
        while (true) {
            const unsigned v = __hip_atomic_load(flag, __ATOMIC_RELAXED, __HIP_MEMORY_SCOPE_SYSTEM);
            have = (unsigned)__builtin_amdgcn_ballot_w64((int)(v - call) >= 0) & 0xffffu;
            if ((have >> c) & 1) break;
            if ((++it & 255) == 0) {
                unsigned t = __hip_atomic_load(tmo, __ATOMIC_RELAXED, __HIP_MEMORY_SCOPE_AGENT);
                t = __builtin_amdgcn_readfirstlane(t);
                if (t == call || __builtin_amdgcn_s_memrealtime() - t0 > DEADLINE_TICKS) {
                    __hip_atomic_store(tmo, call, __ATOMIC_RELAXED, __HIP_MEMORY_SCOPE_AGENT);
                    dead = true; have = 0xffffu;
                    break;
                }
            }
            __builtin_amdgcn_s_sleep(POLL_SLEEP);
        }
        asm volatile("" ::: "memory");
    };
    auto load_a = [&](int c) {
        const unsigned char* sb = abase + (size_t)c * a_cs;
        const unsigned ld = lds0 + LDS_A + wave * 8192 + (c & 1) * 4096;
#pragma unroll
        for (int j = 0; j < 4; ++j) glds16_sys(sb + j * a_js, avoff, ld + j * 1024);
    };

    const int G = lane >> 4, i16 = lane & 15;
    const int fl = ((i16 >> 2) & 3) | ((G & 1) << 2);
    const int ldsb = (16 * G + (i16 >> 1)) * 128 + fl * 16 + 8 * (i16 & 1);

    f32x4 acc[8];
#pragma unroll
    for (int n = 0; n < 8; ++n) acc[n] = f32x4{0.f, 0.f, 0.f, 0.f};
    i32x8 wf[2][8];
    auto read_w = [&](int buf) {
#pragma unroll
        for (int ks = 0; ks < 2; ++ks)
#pragma unroll
            for (int n = 0; n < 8; ++n) {
                const unsigned char* lp = smem + buf * STAGE + ks * 16384 + (ldsb ^ (n << 4));
                i32x2 t0 = __builtin_amdgcn_ds_read_tr8_b64_v2i32((LAS i32x2*)(lp));
                i32x2 t1 = __builtin_amdgcn_ds_read_tr8_b64_v2i32((LAS i32x2*)(lp + 1024));
                i32x2 t2 = __builtin_amdgcn_ds_read_tr8_b64_v2i32((LAS i32x2*)(lp + 8192));
                i32x2 t3 = __builtin_amdgcn_ds_read_tr8_b64_v2i32((LAS i32x2*)(lp + 8192 + 1024));
                wf[ks][n] = i32x8{t0[0], t0[1], t1[0], t1[1], t2[0], t2[1], t3[0], t3[1]};
            }
    };
    auto mma = [&](int buf) {
        u32x4 a[4];
#pragma unroll
        for (int j = 0; j < 4; ++j)
            a[j] = *(const u32x4*)(smem + LDS_A + wave * 8192 + buf * 4096 + j * 1024 + lane * 16);
#pragma unroll
        for (int ks = 0; ks < 2; ++ks) {
            const i32x8 xa = {(int)a[2 * ks][0], (int)a[2 * ks][1], (int)a[2 * ks][2], (int)a[2 * ks][3],
                              (int)a[2 * ks + 1][0], (int)a[2 * ks + 1][1], (int)a[2 * ks + 1][2], (int)a[2 * ks + 1][3]};
#pragma unroll
            for (int n = 0; n < 8; ++n)
                acc[n] = __builtin_amdgcn_mfma_scale_f32_16x16x128_f8f6f4(wf[ks][n], xa, acc[n], 0, 0, 0, 0, 0, 0);
        }
    };

    STAMP(1);
    bool pre = false;
    unsigned tm = 0;
    auto step = [&](int c, int buf, const bool more) {
        if (pre) wait_vm<4>(); else wait_vm<0>();
        __builtin_amdgcn_s_barrier();
        if (more) stage_w(c + 1);
        read_w(buf);
        __builtin_amdgcn_sched_barrier(0);
        const bool was_pre = pre;
        if (!was_pre) { wait_chunk(c); STAMP(32 + c); load_a(c); }
        pre = more && ((have >> (c + 1)) & 1);
        if (pre) load_a(c + 1);
        if (!more) tm = __hip_atomic_load(tmo, __ATOMIC_RELAXED, __HIP_MEMORY_SCOPE_AGENT);
        if (!more) wait_vm<0>();
        else if (was_pre && pre) wait_vm<8>();
        else if (was_pre || pre) wait_vm<4>();
        else wait_vm<0>();
        __builtin_amdgcn_sched_barrier(0);
        STAMP(48 + c);
        mma(buf);
        __builtin_amdgcn_sched_barrier(0);
        STAMP(2 + c);
    };
#pragma unroll 1
    for (int c = 0; c < NCH - 2; c += 2) {
        step(c, 0, true);
        step(c + 1, 1, true);
    }
    step(NCH - 2, 0, true);
    step(NCH - 1, 1, false);

    const bool bad = dead || (tm == call);
    const __amdgpu_buffer_rsrc_t orsrc = __builtin_amdgcn_make_buffer_rsrc((void*)outp, 0, NR * MS * NL * 4, 0x00020000);
    const int ooff = ((s * MS + b * 16 + i16) * NL + cb * 128 + G * 4) * 4;
#pragma unroll
    for (int n = 0; n < 8; ++n) {
        f32x4 v = acc[n] * scale;
        if (bad) v = f32x4{__builtin_nanf(""), __builtin_nanf(""), __builtin_nanf(""), __builtin_nanf("")};
        __builtin_amdgcn_raw_buffer_store_b128(__builtin_bit_cast(u32x4, v), orsrc, ooff + n * 64, 0, OUT_AUX);
    }
}

extern "C" void kernel_launch(int rank, int nranks, ncclComm_t comm,
                              hipStream_t stream,
                              void* const* d_in, const int* in_sizes, int n_in,
                              void* d_out, int out_size,
                              const int* in_dtypes, int out_dtype,
                              void* const* d_out_all,
                              void* const* d_ws_all, size_t ws_bytes) {
    (void)comm; (void)in_dtypes; (void)out_dtype; (void)d_out_all;
    if (nranks != NR || (unsigned)rank >= (unsigned)NR || n_in != 4 || in_sizes[0] != MS * KD ||
        in_sizes[1] != KD * NL || out_size != NR * MS * NL || ws_bytes < WS_NEED) {
        fprintf(stderr, "unexpected problem shape\n");
        return;
    }
    Params p;
    for (int j = 0; j < NR; ++j) p.ws[j] = (unsigned char*)d_ws_all[(rank + j) % NR];
    p.x = (const unsigned char*)d_in[0];
    p.w = (const unsigned char*)d_in[1];
    p.sx = (const float*)d_in[2];
    p.sw = (const float*)d_in[3];
    p.out = (float*)d_out;
    p.rank = rank;
    p.pad = 0;
    HIP_CHECK(hipFuncSetAttribute((const void*)ag_gemm, hipFuncAttributeMaxDynamicSharedMemorySize, LDS_BYTES));
    hipLaunchKernelGGL(ag_gemm, dim3(GRIDX), dim3(THREADS), LDS_BYTES, stream, p);
    HIP_CHECK(hipGetLastError());
}
```

```cpp
#include <hip/hip_runtime.h>
#include <rccl/rccl.h>
#include <cstdio>

#define HIP_CHECK(expr) do { hipError_t _e = (expr); if (_e != hipSuccess) { \
    fprintf(stderr, "HIP error %s at %s:%d\n", hipGetErrorString(_e), __FILE__, __LINE__); \
    return; } } while (0)

typedef unsigned u32x4 __attribute__((ext_vector_type(4)));
typedef unsigned short u16x2 __attribute__((ext_vector_type(2)));
typedef int i32x2 __attribute__((ext_vector_type(2)));
typedef int i32x8 __attribute__((ext_vector_type(8)));
typedef float f32x4 __attribute__((ext_vector_type(4)));
typedef unsigned long long u64;
#define LAS __attribute__((address_space(3)))

#define WIN 6
#define DEPTH 4
#define SPIN0 6
#define NFEED 4
#define CSLEEP 16
#define MPRIO 2
#define NLOAD 7
#define CDELAY 450
#define EARLY 1
#define OUT_AUX 17
#define ENC_LO 32
#define ENC_HI 480
#define GC 0
#define MSTR 32
#define POLL_SLEEP 2

constexpr int NR = 8;
constexpr int KD = 4096;
constexpr int MS = 512;
constexpr int NL = 512;
constexpr int NB16 = MS / 16;
constexpr int CHK = 256;
constexpr int NCH = KD / CHK;
constexpr int NU = NB16 * NCH;
constexpr int PPP = 8;
constexpr int RING = 16;
constexpr int GEMM0 = 7 * PPP;
constexpr int SR = 5;
constexpr int SL = 7;
constexpr int NRW = (7 * NB16 + SR - 1) / SR;
constexpr int NLW = (NB16 + SL - 1) / SL;
constexpr int NGEMM = 4 * (NRW + NLW);
constexpr int GRIDX = GEMM0 + NGEMM;
constexpr int THREADS = 64 * (5 + NLOAD);
constexpr int STAGE = CHK * 128;
constexpr int ARING = 15;
constexpr int LDS_A = 3 * STAGE;
constexpr int LDS_W = LDS_A + ARING * 4096 + 64;
constexpr int LDS_BYTES = LDS_W + 1024;
static_assert(ENC_LO % 2 == 0 && ENC_HI % 2 == 0, "pairs");
static_assert(GRIDX <= 256 && LDS_BYTES <= 163840 && ENC_HI - ENC_LO <= 4 * NGEMM, "resident grid");
static_assert(WIN >= 1 && WIN <= 32 && DEPTH >= 1 && DEPTH <= RING && NFEED <= RING && 3 + NFEED <= 5 + NLOAD, "push geometry");

constexpr size_t WS_FLAG = 0;
constexpr size_t WS_TMO = 64 << 10;
constexpr size_t WS_GFLAG = 72 << 10;
constexpr size_t WS_TICKET = 96 << 10;
constexpr size_t WS_CNT = 128 << 10;
constexpr size_t WS_META = 256 << 10;
constexpr size_t WS_GCNT = 384 << 10;
constexpr size_t WS_TRACE = 512 << 10;
constexpr size_t SLOT_BYTES = (size_t)MS * KD;
constexpr size_t WS_ENC = 2 << 20;
constexpr size_t WS_SLOT = 4 << 20;
constexpr size_t WS_NEED = WS_SLOT + 2 * NR * SLOT_BYTES;
constexpr unsigned INFO_RAW = 15u;
constexpr u64 DEADLINE_TICKS = 100000000ull;

struct Params {
    unsigned char* ws[NR];
    const unsigned char* x;
    const unsigned char* w;
    const float* sx;
    const float* sw;
    float* out;
    int rank;
    int pad;
};

#define STAMP(i) do {} while (0)
#define STAMP_OUT() do {} while (0)

template <int N> __device__ __forceinline__ void wait_vm() {
    asm volatile("s_waitcnt vmcnt(%0)" :: "i"(N) : "memory");
}
__device__ __forceinline__ void wait_lds() { asm volatile("s_waitcnt lgkmcnt(0)" ::: "memory"); }
__device__ __forceinline__ void glds16(const unsigned char* sbase, unsigned voff, unsigned lds) {
    unsigned keep;
    asm volatile("s_mov_b32 %0, m0\n\ts_mov_b32 m0, %3\n\ts_nop 0\n\tglobal_load_lds_dwordx4 %1, %2\n\ts_mov_b32 m0, %0"
                 : "=&s"(keep) : "v"(voff), "s"(sbase), "s"(lds) : "memory");
}
__device__ __forceinline__ void glds16_sys(const unsigned char* sbase, unsigned voff, unsigned lds) {
    unsigned keep;
    asm volatile("s_mov_b32 %0, m0\n\ts_mov_b32 m0, %3\n\ts_nop 0\n\tglobal_load_lds_dwordx4 %1, %2 sc0 sc1\n\ts_mov_b32 m0, %0"
                 : "=&s"(keep) : "v"(voff), "s"(sbase), "s"(lds) : "memory");
}
template <int OFF> __device__ __forceinline__ void store_sys(const u32x4& d, unsigned char* sbase, unsigned voff) {
    asm volatile("global_store_dwordx4 %1, %0, %2 offset:%3 sc0 sc1\n\ts_nop 1" :: "v"(d), "v"(voff), "s"(sbase), "i"(OFF) : "memory");
}
template <int OFF> __device__ __forceinline__ void store_sys_short(const u32x4& d, unsigned char* sbase, unsigned voff, unsigned drop) {
    unsigned t;
    asm volatile("s_nop 0\n\tv_readfirstlane_b32 %0, %5\n\ts_lshr_b64 exec, -1, %0\n\t"
                 "global_store_dwordx4 %2, %1, %3 offset:%4 sc0 sc1\n\ts_mov_b64 exec, -1\n\ts_nop 0"
                 : "=&s"(t) : "v"(d), "v"(voff), "s"(sbase), "i"(OFF), "v"(drop) : "memory", "scc");
}
__device__ __forceinline__ void put_word(unsigned char* sbase, unsigned zero, unsigned val) {
    asm volatile("s_mov_b64 exec, 1\n\tglobal_store_dword %0, %1, %2 sc0 sc1\n\ts_mov_b64 exec, -1\n\ts_nop 0"
                 :: "v"(zero), "v"(val), "s"(sbase) : "memory");
}
__device__ __forceinline__ void draw(unsigned& t, unsigned* ctr, unsigned zero, unsigned one) {
    asm volatile("s_mov_b64 exec, 1\n\tglobal_atomic_add %0, %1, %2, %3 sc0\n\ts_mov_b64 exec, -1"
                 : "+v"(t) : "v"(zero), "v"(one), "s"(ctr) : "memory");
}
template <int N> __device__ __forceinline__ unsigned take(unsigned& t) {
    asm volatile("s_waitcnt vmcnt(%1) ; take" : "+v"(t) : "i"(N) : "memory");
    return __builtin_amdgcn_readfirstlane(t);
}
__device__ __forceinline__ unsigned lds_get(unsigned char* a) {
    const unsigned v = __hip_atomic_load((LAS unsigned*)a, __ATOMIC_RELAXED, __HIP_MEMORY_SCOPE_WORKGROUP);
    asm volatile("" ::: "memory");
    return v;
}
__device__ __forceinline__ void lds_set(unsigned char* a, unsigned v) {
    asm volatile("" ::: "memory");
    __hip_atomic_store((LAS unsigned*)a, v, __ATOMIC_RELAXED, __HIP_MEMORY_SCOPE_WORKGROUP);
}

__device__ __forceinline__ unsigned scan_incl(unsigned v) {
    v += __builtin_amdgcn_update_dpp(0u, v, 0x111, 0xf, 0xf, false);
    v += __builtin_amdgcn_update_dpp(0u, v, 0x112, 0xf, 0xf, false);
    v += __builtin_amdgcn_update_dpp(0u, v, 0x114, 0xf, 0xf, false);
    v += __builtin_amdgcn_update_dpp(0u, v, 0x118, 0xf, 0xf, false);
    v += __builtin_amdgcn_update_dpp(0u, v, 0x142, 0xa, 0xf, false);
    v += __builtin_amdgcn_update_dpp(0u, v, 0x143, 0xc, 0xf, false);
    return v;
}
__device__ __forceinline__ int window_lo(int e0) { return e0 < 6 ? 0 : (e0 > 11 ? 5 : e0 - 6); }
__device__ __forceinline__ void decode_table(int e0, unsigned& tlo, unsigned& thi) {
    const int w0 = window_lo(e0);
    unsigned t[2] = {0u, 0u};
#pragma unroll
    for (int y = 0; y < 8; ++y) {
        const int z = w0 + y, e = z < e0 ? z : z + 3;
        t[y >> 2] |= (unsigned)(((e0 + 3) ^ e) << 3) << (8 * (y & 3));
    }
    tlo = t[0]; thi = t[1];
}
__device__ __forceinline__ void encode_table(int e0, unsigned (&t)[4]) {
    const int w0 = window_lo(e0);
    unsigned a[16];
#pragma unroll
    for (int e = 0; e < 16; ++e) {
        const int q = e - e0, y = (e < e0 ? e : e - 3) - w0;
        a[e] = (q >= 0 && q <= 2) ? (unsigned)(q << 3) : ((y >= 0 && y <= 7) ? (unsigned)(0x78 | y) : 0x98u);
    }
    auto sg = [](unsigned v) { return (v & 0x80u) ? 0xffu : 0u; };
#pragma unroll
    for (int i = 5; i < 8; ++i) { a[i] ^= 0xffu; a[8 + i] ^= 0xffu; }
    a[3] ^= sg(a[15]); a[2] ^= sg(a[13]); a[11] ^= sg(a[7]); a[10] ^= sg(a[5]);
    a[1] ^= sg(a[11]); a[9] ^= sg(a[3]);
    a[0] ^= sg(a[9]); a[8] ^= sg(a[1]);
#pragma unroll
    for (int i = 0; i < 4; ++i) t[i] = a[4 * i] | (a[4 * i + 1] << 8) | (a[4 * i + 2] << 16) | (a[4 * i + 3] << 24);
}
__device__ __forceinline__ unsigned encode_unit(const unsigned (&d)[16], const unsigned (&te)[4], unsigned char* scr,
                                                int lane, u32x4 (&o)[4]) {
    *(u32x4*)(scr + lane * 16) = u32x4{0u, 0u, 0u, 0u};
    unsigned es = 0;
#pragma unroll
    for (int i = 0; i < 16; i += 5) es = __builtin_amdgcn_sad_u8(d[i] & 0x78787878u, 0u, es);
    int e0 = ((int)__builtin_amdgcn_readlane(scan_incl(es), 63) - 1638) >> 13;
    e0 = e0 < 0 ? 0 : (e0 > 12 ? 12 : e0);
    const unsigned t0 = __builtin_amdgcn_readlane(te[0], e0), t1 = __builtin_amdgcn_readlane(te[1], e0);
    const unsigned t2 = __builtin_amdgcn_readlane(te[2], e0), t3 = __builtin_amdgcn_readlane(te[3], e0);
    unsigned c[16], bad = 0, nb[2];
    unsigned long long str[2];
#pragma unroll
    for (int h = 0; h < 2; ++h) {
        unsigned v[8], n[8];
#pragma unroll
        for (int k = 0; k < 8; ++k) {
            const unsigned x = d[8 * h + k];
            const unsigned e = (x >> 3) & 0x0f0f0f0fu;
            const unsigned inf = __builtin_amdgcn_perm(t1, t0, e) ^ __builtin_amdgcn_perm(t3, t2, e ^ 0x08080808u);
            c[8 * h + k] = (x & 0x87878787u) | (inf & 0x18181818u);
            bad |= inf;
            const unsigned sh = (inf >> 5) & 0x03030303u;
            const unsigned sy = inf & 0x07070707u;
            const u16x2 up = (__builtin_bit_cast(u16x2, sy) >> (unsigned short)8) << (__builtin_bit_cast(u16x2, sh) & (unsigned short)15);
            const unsigned m = (sy & 0x00ff00ffu) | __builtin_bit_cast(unsigned, up);
            v[k] = (m & 0xffffu) | ((m >> 16) << ((sh + (sh >> 8)) & 31u));
            n[k] = __builtin_amdgcn_sad_u8(sh, 0u, 0u);
        }
        unsigned long long acc = 0;
        unsigned off = 0;
#pragma unroll
        for (int k = 0; k < 8; k += 2) {
            acc |= (unsigned long long)(v[k] | (v[k + 1] << n[k])) << (off & 63u);
            off += n[k] + n[k + 1];
        }
        str[h] = acc; nb[h] = off;
    }
#pragma unroll
    for (int q = 0; q < 4; ++q) {
        const unsigned z = c[4 * q + 3];
        const unsigned p0 = c[4 * q] | ((z << 5) & 0x60606060u);
        const unsigned p1 = c[4 * q + 1] | ((z << 3) & 0x60606060u);
        const unsigned p2 = c[4 * q + 2] | ((z << 1) & 0x20202020u) | ((z >> 1) & 0x40404040u);
        o[(3 * q) >> 2][(3 * q) & 3] = p0; o[(3 * q + 1) >> 2][(3 * q + 1) & 3] = p1; o[(3 * q + 2) >> 2][(3 * q + 2) & 3] = p2;
    }
    const unsigned tot = nb[0] + nb[1];
    const unsigned inc = scan_incl(tot);
    const unsigned bits = __builtin_amdgcn_readlane(inc, 63);
    const bool fail = (bad & 0x80808080u) || nb[0] > 63u || nb[1] > 63u;
    if (__builtin_amdgcn_ballot_w64(fail) != 0 || bits > 8192u - 128u) return 0u;
#pragma unroll
    for (int h = 0; h < 2; ++h) {
        const unsigned ob = inc - tot + (h ? nb[0] : 0u);
        unsigned* wp = (unsigned*)(scr + (ob >> 5) * 4);
        const unsigned long long a = (unsigned long long)(unsigned)str[h] << (ob & 31u);
        const unsigned long long b = (unsigned long long)(unsigned)(str[h] >> 32) << (ob & 31u);
        __hip_atomic_fetch_or(wp, (unsigned)a, __ATOMIC_RELAXED, __HIP_MEMORY_SCOPE_WORKGROUP);
        __hip_atomic_fetch_or(wp + 1, (unsigned)(a >> 32) | (unsigned)b, __ATOMIC_RELAXED, __HIP_MEMORY_SCOPE_WORKGROUP);
        __hip_atomic_fetch_or(wp + 2, (unsigned)(b >> 32), __ATOMIC_RELAXED, __HIP_MEMORY_SCOPE_WORKGROUP);
    }
    asm volatile("s_waitcnt lgkmcnt(0)" ::: "memory");
    o[3] = *(const u32x4*)(scr + lane * 16);
    const unsigned nl = (bits + 127u) >> 7;
    return ((nl ? nl : 1u) << 4) | (unsigned)e0;
}
__device__ __forceinline__ void decode_unit(const unsigned char* slot, int lane, unsigned e0, unsigned tlo, unsigned thi,
                                            unsigned (&d)[16]) {
    unsigned p[12];
#pragma unroll
    for (int q = 0; q < 3; ++q) {
        const u32x4 v = *(const u32x4*)(slot + q * 1024 + lane * 16);
        p[4 * q] = v[0]; p[4 * q + 1] = v[1]; p[4 * q + 2] = v[2]; p[4 * q + 3] = v[3];
    }
#pragma unroll
    for (int q = 0; q < 4; ++q) {
        d[4 * q] = p[3 * q] & 0x9f9f9f9fu; d[4 * q + 1] = p[3 * q + 1] & 0x9f9f9f9fu; d[4 * q + 2] = p[3 * q + 2] & 0x9f9f9f9fu;
        d[4 * q + 3] = ((p[3 * q] >> 5) & 0x03030303u) | ((p[3 * q + 1] >> 3) & 0x0c0c0c0cu) |
                       ((p[3 * q + 2] >> 1) & 0x10101010u) | ((p[3 * q + 2] << 1) & 0x80808080u);
    }
    unsigned f[16], n0 = 0, n1 = 0;
#pragma unroll
    for (int i = 0; i < 16; ++i) {
        f[i] = d[i] & (d[i] >> 1) & 0x08080808u;
        if (i < 8) n0 += __builtin_popcount(f[i]); else n1 += __builtin_popcount(f[i]);
    }
    const unsigned n = n0 + n1;
    const unsigned o0 = (scan_incl(n) - n) * 3, o1 = o0 + n0 * 3;
    const unsigned e0rep = e0 * 0x08080808u;
#pragma unroll
    for (int h = 0; h < 2; ++h) {
        const unsigned o = h ? o1 : o0;
        const unsigned* wp = (const unsigned*)(slot + 3072 + ((o >> 5) & 255u) * 4);
        const unsigned w0 = wp[0], w1 = wp[1], w2 = wp[2];
        unsigned lo = __builtin_amdgcn_alignbit(w1, w0, o), hi = __builtin_amdgcn_alignbit(w2, w1, o);
#pragma unroll
        for (int i = 8 * h; i < 8 * h + 8; ++i) {
            const unsigned x = lo & 0xfffu;
            const unsigned y = x | (x << 10);
            const unsigned sy = (y & 0x00070007u) | ((y << 5) & 0x07000700u);
            const unsigned fx = __builtin_amdgcn_perm(thi, tlo, sy);
            const unsigned sel = __umul24(f[i], 0x202020u) + (f[i] >> 1);
            d[i] = (d[i] + e0rep) ^ __builtin_amdgcn_perm(fx, 0u, sel);
            if (i != 8 * h + 7) {
                const unsigned k = __builtin_popcount(f[i]) * 3;
                lo = __builtin_amdgcn_alignbit(hi, lo, k); hi >>= k;
            }
        }
    }
}

__global__ void __launch_bounds__(THREADS, 3) ag_gemm(Params p) {
    extern __shared__ __attribute__((aligned(16))) unsigned char smem[];
    const int lane = threadIdx.x & 63;
    const int wave = __builtin_amdgcn_readfirstlane(threadIdx.x >> 6);
    unsigned char* const myws = p.ws[0];
    const unsigned lds0 = (unsigned)(size_t)(LAS unsigned char*)smem;
    unsigned* const cnt = (unsigned*)(myws + WS_CNT + (size_t)(blockIdx.x * 16 + wave) * 32);
    const unsigned sv = lane * 16;
    STAMP(0);

    if (blockIdx.x < GEMM0) {
        constexpr unsigned POS_UNIT = 1u << 30, POS_END = 1u << 31;
        const u64 tstart = __builtin_amdgcn_s_memrealtime();
        const unsigned j = 1 + blockIdx.x % (NR - 1);
        const unsigned pk = blockIdx.x / (NR - 1);
        if (j >= (unsigned)NR || wave >= 3 + NFEED) return;
        unsigned char* const w_pos = smem + RING * 4096;
        unsigned char* const w_meta = w_pos + 4096;
        unsigned char* const w_cons = w_meta + NU * 4;
        unsigned char* const w_told = w_cons + 4;
        if (wave >= 3) {
            unsigned* ctr = (unsigned*)(myws + WS_TICKET + j * 128);
            const unsigned lv = (lane & 15) * KD + (lane >> 4) * 16;
            unsigned n = wave - 3;
            unsigned u = pk + PPP * n, m = 0;
            while (true) {
                const unsigned char* sb = m ? myws + WS_ENC + (size_t)u * 4096
                                            : p.x + (size_t)(u % NB16) * 16 * KD + (u / NB16) * CHK;
                const unsigned vo = m ? sv : lv, st = m ? 1024 : 64;
#pragma unroll
                for (int q = 0; q < 4; ++q) glds16_sys(sb + q * st, vo, lds0 + (n % RING) * 4096 + q * 1024);
                unsigned t = 0;
                if (EARLY) draw(t, ctr, 0u, 1u);
                if (n < NFEED) __builtin_amdgcn_s_barrier();
                wait_vm<EARLY>();
                lds_set(w_pos + n * 4, POS_UNIT | (u << 12) | (m ? (64u - (m >> 4)) | ((m & 15u) << 6) : INFO_RAW << 6));
                n += NFEED;
                if (EARLY) t = take<0>(t);
                while (n >= __builtin_amdgcn_readfirstlane(lds_get(w_cons)) + DEPTH) __builtin_amdgcn_s_sleep(1);
                if (!EARLY) { draw(t, ctr, 0u, 1u); t = take<0>(t); }
                u = PPP * NFEED + t % NU;
                if (u >= (unsigned)NU) { lds_set(w_pos + n * 4, POS_END); return; }
                m = __builtin_amdgcn_readfirstlane(lds_get(w_meta + u * 4));
                if (m == 0 && u >= ENC_LO && u < ENC_HI) {
                    const unsigned e = __builtin_amdgcn_readfirstlane(__hip_atomic_load(
                        (unsigned*)(myws + WS_META + (size_t)u * MSTR), __ATOMIC_RELAXED, __HIP_MEMORY_SCOPE_SYSTEM));
                    if ((e >> 12) == ((t / NU + 1) & 0xfffffu)) m = e & 0xfffu;
                    asm volatile("" ::: "memory");
                }
                if ((m >> 4) < 1u || (m >> 4) > 64u) m = 0;
            }
        }
        unsigned char* pws = p.ws[j];
        unsigned cv = 0;
        draw(cv, cnt, 0u, 1u);
#pragma unroll
        for (int q = 0; q < 3; ++q) *(LAS u32x4*)(w_pos + (wave * 3 + q) * 1024 + sv) = u32x4{0u, 0u, 0u, 0u};
        wait_lds();
        __builtin_amdgcn_s_barrier();
        const unsigned old = take<0>(cv), call = old + 1;
        if (wave == 0) {
            unsigned char* dst = pws + WS_SLOT + (size_t)((old & 1) * NR + p.rank) * SLOT_BYTES;
            unsigned char* const cons = w_cons, * const told = w_told;
            STAMP(1);
            constexpr int FJ = (3 + WIN) % 4, FB = (3 + WIN) / 4;
            static_assert(FB <= 3, "window");
            unsigned n = 0, live = (1u << NFEED) - 1u, dsc = 0;
            unsigned e1 = 0, e2 = 0, e3 = 0;
            auto look = [&]() { return __builtin_amdgcn_readfirstlane(lds_get(w_pos + n * 4)); };
            auto acquire = [&]() {
                while (live) {
                    if ((live >> (n % NFEED)) & 1u) {
                        dsc = look();
                        for (int k = 0; k < SPIN0 && !dsc; ++k) dsc = look();
                        if (!dsc) {
                            wait_vm<4>(); lds_set(told, e2); e3 = e2;
                            dsc = look();
                            if (!dsc) {
                                wait_vm<0>(); lds_set(told, e1); e3 = e2 = e1;
                                do dsc = look(); while (!dsc);
                            }
                        }
                        if (dsc & POS_UNIT) return true;
                        live &= ~(1u << (n % NFEED));
                    }
                    n += 1;
                    lds_set(cons, n);
                }
                return false;
            };
            u32x4 r[4], q[4];
            auto grab = [&](u32x4 (&z)[4]) {
#pragma unroll
                for (int k = 0; k < 4; ++k) z[k] = *(const u32x4*)(smem + (n % RING) * 4096 + k * 1024 + sv);
            };
            auto send = [&](u32x4 (&a)[4], u32x4 (&z)[4]) {
                const unsigned vo = (dsc & 0x1ff000u) | sv;
                const unsigned drop = dsc;
                const unsigned e0 = n + 1, rep = FB == 1 ? e1 : (FB == 2 ? e2 : e3);
                n += 1;
                unsigned w = lds_get(w_pos + n * 4);
                wait_vm<WIN - 1>(); if (FJ == 0) lds_set(told, rep); store_sys<0>(a[0], dst, vo);
                wait_vm<WIN - 1>(); if (FJ == 1) lds_set(told, rep); store_sys<1024>(a[1], dst, vo);
                asm volatile("" : "+v"(w));
                dsc = __builtin_amdgcn_readfirstlane(w);
                bool ok = (dsc & POS_UNIT) != 0;
                if (ok) grab(z);
                wait_vm<WIN - 1>(); if (FJ == 2) lds_set(told, rep); store_sys<2048>(a[2], dst, vo);
                wait_vm<WIN - 1>(); if (FJ == 3) lds_set(told, rep); store_sys_short<3072>(a[3], dst, vo, drop);
                e3 = e2; e2 = e1; e1 = e0;
                if (!ok) {
                    STAMP(3);
                    ok = acquire();
                    if (ok) grab(z);
                }
                lds_set(cons, n + 1);
                return ok;
            };
            if (acquire()) {
                grab(r); lds_set(cons, n + 1);
                while (send(r, q) && send(q, r)) {}
            }
            wait_vm<4>(); lds_set(told, e2);
            wait_vm<0>(); lds_set(told, e1 | 0x80000000u);
            STAMP(2);
            STAMP_OUT();
            return;
        }
        if (wave == 2) {
            constexpr int NQ = (ENC_HI - ENC_LO + 63) / 64;
            const __amdgpu_buffer_rsrc_t mr = __builtin_amdgcn_make_buffer_rsrc(myws + WS_META, 0, ENC_HI * MSTR, 0x00020000);
            while (__builtin_amdgcn_s_memrealtime() - tstart < CDELAY) __builtin_amdgcn_s_sleep(4);
            while (true) {
                unsigned e[NQ];
#pragma unroll
                for (int q = 0; q < NQ; ++q) e[q] = __builtin_amdgcn_raw_buffer_load_b32(mr, (ENC_LO + q * 64 + lane) * MSTR, 0, 17);
                bool ok = true;
#pragma unroll
                for (int q = 0; q < NQ; ++q) {
                    const unsigned u = ENC_LO + q * 64 + lane;
                    if ((e[q] >> 12) == (call & 0xfffffu) && u < (unsigned)ENC_HI) *(LAS unsigned*)(w_meta + u * 4) = (e[q] & 0xfffu) ? (e[q] & 0xfffu) : 1u;
                    else if (u < (unsigned)ENC_HI) ok = false;
                }
                if (__builtin_amdgcn_ballot_w64(!ok) == 0 || (__builtin_amdgcn_readfirstlane(lds_get(w_told)) >> 31)) break;
                __builtin_amdgcn_s_sleep(CSLEEP);
            }
            return;
        }
        unsigned char* fl = pws + WS_FLAG + (size_t)((old & 1) * NR + p.rank) * (NB16 * 128);
        unsigned char* gfl = pws + WS_GFLAG + (size_t)((old & 1) * NR + p.rank) * (4 * 128);
        unsigned char* gct = myws + WS_GCNT + (size_t)j * (NCH * 4 * 64);
        unsigned seen = 0;
        while (true) {
            const unsigned td = __builtin_amdgcn_readfirstlane(lds_get(w_told));
            const unsigned lim = td & 0x7fffffffu, n = seen + lane;
            if (seen >= lim) {
                if (td >> 31) break;
                __builtin_amdgcn_s_sleep(1);
                continue;
            }
            const unsigned dsc = n < lim ? lds_get(w_pos + n * 4) : 0u;
            if (dsc & POS_UNIT) {
                const unsigned u = (dsc >> 12) & 0x1ffu, nib = ((dsc >> 6) & 15u) << ((u & 7u) * 4);
                if (u >= GC * NB16) {
                    __hip_atomic_store((u64*)(fl + (u % NB16) * 128 + (u / NB16) * 8), ((u64)call << 32) | nib,
                                       __ATOMIC_RELAXED, __HIP_MEMORY_SCOPE_SYSTEM);
                } else {
                    u64* gc = (u64*)(gct + (u >> 3) * 64);
                    const u64 mine = (1ull << 32) | nib;
                    const u64 tot = __hip_atomic_fetch_add(gc, mine, __ATOMIC_RELAXED, __HIP_MEMORY_SCOPE_AGENT) + mine;
                    if ((tot >> 32) == 8u) {
                        __hip_atomic_store((u64*)(gfl + ((u >> 3) & 3u) * 128 + (u / NB16) * 8),
                                           ((u64)call << 32) | (unsigned)tot, __ATOMIC_RELAXED, __HIP_MEMORY_SCOPE_SYSTEM);
                        __hip_atomic_store(gc, 0ull, __ATOMIC_RELAXED, __HIP_MEMORY_SCOPE_AGENT);
                    }
                }
            }
            seen = seen + 64 < lim ? seen + 64 : lim;
        }
        STAMP(2);
        STAMP_OUT();
        return;
    }

    const int g = blockIdx.x - GEMM0;
    const int cb = g & 3, rg = g >> 2;
    const bool local = rg >= NRW;
    const int S = local ? (NB16 - (rg - NRW) + NLW - 1) / NLW : (7 * NB16 - rg + NRW - 1) / NRW;
    unsigned char* const w_item = smem + LDS_W;
    unsigned char* const w_land = w_item + 32;
    unsigned char* const w_got = w_item + 36;
    unsigned char* const w_used = w_item + 48;
    unsigned char* const w_read = w_item + 64;
    if (threadIdx.x < 32) *(LAS unsigned*)(w_item + threadIdx.x * 4) = 0u;
    wait_lds();
    __builtin_amdgcn_s_barrier();

    if (wave == 4 + NLOAD) {
        const unsigned wv0 = (lane >> 3) * NL + cb * 128 + (((lane & 7) ^ ((lane >> 4) & 3)) << 4);
        const unsigned wv1 = wv0 ^ 64;
        while (__builtin_amdgcn_readfirstlane(lds_get(w_got)) < (unsigned)NLOAD) __builtin_amdgcn_s_sleep(2);
        int slot = 0;
#pragma unroll 1
        for (int c = 0; c < NCH; ++c) {
            if (c >= 3) while (__builtin_amdgcn_ballot_w64(lds_get(w_read + (lane & 3) * 4) + 2 < (unsigned)c) != 0) __builtin_amdgcn_s_sleep(4);
            const unsigned char* sb = p.w + (size_t)c * (CHK * NL);
            const unsigned ld = lds0 + slot * STAGE;
#pragma unroll
            for (int i = 0; i < 32; ++i) glds16(sb + i * 8 * NL, (i & 2) ? wv1 : wv0, ld + i * 1024);
            wait_vm<0>();
            lds_set(w_land, c + 1);
            slot = slot == 2 ? 0 : slot + 1;
        }
        return;
    }
    if (wave >= 4) {
        const int d = wave - 4;
        unsigned cv = 0;
        draw(cv, cnt, 0u, 1u);
        const unsigned eu = ENC_LO + (unsigned)(d * NGEMM + g);
        const bool enc = eu < (unsigned)ENC_HI;
        unsigned xd[16], te[4], tlo, thi;
        if (enc) {
            const unsigned char* xs = p.x + (size_t)((eu % NB16) * 16 + (lane & 15)) * KD + (eu / NB16) * CHK + (lane >> 4) * 16;
#pragma unroll
            for (int q = 0; q < 4; ++q) {
                const u32x4 v = *(const u32x4*)(xs + q * 64);
                xd[4 * q] = v[0]; xd[4 * q + 1] = v[1]; xd[4 * q + 2] = v[2]; xd[4 * q + 3] = v[3];
            }
            encode_table(lane < 12 ? lane : 12, te);
        }
        const unsigned old = take<0>(cv), call = old + 1;
        if (lane == 0) __hip_atomic_fetch_add((LAS unsigned*)w_got, 1u, __ATOMIC_RELAXED, __HIP_MEMORY_SCOPE_WORKGROUP);
        if (enc) {
            u32x4 o[4];
            const unsigned m = encode_unit(xd, te, smem + LDS_A + d * 4096, lane, o);
            if (m) {
                unsigned char* eb = myws + WS_ENC + (size_t)eu * 4096;
                store_sys<0>(o[0], eb, sv); store_sys<1024>(o[1], eb, sv);
                store_sys<2048>(o[2], eb, sv); store_sys<3072>(o[3], eb, sv);
            }
            STAMP(20);
            wait_vm<0>();
            put_word(myws + WS_META + (size_t)eu * MSTR, 0u, (call << 12) | m);
            STAMP(21);
        }
        decode_table(lane < 12 ? lane : 12, tlo, thi);
        unsigned* tmo = (unsigned*)(myws + WS_TMO);
        const unsigned par = (old & 1) * NR;
        const unsigned lv = (lane & 15) * KD + (lane >> 4) * 16;
        const int nitem = S * NCH;
        unsigned dead = 0;
        struct Item { int i, c; unsigned s, b; };
        auto make = [&](int i, int c, int k) {
            Item t; t.i = i; t.c = c;
            const unsigned st = rg + k * NRW;
            t.s = (p.rank + 1 + st % 7) % NR; t.b = local ? (rg - NRW) + k * NLW : st / 7;
            return t;
        };
        auto flagp = [&](const Item& t) {
            return (const u64*)(t.c < GC ? myws + WS_GFLAG + (size_t)((par + t.s) * 4 + (t.b >> 3)) * 128
                                         : myws + WS_FLAG + (size_t)((par + t.s) * NB16 + t.b) * 128) + t.c;
        };
        auto room = [&](int i) {
            return __builtin_amdgcn_ballot_w64(lds_get(w_used + (lane & 3) * 4) + ARING <= (unsigned)i) == 0;
        };
        auto fetch = [&](const Item& t) {
            const unsigned ld = lds0 + LDS_A + (t.i % ARING) * 4096;
            const unsigned char* sb = local ? p.x + (size_t)t.b * 16 * KD + t.c * CHK
                                            : myws + WS_SLOT + (size_t)(par + t.s) * SLOT_BYTES + (size_t)(t.c * NB16 + t.b) * 4096;
            const unsigned vo = local ? lv : sv, st = local ? 64 : 1024;
#pragma unroll
            for (int q = 0; q < 4; ++q) glds16_sys(sb + q * st, vo, ld + q * 1024);
        };
        int c = 0, k = d;
#pragma unroll 1
        for (int i = d; i < nitem; i += NLOAD) {
            while (k >= S) { k -= S; ++c; }
            const Item t = make(i, c, k);
            k += NLOAD;
            unsigned info = INFO_RAW;
            while (!room(i)) __builtin_amdgcn_s_sleep(2);
            if (!local && !dead) {
                const u64 t0 = __builtin_amdgcn_s_memrealtime();
                unsigned it = 0;
                while (true) {
                    const u64 fv = __hip_atomic_load(flagp(t), __ATOMIC_RELAXED, __HIP_MEMORY_SCOPE_SYSTEM);
                    info = (__builtin_amdgcn_readfirstlane((unsigned)fv) >> ((t.b & 7u) * 4)) & 15u;
                    if (__builtin_amdgcn_readfirstlane((unsigned)(fv >> 32)) == call) break;
                    if ((++it & 255) == 0) {
                        const unsigned tv = __builtin_amdgcn_readfirstlane(
                            __hip_atomic_load(tmo, __ATOMIC_RELAXED, __HIP_MEMORY_SCOPE_AGENT));
                        if (tv == call || __builtin_amdgcn_s_memrealtime() - t0 > DEADLINE_TICKS) {
                            __hip_atomic_store(tmo, call, __ATOMIC_RELAXED, __HIP_MEMORY_SCOPE_AGENT);
                            dead = 0x80000000u;
                            break;
                        }
                    }
                    __builtin_amdgcn_s_sleep(POLL_SLEEP);
                }
                asm volatile("" ::: "memory");
            }
            STAMP(32 + t.c);
            if (!dead) fetch(t);
            if (t.c == NCH - 1 && __builtin_amdgcn_readfirstlane(
                    __hip_atomic_load(tmo, __ATOMIC_RELAXED, __HIP_MEMORY_SCOPE_AGENT)) == call) dead = 0x80000000u;
            wait_vm<0>();
            STAMP(48 + t.c);
            if (info <= 12u && !dead) {
                unsigned char* unit = smem + LDS_A + (i % ARING) * 4096;
                unsigned v[16];
                decode_unit(unit, lane, info, __builtin_amdgcn_readlane(tlo, info), __builtin_amdgcn_readlane(thi, info), v);
#pragma unroll
                for (int q = 0; q < 4; ++q)
                    *(u32x4*)(unit + q * 1024 + sv) = u32x4{v[4 * q], v[4 * q + 1], v[4 * q + 2], v[4 * q + 3]};
            }
            lds_set(w_item + d * 4, (i + 1) | dead);
            STAMP(2 + t.c);
        }
        STAMP_OUT();
        return;
    }

    __builtin_amdgcn_s_setprio(MPRIO);
    const float scale = p.sx[0] * p.sw[0];
    const __amdgpu_buffer_rsrc_t orsrc = __builtin_amdgcn_make_buffer_rsrc((void*)p.out, 0, NR * MS * NL * 4, 0x00020000);
    const int G = lane >> 4, i16 = lane & 15;
    const int ldsb = ((16 * G + (i16 >> 1)) * 128 + (((i16 >> 2) & 3) | ((G & 1) << 2)) * 16 + 8 * (i16 & 1)) ^ (wave << 5);
    const int ocol = ((i16 & 7) * NL + cb * 128 + wave * 32 + (i16 >> 3) * 16 + G * 4) * 4;
    unsigned ctl = 0;
    auto wait_ctl = [&](int l, unsigned v) {
        unsigned x = __builtin_amdgcn_readlane(ctl, l);
        while ((x & 0x7fffffffu) <= v) {
            ctl = lds_get(w_item + (lane & 15) * 4);
            x = __builtin_amdgcn_readlane(ctl, l);
            if ((x & 0x7fffffffu) <= v) __builtin_amdgcn_s_sleep(1);
        }
        return x;
    };
    i32x8 wf[2][2], wn[2][2];
    auto read_w = [&](int slot) {
#pragma unroll
        for (int ks = 0; ks < 2; ++ks)
#pragma unroll
            for (int n = 0; n < 2; ++n) {
                const unsigned char* lp = smem + slot * STAGE + ks * 16384 + (ldsb ^ (n << 4));
                i32x2 t0 = __builtin_amdgcn_ds_read_tr8_b64_v2i32((LAS i32x2*)(lp));
                i32x2 t1 = __builtin_amdgcn_ds_read_tr8_b64_v2i32((LAS i32x2*)(lp + 1024));
                i32x2 t2 = __builtin_amdgcn_ds_read_tr8_b64_v2i32((LAS i32x2*)(lp + 8192));
                i32x2 t3 = __builtin_amdgcn_ds_read_tr8_b64_v2i32((LAS i32x2*)(lp + 8192 + 1024));
                wn[ks][n] = i32x8{t0[0], t0[1], t1[0], t1[1], t2[0], t2[1], t3[0], t3[1]};
            }
    };
    f32x4 acc[SL][2];
#pragma unroll
    for (int k = 0; k < SL; ++k) acc[k][0] = acc[k][1] = f32x4{0.f, 0.f, 0.f, 0.f};
    asm volatile("" :: "v"(scale) : "memory");
    wait_ctl(8, 0u);
    read_w(0);
    STAMP(1);
    unsigned i = 0, as = 0, ld = 0, bad = 0;
    int slot = 0;
#pragma unroll 1
    for (int c = 0; c < NCH; ++c) {
#pragma unroll
        for (int ks = 0; ks < 2; ++ks) { wf[ks][0] = wn[ks][0]; wf[ks][1] = wn[ks][1]; }
        wait_lds();
        lds_set(w_read + wave * 4, c + 1);
        slot = slot == 2 ? 0 : slot + 1;
        if (c + 1 < NCH) { wait_ctl(8, c + 1); read_w(slot); }
#pragma unroll
        for (int k = 0; k < SL; ++k) {
            if (k < S) {
                const unsigned v = wait_ctl(ld, i);
                ld = ld + 1 == NLOAD ? 0 : ld + 1;
                bad |= v >> 31;
                if (c == NCH - 1) STAMP(32 + k);
                const unsigned char* unit = smem + LDS_A + as * 4096 + sv;
                const u32x4 a0 = *(const u32x4*)unit, a1 = *(const u32x4*)(unit + 1024);
                const u32x4 a2 = *(const u32x4*)(unit + 2048), a3 = *(const u32x4*)(unit + 3072);
                ++i; as = as + 1 == ARING ? 0 : as + 1;
                lds_set(w_used + wave * 4, i);
                __builtin_amdgcn_sched_barrier(0);
                const i32x8 xa0 = {(int)a0[0], (int)a0[1], (int)a0[2], (int)a0[3], (int)a1[0], (int)a1[1], (int)a1[2], (int)a1[3]};
                const i32x8 xa1 = {(int)a2[0], (int)a2[1], (int)a2[2], (int)a2[3], (int)a3[0], (int)a3[1], (int)a3[2], (int)a3[3]};
                acc[k][0] = __builtin_amdgcn_mfma_scale_f32_16x16x128_f8f6f4(wf[0][0], xa0, acc[k][0], 0, 0, 0, 0, 0, 0);
                acc[k][1] = __builtin_amdgcn_mfma_scale_f32_16x16x128_f8f6f4(wf[0][1], xa0, acc[k][1], 0, 0, 0, 0, 0, 0);
                acc[k][0] = __builtin_amdgcn_mfma_scale_f32_16x16x128_f8f6f4(wf[1][0], xa1, acc[k][0], 0, 0, 0, 0, 0, 0);
                acc[k][1] = __builtin_amdgcn_mfma_scale_f32_16x16x128_f8f6f4(wf[1][1], xa1, acc[k][1], 0, 0, 0, 0, 0, 0);
                if (c == NCH - 1) {
                    const unsigned st = rg + k * NRW;
                    const unsigned row = local ? p.rank * MS + ((rg - NRW) + k * NLW) * 16
                                               : ((p.rank + 1 + st % 7) % NR) * MS + (st / 7) * 16;
                    const float sc = bad ? __builtin_nanf("") : scale;
                    const u32x4 lo = __builtin_bit_cast(u32x4, acc[k][0] * sc), hi = __builtin_bit_cast(u32x4, acc[k][1] * sc);
                    u32x4 x, y;
#pragma unroll
                    for (int e = 0; e < 4; ++e) {
                        x[e] = __builtin_amdgcn_update_dpp(lo[e], hi[e], 0x128, 0xf, 0xc, false);
                        y[e] = __builtin_amdgcn_update_dpp(hi[e], lo[e], 0x128, 0xf, 0x3, false);
                    }
                    int o1 = ocol + row * (NL * 4), o2 = o1 + 8 * NL * 4;
                    asm volatile("" : "+v"(o1), "+v"(o2));
                    __builtin_amdgcn_raw_buffer_store_b128(x, orsrc, o1, 0, OUT_AUX);
                    __builtin_amdgcn_raw_buffer_store_b128(y, orsrc, o2, 0, OUT_AUX);
                }
            }
        }
        STAMP(2 + c);
    }
}

extern "C" void kernel_launch(int rank, int nranks, ncclComm_t comm,
                              hipStream_t stream,
                              void* const* d_in, const int* in_sizes, int n_in,
                              void* d_out, int out_size,
                              const int* in_dtypes, int out_dtype,
                              void* const* d_out_all,
                              void* const* d_ws_all, size_t ws_bytes) {
    (void)comm; (void)in_dtypes; (void)out_dtype; (void)d_out_all;
    if (nranks != NR || (unsigned)rank >= (unsigned)NR || n_in != 4 || in_sizes[0] != MS * KD ||
        in_sizes[1] != KD * NL || out_size != NR * MS * NL || ws_bytes < WS_NEED) {
        fprintf(stderr, "unexpected problem shape\n");
        return;
    }
    Params p;
    for (int j = 0; j < NR; ++j) p.ws[j] = (unsigned char*)d_ws_all[(rank + j) % NR];
    p.x = (const unsigned char*)d_in[0];
    p.w = (const unsigned char*)d_in[1];
    p.sx = (const float*)d_in[2];
    p.sw = (const float*)d_in[3];
    p.out = (float*)d_out;
    p.rank = rank;
    p.pad = 0;
    HIP_CHECK(hipFuncSetAttribute((const void*)ag_gemm, hipFuncAttributeMaxDynamicSharedMemorySize, LDS_BYTES));
    hipLaunchKernelGGL(ag_gemm, dim3(GRIDX), dim3(THREADS), LDS_BYTES, stream, p);
    HIP_CHECK(hipGetLastError());
}
```

```cpp
#include <hip/hip_runtime.h>
#include <rccl/rccl.h>
#include <cstdio>

#define HIP_CHECK(expr) do { hipError_t _e = (expr); if (_e != hipSuccess) { \
    fprintf(stderr, "HIP error %s at %s:%d\n", hipGetErrorString(_e), __FILE__, __LINE__); \
    return; } } while (0)

typedef unsigned u32x4 __attribute__((ext_vector_type(4)));
typedef unsigned short u16x2 __attribute__((ext_vector_type(2)));
typedef int i32x2 __attribute__((ext_vector_type(2)));
typedef int i32x8 __attribute__((ext_vector_type(8)));
typedef float f32x4 __attribute__((ext_vector_type(4)));
typedef unsigned long long u64;
#define LAS __attribute__((address_space(3)))

#define WIN 6
#define DEPTH 4
#define SPIN0 6
#define NFEED 3
#define CSLEEP 16
#define MPRIO 2
#define NLOAD 7
#define CDELAY 450
#define NWS 2
#define NSTG 1
#define RUSH 0
#define DIRECT 1
#define OUT_AUX 17
#define ENC_LO 32
#define ENC_HI 492
#define GC 10
#define MSTR 32
#define POLL_SLEEP 2

constexpr int NR = 8;
constexpr int KD = 4096;
constexpr int MS = 512;
constexpr int NL = 512;
constexpr int NB16 = MS / 16;
constexpr int CHK = 256;
constexpr int NCH = KD / CHK;
constexpr int NU = NB16 * NCH;
constexpr int PPP = 8;
constexpr int RING = 16;
constexpr int GEMM0 = 7 * PPP;
constexpr int NRW = 50;
constexpr int SL = 6;
constexpr int NGEMM = 4 * NRW;
constexpr int GRIDX = GEMM0 + NGEMM;
constexpr int THREADS = 64 * (4 + NLOAD + NSTG);
constexpr int STAGE = CHK * 128;
constexpr int ARING = NWS == 2 ? 19 : 11;
constexpr int LDS_T = 0;
constexpr int LDS_S = 16384;
constexpr int LDS_A = LDS_S + NWS * STAGE;
constexpr int LDS_W = LDS_A + ARING * 4096 + 64;
constexpr int LDS_BYTES = LDS_W + 256;
static_assert(ENC_LO % 2 == 0 && ENC_HI % 2 == 0, "pairs");
static_assert(GRIDX <= 256 && LDS_BYTES <= 163840 && ENC_HI - ENC_LO <= 4 * NGEMM, "resident grid");
static_assert(WIN >= 1 && WIN <= 32 && DEPTH >= 1 && DEPTH <= RING && NFEED <= RING && NFEED + 8 <= THREADS / 64 && THREADS <= 768 && NSTG <= 2 && NWS == 2, "push geometry");

__device__ __forceinline__ unsigned unit_at(unsigned q) {
    return (q & ~63u) + (q & 1u) * NB16 + ((q & 63u) >> 1);
}

constexpr size_t WS_FLAG = 0;
constexpr size_t WS_TMO = 64 << 10;
constexpr size_t WS_GFLAG = 72 << 10;
constexpr size_t WS_TICKET = 96 << 10;
constexpr size_t WS_CNT = 128 << 10;
constexpr size_t WS_META = 256 << 10;
constexpr size_t WS_GCNT = 384 << 10;
constexpr size_t WS_TRACE = 512 << 10;
constexpr size_t SLOT_BYTES = (size_t)MS * KD;
constexpr size_t WS_ENC = 2 << 20;
constexpr size_t WS_SLOT = 4 << 20;
constexpr size_t WS_NEED = WS_SLOT + 2 * NR * SLOT_BYTES;
constexpr unsigned INFO_RAW = 15u;
constexpr u64 DEADLINE_TICKS = 100000000ull;

struct Params {
    unsigned char* ws[NR];
    const unsigned char* x;
    const unsigned char* w;
    const float* sx;
    const float* sw;
    float* out;
    int rank;
    int pad;
};

#define STAMP(i) do {} while (0)
#define STAMP_OUT() do {} while (0)

template <int N> __device__ __forceinline__ void wait_vm() {
    asm volatile("s_waitcnt vmcnt(%0)" :: "i"(N) : "memory");
}
__device__ __forceinline__ void wait_lds() { asm volatile("s_waitcnt lgkmcnt(0)" ::: "memory"); }
__device__ __forceinline__ void glds16(const unsigned char* sbase, unsigned voff, unsigned lds) {
    unsigned keep;
    asm volatile("s_mov_b32 %0, m0\n\ts_mov_b32 m0, %3\n\ts_nop 0\n\tglobal_load_lds_dwordx4 %1, %2\n\ts_mov_b32 m0, %0"
                 : "=&s"(keep) : "v"(voff), "s"(sbase), "s"(lds) : "memory");
}
__device__ __forceinline__ void glds16_sys(const unsigned char* sbase, unsigned voff, unsigned lds) {
    unsigned keep;
    asm volatile("s_mov_b32 %0, m0\n\ts_mov_b32 m0, %3\n\ts_nop 0\n\tglobal_load_lds_dwordx4 %1, %2 sc0 sc1\n\ts_mov_b32 m0, %0"
                 : "=&s"(keep) : "v"(voff), "s"(sbase), "s"(lds) : "memory");
}
template <int OFF> __device__ __forceinline__ void store_sys(const u32x4& d, unsigned char* sbase, unsigned voff) {
    asm volatile("global_store_dwordx4 %1, %0, %2 offset:%3 sc0 sc1\n\ts_nop 1" :: "v"(d), "v"(voff), "s"(sbase), "i"(OFF) : "memory");
}
template <int OFF> __device__ __forceinline__ void store_sys_short(const u32x4& d, unsigned char* sbase, unsigned voff, unsigned drop) {
    unsigned t;
    asm volatile("s_nop 0\n\tv_readfirstlane_b32 %0, %5\n\ts_lshr_b64 exec, -1, %0\n\t"
                 "global_store_dwordx4 %2, %1, %3 offset:%4 sc0 sc1\n\ts_mov_b64 exec, -1\n\ts_nop 0"
                 : "=&s"(t) : "v"(d), "v"(voff), "s"(sbase), "i"(OFF), "v"(drop) : "memory", "scc");
}
__device__ __forceinline__ void put_word(unsigned char* sbase, unsigned zero, unsigned val) {
    asm volatile("s_mov_b64 exec, 1\n\tglobal_store_dword %0, %1, %2 sc0 sc1\n\ts_mov_b64 exec, -1\n\ts_nop 0"
                 :: "v"(zero), "v"(val), "s"(sbase) : "memory");
}
__device__ __forceinline__ void put_pair(unsigned char* sbase, unsigned lo, unsigned hi) {
    const u64 v = ((u64)hi << 32) | lo;
    asm volatile("s_mov_b64 exec, 1\n\tglobal_store_dwordx2 %0, %1, %2 sc0 sc1\n\ts_mov_b64 exec, -1\n\ts_nop 0"
                 :: "v"(0u), "v"(v), "s"(sbase) : "memory");
}
__device__ __forceinline__ void draw(unsigned& t, unsigned* ctr, unsigned zero, unsigned one) {
    asm volatile("s_mov_b64 exec, 1\n\tglobal_atomic_add %0, %1, %2, %3 sc0\n\ts_mov_b64 exec, -1"
                 : "+v"(t) : "v"(zero), "v"(one), "s"(ctr) : "memory");
}
template <int N> __device__ __forceinline__ unsigned take(unsigned& t) {
    asm volatile("s_waitcnt vmcnt(%1) ; take" : "+v"(t) : "i"(N) : "memory");
    return __builtin_amdgcn_readfirstlane(t);
}
__device__ __forceinline__ unsigned lds_get(unsigned char* a) {
    const unsigned v = __hip_atomic_load((LAS unsigned*)a, __ATOMIC_RELAXED, __HIP_MEMORY_SCOPE_WORKGROUP);
    asm volatile("" ::: "memory");
    return v;
}
__device__ __forceinline__ void lds_set(unsigned char* a, unsigned v) {
    asm volatile("" ::: "memory");
    __hip_atomic_store((LAS unsigned*)a, v, __ATOMIC_RELAXED, __HIP_MEMORY_SCOPE_WORKGROUP);
}

__device__ __forceinline__ unsigned scan_incl(unsigned v) {
    v += __builtin_amdgcn_update_dpp(0u, v, 0x111, 0xf, 0xf, false);
    v += __builtin_amdgcn_update_dpp(0u, v, 0x112, 0xf, 0xf, false);
    v += __builtin_amdgcn_update_dpp(0u, v, 0x114, 0xf, 0xf, false);
    v += __builtin_amdgcn_update_dpp(0u, v, 0x118, 0xf, 0xf, false);
    v += __builtin_amdgcn_update_dpp(0u, v, 0x142, 0xa, 0xf, false);
    v += __builtin_amdgcn_update_dpp(0u, v, 0x143, 0xc, 0xf, false);
    return v;
}
__device__ __forceinline__ unsigned symbol_entry(unsigned x) {
    unsigned v = 0;
#pragma unroll
    for (int i = 0; i < 4; ++i) { const unsigned y = (x >> (3 * i)) & 7u; v |= ((y < 6u ? y : y + 3u) << 3) << (8 * i); }
    return v;
}
__device__ __forceinline__ void encode_table(int e0, unsigned (&t)[4]) {
    unsigned a[16];
#pragma unroll
    for (int e = 0; e < 16; ++e) {
        const int q = e - e0, y = q < 0 ? q + 6 : q + 3;
        a[e] = (q >= 0 && q <= 2) ? (unsigned)((q + 1) << 3) : ((q >= -6 && q <= 4) ? (unsigned)(0x60 | y) : 0x80u);
    }
    auto sg = [](unsigned v) { return (v & 0x80u) ? 0xffu : 0u; };
#pragma unroll
    for (int i = 5; i < 8; ++i) { a[i] ^= 0xffu; a[8 + i] ^= 0xffu; }
    a[3] ^= sg(a[15]); a[2] ^= sg(a[13]); a[11] ^= sg(a[7]); a[10] ^= sg(a[5]);
    a[1] ^= sg(a[11]); a[9] ^= sg(a[3]);
    a[0] ^= sg(a[9]); a[8] ^= sg(a[1]);
#pragma unroll
    for (int i = 0; i < 4; ++i) t[i] = a[4 * i] | (a[4 * i + 1] << 8) | (a[4 * i + 2] << 16) | (a[4 * i + 3] << 24);
}
__device__ __forceinline__ unsigned encode_unit(const unsigned (&d)[16], const unsigned (&te)[4], unsigned char* scr,
                                                int lane, u32x4 (&o)[4]) {
    *(u32x4*)(scr + lane * 16) = u32x4{0u, 0u, 0u, 0u};
    unsigned es = 0;
#pragma unroll
    for (int i = 0; i < 16; i += 5) es = __builtin_amdgcn_sad_u8(d[i] & 0x78787878u, 0u, es);
    int e0 = ((int)__builtin_amdgcn_readlane(scan_incl(es), 63) - 1638) >> 13;
    e0 = e0 < 0 ? 0 : (e0 > 13 ? 13 : e0);
    const unsigned t0 = __builtin_amdgcn_readlane(te[0], e0), t1 = __builtin_amdgcn_readlane(te[1], e0);
    const unsigned t2 = __builtin_amdgcn_readlane(te[2], e0), t3 = __builtin_amdgcn_readlane(te[3], e0);
    unsigned c[16], bad = 0, nb[2];
    unsigned long long str[2];
#pragma unroll
    for (int h = 0; h < 2; ++h) {
        unsigned v[8], n[8];
#pragma unroll
        for (int k = 0; k < 8; ++k) {
            const unsigned x = d[8 * h + k];
            const unsigned e = (x >> 3) & 0x0f0f0f0fu;
            const unsigned inf = __builtin_amdgcn_perm(t1, t0, e) ^ __builtin_amdgcn_perm(t3, t2, e ^ 0x08080808u);
            c[8 * h + k] = (x & 0x87878787u) | (inf & 0x18181818u);
            bad |= inf;
            const unsigned sh = (inf >> 5) & 0x03030303u;
            const unsigned sy = inf & 0x07070707u;
            const u16x2 up = (__builtin_bit_cast(u16x2, sy) >> (unsigned short)8) << (__builtin_bit_cast(u16x2, sh) & (unsigned short)15);
            const unsigned m = (sy & 0x00ff00ffu) | __builtin_bit_cast(unsigned, up);
            v[k] = (m & 0xffffu) | ((m >> 16) << ((sh + (sh >> 8)) & 31u));
            n[k] = __builtin_amdgcn_sad_u8(sh, 0u, 0u);
        }
        unsigned long long acc = 0;
        unsigned off = 0;
#pragma unroll
        for (int k = 0; k < 8; k += 2) {
            acc |= (unsigned long long)(v[k] | (v[k + 1] << n[k])) << (off & 63u);
            off += n[k] + n[k + 1];
        }
        str[h] = acc; nb[h] = off;
    }
#pragma unroll
    for (int q = 0; q < 4; ++q) {
        const unsigned z = c[4 * q + 3];
        const unsigned p0 = c[4 * q] | ((z << 5) & 0x60606060u);
        const unsigned p1 = c[4 * q + 1] | ((z << 3) & 0x60606060u);
        const unsigned p2 = c[4 * q + 2] | ((z << 1) & 0x20202020u) | ((z >> 1) & 0x40404040u);
        o[(3 * q) >> 2][(3 * q) & 3] = p0; o[(3 * q + 1) >> 2][(3 * q + 1) & 3] = p1; o[(3 * q + 2) >> 2][(3 * q + 2) & 3] = p2;
    }
    const unsigned tot = nb[0] + nb[1];
    const unsigned inc = scan_incl(tot);
    const unsigned bits = __builtin_amdgcn_readlane(inc, 63);
    const bool fail = (bad & 0x80808080u) || nb[0] > 60u || nb[1] > 60u;
    if (__builtin_amdgcn_ballot_w64(fail) != 0 || bits > 8192u - 128u) return 0u;
#pragma unroll
    for (int h = 0; h < 2; ++h) {
        const unsigned ob = inc - tot + (h ? nb[0] : 0u);
        unsigned* wp = (unsigned*)(scr + (ob >> 5) * 4);
        const unsigned long long a = (unsigned long long)(unsigned)str[h] << (ob & 31u);
        const unsigned long long b = (unsigned long long)(unsigned)(str[h] >> 32) << (ob & 31u);
        __hip_atomic_fetch_or(wp, (unsigned)a, __ATOMIC_RELAXED, __HIP_MEMORY_SCOPE_WORKGROUP);
        __hip_atomic_fetch_or(wp + 1, (unsigned)(a >> 32) | (unsigned)b, __ATOMIC_RELAXED, __HIP_MEMORY_SCOPE_WORKGROUP);
        __hip_atomic_fetch_or(wp + 2, (unsigned)(b >> 32), __ATOMIC_RELAXED, __HIP_MEMORY_SCOPE_WORKGROUP);
    }
    asm volatile("s_waitcnt lgkmcnt(0)" ::: "memory");
    o[3] = *(const u32x4*)(scr + lane * 16);
    const unsigned nl = (bits + 127u) >> 7;
    return ((nl ? nl : 1u) << 4) | (unsigned)e0;
}
__device__ __forceinline__ void decode_unit(const u32x4 (&pk)[3], const unsigned char* slot, unsigned e0, unsigned (&d)[16]) {
    unsigned p[12];
#pragma unroll
    for (int q = 0; q < 12; ++q) p[q] = pk[q >> 2][q & 3];
#pragma unroll
    for (int q = 0; q < 4; ++q) {
        d[4 * q] = p[3 * q] & 0x9f9f9f9fu; d[4 * q + 1] = p[3 * q + 1] & 0x9f9f9f9fu; d[4 * q + 2] = p[3 * q + 2] & 0x9f9f9f9fu;
        d[4 * q + 3] = ((p[3 * q] >> 5) & 0x03030303u) | ((p[3 * q + 1] >> 3) & 0x0c0c0c0cu) |
                       ((p[3 * q + 2] >> 1) & 0x10101010u) | ((p[3 * q + 2] << 1) & 0x80808080u);
    }
    unsigned f[16], c[2][9];
    c[0][0] = c[1][0] = 0;
#pragma unroll
    for (int i = 0; i < 16; ++i) {
        f[i] = ~(d[i] | (d[i] >> 1)) & 0x08080808u;
        c[i >> 3][(i & 7) + 1] = __builtin_popcount(f[i]) + c[i >> 3][i & 7];
    }
    const unsigned n = c[0][8] + c[1][8];
    const unsigned o0 = (scan_incl(n) - n) * 3 + 30, o1 = o0 + c[0][8] * 3;
    const unsigned bias = (e0 - 6u) * 0x08080808u;
    unsigned y[16];
#pragma unroll
    for (int h = 0; h < 2; ++h) {
        const unsigned o = h ? o1 : o0;
        const unsigned* wp = (const unsigned*)(slot + 3068 + ((o >> 5) & 255u) * 4);
        const unsigned w0 = wp[0], w1 = wp[1], w2 = wp[2];
        const u64 win = __builtin_amdgcn_alignbit(w1, w0, o) | ((u64)__builtin_amdgcn_alignbit(w2, w1, o) << 32);
#pragma unroll
        for (int i = 0; i < 8; ++i)
            y[8 * h + i] = *(const LAS unsigned*)(size_t)((unsigned)(win >> ((c[h][i] * 3) & 63u)) & 0x3ffcu);
    }
#pragma unroll
    for (int i = 0; i < 16; ++i) {
        const unsigned sel = __umul24(f[i], 0x202020u) + (f[i] >> 1);
        d[i] = d[i] + __builtin_amdgcn_perm(y[i], 0x28282828u, sel) + bias;
    }
}

__global__ void __launch_bounds__(THREADS, 3) ag_gemm(Params p) {
    extern __shared__ __attribute__((aligned(16))) unsigned char smem[];
    const int lane = threadIdx.x & 63;
    const int wave = __builtin_amdgcn_readfirstlane(threadIdx.x >> 6);
    const unsigned j = 1 + blockIdx.x % (NR - 1);
    unsigned char* const myws = p.ws[0];
    unsigned char* const pws = p.ws[j];
    const unsigned char* const xp = p.x;
    const unsigned rank = p.rank;
    asm volatile("" :: "s"(myws), "s"(pws), "s"(xp), "s"(rank));
    const unsigned lds0 = (unsigned)(size_t)(LAS unsigned char*)smem;
    unsigned* const cnt = (unsigned*)(myws + WS_CNT + (size_t)(blockIdx.x * 16 + wave) * 32);
    const unsigned sv = lane * 16;
    STAMP(0);

    if (__builtin_expect(blockIdx.x < GEMM0, 1)) {
        constexpr unsigned POS_LAST = 1u << 29, POS_UNIT = 1u << 30, POS_END = 1u << 31;
        const unsigned pk = blockIdx.x / (NR - 1);
        const unsigned lv = (lane & 15) * KD + (lane >> 4) * 16;
        unsigned char* const w_pos = smem + RING * 4096;
        unsigned char* const w_meta = w_pos + 4096;
        unsigned char* const w_cons = w_meta + NU * 4;
        unsigned char* const w_told = w_cons + 4;
        constexpr unsigned NFIX = PPP * (NFEED + 1), NDRAW = NU - NFIX + PPP * NFEED;
        const bool feeder = wave >= 1 && wave <= NFEED;
        unsigned* const ctr = feeder ? (unsigned*)(myws + WS_TICKET + j * 128) : cnt;
        unsigned u = unit_at(pk + PPP * wave);
        if (wave <= NFEED) {
            const unsigned char* sb = xp + (size_t)(u % NB16) * 16 * KD + (u / NB16) * CHK;
#pragma unroll
            for (int q = 0; q < 4; ++q) glds16(sb + q * 64, lv, lds0 + wave * 4096 + q * 1024);
        } else {
            if (wave - NFEED - 1 < 7) *(LAS u32x4*)(w_pos + (wave - NFEED - 1) * 1024 + sv) = u32x4{0u, 0u, 0u, 0u};
            wait_lds();
        }
        if (j >= (unsigned)NR || wave > NFEED + 2) return;
        unsigned cv = 0;
        draw(cv, ctr, 0u, 1u);
        const u64 tstart = __builtin_amdgcn_s_memrealtime();
        __builtin_amdgcn_s_barrier();
        unsigned t = take<0>(cv);
        if (feeder) {
            unsigned n = wave, m = 0;
            while (true) {
                const unsigned o = NFIX + t % NDRAW;
                lds_set(w_pos + n * 4, POS_UNIT | (o >= NU - RUSH ? POS_LAST : 0u) | (u << 12) |
                                       (m ? (64u - (m >> 4)) | ((m & 15u) << 6) : INFO_RAW << 6));
                n += NFEED;
                if (o >= (unsigned)NU) { lds_set(w_pos + n * 4, POS_END); return; }
                u = unit_at(o);
                while (n >= __builtin_amdgcn_readfirstlane(lds_get(w_cons)) + DEPTH) __builtin_amdgcn_s_sleep(1);
                m = __builtin_amdgcn_readfirstlane(lds_get(w_meta + o * 4));
                if (m == 0 && o >= ENC_LO && o < ENC_HI) {
                    const unsigned e = __builtin_amdgcn_readfirstlane(__hip_atomic_load(
                        (unsigned*)(myws + WS_META + (size_t)o * MSTR), __ATOMIC_RELAXED, __HIP_MEMORY_SCOPE_SYSTEM));
                    if ((e >> 12) == ((t / NDRAW + 1) & 0xfffffu)) m = e & 0xfffu;
                    asm volatile("" ::: "memory");
                }
                if ((m >> 4) < 1u || (m >> 4) > 64u) m = 0;
                unsigned tv = 0;
                draw(tv, ctr, 0u, 1u);
                const unsigned char* sb = m ? myws + WS_ENC + (size_t)u * 4096
                                            : xp + (size_t)(u % NB16) * 16 * KD + (u / NB16) * CHK;
                const unsigned vo = m ? sv : lv, st = m ? 1024 : 64;
#pragma unroll
                for (int q = 0; q < 4; ++q) glds16_sys(sb + q * st, vo, lds0 + (n % RING) * 4096 + q * 1024);
                t = take<0>(tv);
            }
        }
        const unsigned old = t, call = old + 1;
        if (__builtin_expect(wave == 0, 1)) {
            unsigned char* dst = pws + WS_SLOT + (size_t)((old & 1) * NR + rank) * SLOT_BYTES;
            unsigned char* const cons = w_cons, * const told = w_told;
            STAMP(1);
            constexpr int FJ = (3 + WIN) % 4, FB = (3 + WIN) / 4;
            static_assert(FB <= 3, "window");
            unsigned n = 0, live = (1u << NFEED) - 1u, dsc = POS_UNIT | (u << 12) | (INFO_RAW << 6);
            unsigned e1 = 0, e2 = 0, e3 = 0;
            unsigned d1 = 0, d2 = 0;
            auto look = [&]() { return __builtin_amdgcn_readfirstlane(lds_get(w_pos + n * 4)); };
            auto acquire = [&]() {
                while (live) {
                    if ((live >> (n % NFEED)) & 1u) {
                        dsc = look();
                        for (int k = 0; k < SPIN0 && !dsc; ++k) dsc = look();
                        if (!dsc) {
                            wait_vm<4>(); lds_set(told, e2); e3 = e2;
                            dsc = look();
                            if (!dsc) {
                                wait_vm<0>(); lds_set(told, e1); e3 = e2 = e1;
                                do dsc = look(); while (!dsc);
                            }
                        }
                        if (dsc & POS_UNIT) return true;
                        live &= ~(1u << (n % NFEED));
                    }
                    n += 1;
                    lds_set(cons, n);
                }
                return false;
            };
            u32x4 r[4], q[4];
            auto grab = [&](u32x4 (&z)[4]) {
#pragma unroll
                for (int k = 0; k < 4; ++k) z[k] = *(const u32x4*)(smem + (n % RING) * 4096 + k * 1024 + sv);
            };
            auto send = [&](u32x4 (&a)[4], u32x4 (&z)[4]) {
                const unsigned vo = (dsc & 0x1ff000u) | sv;
                const unsigned drop = dsc;
                const unsigned e0 = n + 1, rep = FB == 1 ? e1 : (FB == 2 ? e2 : e3);
                n += 1;
                unsigned w = lds_get(w_pos + n * 4);
                const bool pace = !(dsc & POS_LAST);
                auto point = [&](int jj) { if (jj == FJ) { wait_vm<WIN - 1>(); lds_set(told, rep); } else if (pace) wait_vm<WIN - 1>(); };
                d2 = d1; d1 = dsc;
                point(0); store_sys<0>(a[0], dst, vo);
                point(1); store_sys<1024>(a[1], dst, vo);
                asm volatile("" : "+v"(w));
                dsc = __builtin_amdgcn_readfirstlane(w);
                bool ok = (dsc & POS_UNIT) != 0;
                if (ok) grab(z);
                point(2); store_sys<2048>(a[2], dst, vo);
                point(3); store_sys_short<3072>(a[3], dst, vo, drop);
                e3 = e2; e2 = e1; e1 = e0;
                if (!ok) {
                    STAMP(3);
                    ok = acquire();
                    if (ok) grab(z);
                }
                lds_set(cons, n + 1);
                return ok;
            };
            STAMP(5);
            grab(r); lds_set(w_pos, dsc); lds_set(cons, 1u);
            while (send(r, q) && send(q, r)) {}
            const unsigned u1 = (d1 >> 12) & 0x1ffu, u2 = (d2 >> 12) & 0x1ffu;
            if (DIRECT && FB == 2 && e3 < e2 && e2 < e1 && u1 >= GC * NB16 && u2 >= GC * NB16) {
                unsigned char* fl = pws + WS_FLAG + (size_t)((old & 1) * NR + rank) * (NB16 * 128);
                lds_set(told, e3 | 0x80000000u);
                wait_vm<4>();
                put_pair(fl + (u2 % NB16) * 128 + (u2 / NB16) * 8, ((d2 >> 6) & 15u) << ((u2 & 7u) * 4), call);
                wait_vm<1>();
                put_pair(fl + (u1 % NB16) * 128 + (u1 / NB16) * 8, ((d1 >> 6) & 15u) << ((u1 & 7u) * 4), call);
            } else {
                wait_vm<4>(); lds_set(told, e2);
                wait_vm<0>(); lds_set(told, e1 | 0x80000000u);
            }
            STAMP(2);
            STAMP_OUT();
            return;
        }
        if (wave == NFEED + 2) {
            constexpr int NQ = (ENC_HI - ENC_LO + 63) / 64;
            const __amdgpu_buffer_rsrc_t mr = __builtin_amdgcn_make_buffer_rsrc(myws + WS_META, 0, ENC_HI * MSTR, 0x00020000);
            while (__builtin_amdgcn_s_memrealtime() - tstart < CDELAY) __builtin_amdgcn_s_sleep(4);
            while (true) {
                unsigned e[NQ];
#pragma unroll
                for (int q = 0; q < NQ; ++q) e[q] = __builtin_amdgcn_raw_buffer_load_b32(mr, (ENC_LO + q * 64 + lane) * MSTR, 0, 17);
                bool ok = true;
#pragma unroll
                for (int q = 0; q < NQ; ++q) {
                    const unsigned u = ENC_LO + q * 64 + lane;
                    if ((e[q] >> 12) == (call & 0xfffffu) && u < (unsigned)ENC_HI) *(LAS unsigned*)(w_meta + u * 4) = (e[q] & 0xfffu) ? (e[q] & 0xfffu) : 1u;
                    else if (u < (unsigned)ENC_HI) ok = false;
                }
                if (__builtin_amdgcn_ballot_w64(!ok) == 0 || (__builtin_amdgcn_readfirstlane(lds_get(w_told)) >> 31)) break;
                __builtin_amdgcn_s_sleep(CSLEEP);
            }
            return;
        }
        unsigned char* fl = pws + WS_FLAG + (size_t)((old & 1) * NR + rank) * (NB16 * 128);
        unsigned char* gfl = pws + WS_GFLAG + (size_t)((old & 1) * NR + rank) * (4 * 128);
        unsigned char* gct = myws + WS_GCNT + (size_t)j * (NCH * 4 * 64);
        unsigned seen = 0;
        while (true) {
            const unsigned td = __builtin_amdgcn_readfirstlane(lds_get(w_told));
            const unsigned lim = td & 0x7fffffffu, n = seen + lane;
            if (seen >= lim) {
                if (td >> 31) break;
                __builtin_amdgcn_s_sleep(1);
                continue;
            }
            const unsigned dsc = n < lim ? lds_get(w_pos + n * 4) : 0u;
            if (dsc & POS_UNIT) {
                const unsigned u = (dsc >> 12) & 0x1ffu, nib = ((dsc >> 6) & 15u) << ((u & 7u) * 4);
                if (u >= GC * NB16) {
                    __hip_atomic_store((u64*)(fl + (u % NB16) * 128 + (u / NB16) * 8), ((u64)call << 32) | nib,
                                       __ATOMIC_RELAXED, __HIP_MEMORY_SCOPE_SYSTEM);
                } else {
                    u64* gc = (u64*)(gct + (u >> 3) * 64);
                    const u64 mine = (1ull << 32) | nib;
                    const u64 tot = __hip_atomic_fetch_add(gc, mine, __ATOMIC_RELAXED, __HIP_MEMORY_SCOPE_AGENT) + mine;
                    if ((tot >> 32) == 8u) {
                        __hip_atomic_store((u64*)(gfl + ((u >> 3) & 3u) * 128 + (u / NB16) * 8),
                                           ((u64)call << 32) | (unsigned)tot, __ATOMIC_RELAXED, __HIP_MEMORY_SCOPE_SYSTEM);
                        __hip_atomic_store(gc, 0ull, __ATOMIC_RELAXED, __HIP_MEMORY_SCOPE_AGENT);
                    }
                }
            }
            seen = seen + 64 < lim ? seen + 64 : lim;
        }
        STAMP(2);
        STAMP_OUT();
        return;
    }

    if (lds0 != 0) return;
    const int g = blockIdx.x - GEMM0;
    const int cb = g & 3, rg = g >> 2;
    static_assert(7 * NB16 == 4 * NRW + 24 && NB16 == 26 + 6, "block deal");
    const int nloc = rg < 24 ? 0 : (rg < 44 ? 1 : 2), S = rg < 44 ? 5 : 6;
    auto block_of = [&](unsigned k, unsigned& src, unsigned& b) {
        const unsigned t = rg + (k - nloc) * NRW;
        const bool own = k < (unsigned)nloc;
        src = own ? 0u : 1u + t % 7u;
        b = own ? (k ? rg - 18 : rg - 24) : t / 7u;
    };
    unsigned char* const w_item = smem + LDS_W;
    unsigned char* const w_land = w_item + 96;
    unsigned char* const w_got = w_item + 104;
    unsigned char* const w_tab = w_item + 108;
    unsigned char* const w_next = w_item + 112;
    unsigned char* const w_used = w_item + 128;
    unsigned char* const w_read = w_item + 144;
    if (threadIdx.x < 64) *(LAS unsigned*)(w_item + threadIdx.x * 4) = 0u;
    wait_lds();
    __builtin_amdgcn_s_barrier();

    if (wave >= 4 + NLOAD) {
        const int h = wave - 4 - NLOAD;
        const unsigned wv0 = (h * (256 / NSTG) + (lane >> 3)) * NL + cb * 128 + (((lane & 7) ^ ((lane >> 4) & 3)) << 4);
        const unsigned wv1 = wv0 ^ 64;
        while (__builtin_amdgcn_readfirstlane(lds_get(w_got)) < (unsigned)NLOAD) __builtin_amdgcn_s_sleep(2);
        int slot = 0;
#pragma unroll 1
        for (int c = 0; c < NCH; ++c) {
            if (c >= NWS) while (__builtin_amdgcn_ballot_w64(lds_get(w_read + (lane & 3) * 4) + (NWS - 1) < (unsigned)c) != 0) __builtin_amdgcn_s_sleep(2);
            const unsigned char* sb = p.w + (size_t)c * (CHK * NL);
            const unsigned ld = lds0 + LDS_S + slot * STAGE + h * (STAGE / NSTG);
#pragma unroll
            for (int i = 0; i < 32 / NSTG; ++i) glds16(sb + i * 8 * NL, (i & 2) ? wv1 : wv0, ld + i * 1024);
            wait_vm<0>();
            lds_set(w_land + h * 4, c + 1);
            slot = slot == NWS - 1 ? 0 : slot + 1;
        }
        return;
    }
    if (wave >= 4) {
        const int d = wave - 4;
        unsigned cv = 0;
        draw(cv, cnt, 0u, 1u);
        const unsigned eo = ENC_LO + (unsigned)(d * NGEMM + g), eu = unit_at(eo) % NU;
        const bool enc = eo < (unsigned)ENC_HI;
        unsigned xd[16], te[4];
        if (enc) {
            const unsigned char* xs = xp + (size_t)((eu % NB16) * 16 + (lane & 15)) * KD + (eu / NB16) * CHK + (lane >> 4) * 16;
#pragma unroll
            for (int q = 0; q < 4; ++q) {
                const u32x4 v = *(const u32x4*)(xs + q * 64);
                xd[4 * q] = v[0]; xd[4 * q + 1] = v[1]; xd[4 * q + 2] = v[2]; xd[4 * q + 3] = v[3];
            }
            encode_table(lane < 13 ? lane : 13, te);
        }
        const unsigned old = take<0>(cv), call = old + 1;
        if (lane == 0) __hip_atomic_fetch_add((LAS unsigned*)w_got, 1u, __ATOMIC_RELAXED, __HIP_MEMORY_SCOPE_WORKGROUP);
        if (enc) {
            u32x4 o[4];
            const unsigned m = encode_unit(xd, te, smem + LDS_A + d * 4096, lane, o);
            if (m) {
                unsigned char* eb = myws + WS_ENC + (size_t)eu * 4096;
                store_sys<0>(o[0], eb, sv); store_sys<1024>(o[1], eb, sv);
                store_sys<2048>(o[2], eb, sv); store_sys<3072>(o[3], eb, sv);
            }
            STAMP(20);
            wait_vm<0>();
            put_word(myws + WS_META + (size_t)eo * MSTR, 0u, (call << 12) | m);
            STAMP(21);
        }
        while (__builtin_amdgcn_readfirstlane(lds_get(w_tab)) < 4u) __builtin_amdgcn_s_sleep(2);
        unsigned* tmo = (unsigned*)(myws + WS_TMO);
        const unsigned par = (old & 1) * NR;
        const unsigned lv = (lane & 15) * KD + (lane >> 4) * 16;
        const unsigned nitem = S * NCH;
        unsigned dead = 0;
        unsigned o_unit, o_flag, o_gflag, o_nib;
        {
            unsigned src, b;
            block_of(lane < S ? lane : 0, src, b);
            const unsigned s = par + (rank + src) % NR;
            o_unit = src ? (unsigned)WS_SLOT + s * (unsigned)SLOT_BYTES + b * 4096 : b * 16 * KD;
            o_flag = (unsigned)WS_FLAG + (s * NB16 + b) * 128;
            o_gflag = (unsigned)WS_GFLAG + (s * 4 + (b >> 3)) * 128;
            o_nib = (b & 7u) * 4;
        }
#pragma unroll 1
        for (unsigned i = d; i < nitem;) {
            const unsigned sc = S == SL ? i / (2 * SL) : i / (2 * SL - 2), r = i - sc * 2 * S;
            const unsigned k = r >> 1, c = 2 * sc + (r & 1), sl = i % ARING;
            const bool own = k < (unsigned)nloc;
            unsigned info = INFO_RAW;
            while (__builtin_amdgcn_ballot_w64(lds_get(w_used + (lane & 3) * 4) + ARING <= i) != 0) __builtin_amdgcn_s_sleep(2);
            if (!own && !dead) {
                const u64* fp = (const u64*)(myws + __builtin_amdgcn_readlane(c < GC ? o_gflag : o_flag, k)) + c;
                const u64 t0 = __builtin_amdgcn_s_memrealtime();
                unsigned it = 0;
                while (true) {
                    const u64 fv = __hip_atomic_load(fp, __ATOMIC_RELAXED, __HIP_MEMORY_SCOPE_SYSTEM);
                    info = (__builtin_amdgcn_readfirstlane((unsigned)fv) >> __builtin_amdgcn_readlane(o_nib, k)) & 15u;
                    if (__builtin_amdgcn_readfirstlane((unsigned)(fv >> 32)) == call) break;
                    if ((++it & 255) == 0) {
                        const unsigned tv = __builtin_amdgcn_readfirstlane(
                            __hip_atomic_load(tmo, __ATOMIC_RELAXED, __HIP_MEMORY_SCOPE_AGENT));
                        if (tv == call || __builtin_amdgcn_s_memrealtime() - t0 > DEADLINE_TICKS) {
                            __hip_atomic_store(tmo, call, __ATOMIC_RELAXED, __HIP_MEMORY_SCOPE_AGENT);
                            dead = 0x80000000u;
                            break;
                        }
                    }
                    __builtin_amdgcn_s_sleep(POLL_SLEEP);
                }
                asm volatile("" ::: "memory");
            }
            STAMP(32 + c);
            if (!dead) {
                const unsigned ou = __builtin_amdgcn_readlane(o_unit, k);
                const unsigned char* sb = own ? xp + ou + c * CHK : myws + ou + c * (NB16 * 4096);
                const unsigned vo = own ? lv : sv, st = own ? 64 : 1024;
#pragma unroll
                for (int q = 0; q < 4; ++q) glds16_sys(sb + q * st, vo, lds0 + LDS_A + sl * 4096 + q * 1024);
            }
            if (c >= NCH - 2 && __builtin_amdgcn_readfirstlane(
                    __hip_atomic_load(tmo, __ATOMIC_RELAXED, __HIP_MEMORY_SCOPE_AGENT)) == call) dead = 0x80000000u;
            wait_vm<0>();
            STAMP(48 + c);
            if (info <= 13u && !dead) {
                unsigned char* unit = smem + LDS_A + sl * 4096;
                u32x4 pk[3];
#pragma unroll
                for (int q = 0; q < 3; ++q) pk[q] = *(const u32x4*)(unit + q * 1024 + sv);
                unsigned v[16];
                decode_unit(pk, unit, info, v);
#pragma unroll
                for (int q = 0; q < 4; ++q)
                    *(u32x4*)(unit + q * 1024 + sv) = u32x4{v[4 * q], v[4 * q + 1], v[4 * q + 2], v[4 * q + 3]};
            }
            lds_set(w_item + sl * 4, (i + 1) | dead);
            STAMP(2 + c);
            unsigned nx = 0;
            if (lane == 0) nx = __hip_atomic_fetch_add((LAS unsigned*)w_next, 1u, __ATOMIC_RELAXED, __HIP_MEMORY_SCOPE_WORKGROUP);
            i = NLOAD + __builtin_amdgcn_readfirstlane(nx);
        }
        STAMP_OUT();
        return;
    }

    {
        const unsigned b0 = symbol_entry(threadIdx.x), b1 = symbol_entry(threadIdx.x + 256);
#pragma unroll
        for (unsigned x = 0; x < 16; ++x)
            *(LAS unsigned*)(smem + LDS_T + x * 1024 + threadIdx.x * 4) = ((x & 1) ? b1 : b0) | symbol_entry((x >> 1) << 9);
    }
    asm volatile("" ::: "memory");
    if (lane == 0) __hip_atomic_fetch_add((LAS unsigned*)w_tab, 1u, __ATOMIC_RELAXED, __HIP_MEMORY_SCOPE_WORKGROUP);
    __builtin_amdgcn_s_setprio(MPRIO);
    const float scale = p.sx[0] * p.sw[0];
    const __amdgpu_buffer_rsrc_t orsrc = __builtin_amdgcn_make_buffer_rsrc((void*)p.out, 0, NR * MS * NL * 4, 0x00020000);
    const int G = lane >> 4, i16 = lane & 15;
    const int ldsb = ((16 * G + (i16 >> 1)) * 128 + (((i16 >> 2) & 3) | ((G & 1) << 2)) * 16 + 8 * (i16 & 1)) ^ (wave << 5);
    const int ocol = ((i16 & 7) * NL + cb * 128 + wave * 32 + (i16 >> 3) * 16 + G * 4) * 4;
    unsigned ctl = 0;
    auto wait_ctl = [&](int l, unsigned v) {
        unsigned x = __builtin_amdgcn_readlane(ctl, l);
        while ((x & 0x7fffffffu) <= v) {
            ctl = lds_get(w_item + (lane & 31) * 4);
            x = __builtin_amdgcn_readlane(ctl, l);
            if ((x & 0x7fffffffu) <= v) __builtin_amdgcn_s_sleep(1);
        }
        return x;
    };
    i32x8 wf[2][2], wl[2][2];
    auto read_w = [&](i32x8 (&z)[2][2], int slot) {
#pragma unroll
        for (int ks = 0; ks < 2; ++ks)
#pragma unroll
            for (int n = 0; n < 2; ++n) {
                const unsigned char* lp = smem + LDS_S + slot * STAGE + ks * 16384 + (ldsb ^ (n << 4));
                i32x2 t0 = __builtin_amdgcn_ds_read_tr8_b64_v2i32((LAS i32x2*)(lp));
                i32x2 t1 = __builtin_amdgcn_ds_read_tr8_b64_v2i32((LAS i32x2*)(lp + 1024));
                i32x2 t2 = __builtin_amdgcn_ds_read_tr8_b64_v2i32((LAS i32x2*)(lp + 8192));
                i32x2 t3 = __builtin_amdgcn_ds_read_tr8_b64_v2i32((LAS i32x2*)(lp + 8192 + 1024));
                z[ks][n] = i32x8{t0[0], t0[1], t1[0], t1[1], t2[0], t2[1], t3[0], t3[1]};
            }
    };
    f32x4 acc[SL][2];
#pragma unroll
    for (int k = 0; k < SL; ++k) acc[k][0] = acc[k][1] = f32x4{0.f, 0.f, 0.f, 0.f};
    asm volatile("" :: "v"(scale) : "memory");
    STAMP(1);
    unsigned i = 0, as = 0, bad = 0;
    unsigned rowoff;
    {
        unsigned src, b;
        block_of(lane < S ? lane : 0, src, b);
        rowoff = (((rank + src) % NR) * MS + b * 16) * (NL * 4);
        asm volatile("" : "+v"(rowoff));
    }
    u32x4 a[2][4];
    auto read_a = [&](u32x4 (&z)[4], unsigned v) {
        bad |= v >> 31;
        const unsigned char* unit = smem + LDS_A + as * 4096 + sv;
#pragma unroll
        for (int q = 0; q < 4; ++q) z[q] = *(const u32x4*)(unit + q * 1024);
        ++i; as = as + 1 == ARING ? 0 : as + 1;
        lds_set(w_used + wave * 4, i);
    };
    auto mul = [&](f32x4 (&ac)[2], const i32x8 (&w)[2][2], const u32x4 (&z)[4]) {
        const i32x8 xa0 = {(int)z[0][0], (int)z[0][1], (int)z[0][2], (int)z[0][3], (int)z[1][0], (int)z[1][1], (int)z[1][2], (int)z[1][3]};
        const i32x8 xa1 = {(int)z[2][0], (int)z[2][1], (int)z[2][2], (int)z[2][3], (int)z[3][0], (int)z[3][1], (int)z[3][2], (int)z[3][3]};
        ac[0] = __builtin_amdgcn_mfma_scale_f32_16x16x128_f8f6f4(w[0][0], xa0, ac[0], 0, 0, 0, 0, 0, 0);
        ac[1] = __builtin_amdgcn_mfma_scale_f32_16x16x128_f8f6f4(w[0][1], xa0, ac[1], 0, 0, 0, 0, 0, 0);
        ac[0] = __builtin_amdgcn_mfma_scale_f32_16x16x128_f8f6f4(w[1][0], xa1, ac[0], 0, 0, 0, 0, 0, 0);
        ac[1] = __builtin_amdgcn_mfma_scale_f32_16x16x128_f8f6f4(w[1][1], xa1, ac[1], 0, 0, 0, 0, 0, 0);
    };
#pragma unroll 1
    for (int c = 0; c < NCH; c += 2) {
        wait_ctl(24, c + 1); if (NSTG > 1) wait_ctl(25, c + 1);
        read_w(wf, 0);
        read_w(wl, 1);
        lds_set(w_read + wave * 4, c + 2);
#pragma unroll
        for (int k = 0; k < SL; ++k) {
            if (k < SL - 1 || S == SL) {
                read_a(a[0], wait_ctl(as, i));
                read_a(a[1], wait_ctl(as, i));
                if (c == NCH - 2) STAMP(32 + k);
                __builtin_amdgcn_sched_barrier(0);
                mul(acc[k], wf, a[0]);
                mul(acc[k], wl, a[1]);
                __builtin_amdgcn_sched_barrier(0);
                if (c == NCH - 2) {
                    int o1 = ocol + __builtin_amdgcn_readlane(rowoff, k), o2 = o1 + 8 * NL * 4;
                    asm volatile("" : "+v"(o1), "+v"(o2));
                    const float sc = bad ? __builtin_nanf("") : scale;
                    const u32x4 lo = __builtin_bit_cast(u32x4, acc[k][0] * sc), hi = __builtin_bit_cast(u32x4, acc[k][1] * sc);
                    u32x4 x, y;
#pragma unroll
                    for (int e = 0; e < 4; ++e) {
                        x[e] = __builtin_amdgcn_update_dpp(lo[e], hi[e], 0x128, 0xf, 0xc, false);
                        y[e] = __builtin_amdgcn_update_dpp(hi[e], lo[e], 0x128, 0xf, 0x3, false);
                    }
                    __builtin_amdgcn_raw_buffer_store_b128(x, orsrc, o1, 0, OUT_AUX);
                    __builtin_amdgcn_raw_buffer_store_b128(y, orsrc, o2, 0, OUT_AUX);
                }
            }
        }
        STAMP(2 + c); STAMP(3 + c);
    }
}

extern "C" void kernel_launch(int rank, int nranks, ncclComm_t comm,
                              hipStream_t stream,
                              void* const* d_in, const int* in_sizes, int n_in,
                              void* d_out, int out_size,
                              const int* in_dtypes, int out_dtype,
                              void* const* d_out_all,
                              void* const* d_ws_all, size_t ws_bytes) {
    (void)comm; (void)in_dtypes; (void)out_dtype; (void)d_out_all;
    if (nranks != NR || (unsigned)rank >= (unsigned)NR || n_in != 4 || in_sizes[0] != MS * KD ||
        in_sizes[1] != KD * NL || out_size != NR * MS * NL || ws_bytes < WS_NEED) {
        fprintf(stderr, "unexpected problem shape\n");
        return;
    }
    Params p;
    for (int j = 0; j < NR; ++j) p.ws[j] = (unsigned char*)d_ws_all[(rank + j) % NR];
    p.x = (const unsigned char*)d_in[0];
    p.w = (const unsigned char*)d_in[1];
    p.sx = (const float*)d_in[2];
    p.sw = (const float*)d_in[3];
    p.out = (float*)d_out;
    p.rank = rank;
    p.pad = 0;
    HIP_CHECK(hipFuncSetAttribute((const void*)ag_gemm, hipFuncAttributeMaxDynamicSharedMemorySize, LDS_BYTES));
    hipLaunchKernelGGL(ag_gemm, dim3(GRIDX), dim3(THREADS), LDS_BYTES, stream, p);
    HIP_CHECK(hipGetLastError());
}
```

```cpp
#include <hip/hip_runtime.h>
#include <rccl/rccl.h>
#include <cstdio>

#define HIP_CHECK(expr) do { hipError_t _e = (expr); if (_e != hipSuccess) { \
    fprintf(stderr, "HIP error %s at %s:%d\n", hipGetErrorString(_e), __FILE__, __LINE__); \
    return; } } while (0)

typedef unsigned u32x4 __attribute__((ext_vector_type(4)));
typedef unsigned short u16x2 __attribute__((ext_vector_type(2)));
typedef int i32x2 __attribute__((ext_vector_type(2)));
typedef int i32x8 __attribute__((ext_vector_type(8)));
typedef float f32x4 __attribute__((ext_vector_type(4)));
typedef unsigned long long u64;
#define LAS __attribute__((address_space(3)))

#define WIN 6
#define DEPTH 4
#define SPIN0 6
#define NFEED 3
#define CSLEEP 16
#define MPRIO 2
#define NLOAD 7
#define CDELAY 450
#define NWS 2
#define NSTG 1
#define LATE 512
#define DEPTH2 4
#define DIRECT 1
#define WARM 12
#define WARMO 1
#define OUT_AUX 17
#define NRAW 3
#define ENC_HI 480
#define GC 10
#define MSTR 32
#define POLL_SLEEP 2

constexpr int NR = 8;
constexpr int KD = 4096;
constexpr int MS = 512;
constexpr int NL = 512;
constexpr int NB16 = MS / 16;
constexpr int CHK = 256;
constexpr int NCH = KD / CHK;
constexpr int NU = NB16 * NCH;
constexpr int PPP = 8;
constexpr int RING = 16;
constexpr int GEMM0 = 7 * PPP;
constexpr unsigned NPOS0 = NRAW + (4 + NLOAD + NSTG) - (NFEED + 3);
constexpr unsigned NFIX = PPP * NPOS0;
constexpr unsigned ENC_LO = NFIX;
constexpr int NRW = 50;
constexpr int SL = 6;
constexpr int NGEMM = 4 * NRW;
constexpr int GRIDX = GEMM0 + NGEMM;
constexpr int THREADS = 64 * (4 + NLOAD + NSTG);
constexpr int STAGE = CHK * 128;
constexpr int ARING = NWS == 2 ? 19 : 11;
constexpr int LDS_T = 0;
constexpr int LDS_S = 16384;
constexpr int LDS_A = LDS_S + NWS * STAGE;
constexpr int LDS_W = LDS_A + ARING * 4096 + 64;
constexpr int LDS_P = LDS_W + 256;
constexpr int LDS_BYTES = LDS_P + 8 * 256;
static_assert(GRIDX <= 256 && LDS_BYTES <= 163840 && ENC_HI - ENC_LO <= 4 * NGEMM, "resident grid");
static_assert(WIN >= 1 && WIN <= 32 && DEPTH >= 1 && DEPTH <= RING && NFEED <= RING && NRAW <= NFEED + 1 && NPOS0 > NFEED && NPOS0 <= RING && THREADS <= 768 && NSTG <= 2 && NWS == 2, "push geometry");

__device__ __forceinline__ unsigned unit_at(unsigned q) {
    return (q & ~63u) + (q & 1u) * NB16 + ((q & 63u) >> 1);
}

constexpr size_t WS_FLAG = 0;
constexpr size_t WS_TMO = 64 << 10;
constexpr size_t WS_GFLAG = 72 << 10;
constexpr size_t WS_TICKET = 96 << 10;
constexpr size_t WS_CNT = 128 << 10;
constexpr size_t WS_META = 256 << 10;
constexpr size_t WS_GCNT = 384 << 10;
constexpr size_t WS_TRACE = 512 << 10;
constexpr size_t SLOT_BYTES = (size_t)MS * KD;
constexpr size_t WS_ENC = 2 << 20;
constexpr size_t WS_SLOT = 4 << 20;
constexpr size_t WS_NEED = WS_SLOT + 2 * NR * SLOT_BYTES;
constexpr unsigned INFO_RAW = 15u;
constexpr u64 DEADLINE_TICKS = 100000000ull;

struct Params {
    unsigned char* ws[NR];
    const unsigned char* x;
    const unsigned char* w;
    const float* sx;
    const float* sw;
    float* out;
    int rank;
    int pad;
    u32x4 tab[16];
};

#define STAMP(i) do {} while (0)
#define STAMP_OUT() do {} while (0)

template <int N> __device__ __forceinline__ void wait_vm() {
    asm volatile("s_waitcnt vmcnt(%0)" :: "i"(N) : "memory");
}
__device__ __forceinline__ void wait_lds() { asm volatile("s_waitcnt lgkmcnt(0)" ::: "memory"); }
__device__ __forceinline__ void glds16(const unsigned char* sbase, unsigned voff, unsigned lds) {
    unsigned keep;
    asm volatile("s_mov_b32 %0, m0\n\ts_mov_b32 m0, %3\n\ts_nop 0\n\tglobal_load_lds_dwordx4 %1, %2\n\ts_mov_b32 m0, %0"
                 : "=&s"(keep) : "v"(voff), "s"(sbase), "s"(lds) : "memory");
}
__device__ __forceinline__ void glds16_sys(const unsigned char* sbase, unsigned voff, unsigned lds) {
    unsigned keep;
    asm volatile("s_mov_b32 %0, m0\n\ts_mov_b32 m0, %3\n\ts_nop 0\n\tglobal_load_lds_dwordx4 %1, %2 sc0 sc1\n\ts_mov_b32 m0, %0"
                 : "=&s"(keep) : "v"(voff), "s"(sbase), "s"(lds) : "memory");
}
__device__ __forceinline__ void glds4_sys(const unsigned char* sbase, unsigned voff, unsigned lds) {
    unsigned keep;
    asm volatile("s_mov_b32 %0, m0\n\ts_mov_b32 m0, %3\n\ts_nop 0\n\tglobal_load_lds_dword %1, %2 sc0 sc1\n\ts_mov_b32 m0, %0"
                 : "=&s"(keep) : "v"(voff), "s"(sbase), "s"(lds) : "memory");
}
template <int OFF> __device__ __forceinline__ void store_sys(const u32x4& d, unsigned char* sbase, unsigned voff) {
    asm volatile("global_store_dwordx4 %1, %0, %2 offset:%3 sc0 sc1\n\ts_nop 1" :: "v"(d), "v"(voff), "s"(sbase), "i"(OFF) : "memory");
}
template <int OFF> __device__ __forceinline__ void store_sys_short(const u32x4& d, unsigned char* sbase, unsigned voff, unsigned drop) {
    unsigned t;
    asm volatile("s_nop 0\n\tv_readfirstlane_b32 %0, %5\n\ts_lshr_b64 exec, -1, %0\n\t"
                 "global_store_dwordx4 %2, %1, %3 offset:%4 sc0 sc1\n\ts_mov_b64 exec, -1\n\ts_nop 0"
                 : "=&s"(t) : "v"(d), "v"(voff), "s"(sbase), "i"(OFF), "v"(drop) : "memory", "scc");
}
__device__ __forceinline__ void put_word(unsigned char* sbase, unsigned zero, unsigned val) {
    asm volatile("s_mov_b64 exec, 1\n\tglobal_store_dword %0, %1, %2 sc0 sc1\n\ts_mov_b64 exec, -1\n\ts_nop 0"
                 :: "v"(zero), "v"(val), "s"(sbase) : "memory");
}
__device__ __forceinline__ void put_pair(unsigned char* sbase, unsigned lo, unsigned hi) {
    const u64 v = ((u64)hi << 32) | lo;
    asm volatile("s_mov_b64 exec, 1\n\tglobal_store_dwordx2 %0, %1, %2 sc0 sc1\n\ts_mov_b64 exec, -1\n\ts_nop 0"
                 :: "v"(0u), "v"(v), "s"(sbase) : "memory");
}
__device__ __forceinline__ void draw(unsigned& t, unsigned* ctr, unsigned zero, unsigned one) {
    asm volatile("s_mov_b64 exec, 1\n\tglobal_atomic_add %0, %1, %2, %3 sc0\n\ts_mov_b64 exec, -1"
                 : "+v"(t) : "v"(zero), "v"(one), "s"(ctr) : "memory");
}
template <int N> __device__ __forceinline__ unsigned take(unsigned& t) {
    asm volatile("s_waitcnt vmcnt(%1) ; take" : "+v"(t) : "i"(N) : "memory");
    return __builtin_amdgcn_readfirstlane(t);
}
__device__ __forceinline__ unsigned lds_get(unsigned char* a) {
    const unsigned v = __hip_atomic_load((LAS unsigned*)a, __ATOMIC_RELAXED, __HIP_MEMORY_SCOPE_WORKGROUP);
    asm volatile("" ::: "memory");
    return v;
}
__device__ __forceinline__ void lds_set(unsigned char* a, unsigned v) {
    asm volatile("" ::: "memory");
    __hip_atomic_store((LAS unsigned*)a, v, __ATOMIC_RELAXED, __HIP_MEMORY_SCOPE_WORKGROUP);
}

__device__ __forceinline__ unsigned scan_incl(unsigned v) {
    v += __builtin_amdgcn_update_dpp(0u, v, 0x111, 0xf, 0xf, false);
    v += __builtin_amdgcn_update_dpp(0u, v, 0x112, 0xf, 0xf, false);
    v += __builtin_amdgcn_update_dpp(0u, v, 0x114, 0xf, 0xf, false);
    v += __builtin_amdgcn_update_dpp(0u, v, 0x118, 0xf, 0xf, false);
    v += __builtin_amdgcn_update_dpp(0u, v, 0x142, 0xa, 0xf, false);
    v += __builtin_amdgcn_update_dpp(0u, v, 0x143, 0xc, 0xf, false);
    return v;
}
__device__ __forceinline__ unsigned symbol_entry(unsigned x) {
    unsigned v = 0;
#pragma unroll
    for (int i = 0; i < 4; ++i) { const unsigned y = (x >> (3 * i)) & 7u; v |= ((y < 6u ? y : y + 3u) << 3) << (8 * i); }
    return v;
}
__host__ __device__ inline void encode_table(int e0, unsigned* t) {
    unsigned a[16];
#pragma unroll
    for (int e = 0; e < 16; ++e) {
        const int q = e - e0, y = q < 0 ? q + 6 : q + 3;
        a[e] = (q >= 0 && q <= 2) ? (unsigned)((q + 1) << 3) : ((q >= -6 && q <= 4) ? (unsigned)(0x60 | y) : 0x80u);
    }
    auto sg = [](unsigned v) { return (v & 0x80u) ? 0xffu : 0u; };
#pragma unroll
    for (int i = 5; i < 8; ++i) { a[i] ^= 0xffu; a[8 + i] ^= 0xffu; }
    a[3] ^= sg(a[15]); a[2] ^= sg(a[13]); a[11] ^= sg(a[7]); a[10] ^= sg(a[5]);
    a[1] ^= sg(a[11]); a[9] ^= sg(a[3]);
    a[0] ^= sg(a[9]); a[8] ^= sg(a[1]);
#pragma unroll
    for (int i = 0; i < 4; ++i) t[i] = a[4 * i] | (a[4 * i + 1] << 8) | (a[4 * i + 2] << 16) | (a[4 * i + 3] << 24);
}
__device__ __forceinline__ unsigned encode_unit(const unsigned (&d)[16], const u32x4& te, unsigned char* scr,
                                                int lane, u32x4 (&o)[4]) {
    *(u32x4*)(scr + lane * 16) = u32x4{0u, 0u, 0u, 0u};
    unsigned es = 0;
#pragma unroll
    for (int i = 0; i < 16; i += 5) es = __builtin_amdgcn_sad_u8(d[i] & 0x78787878u, 0u, es);
    int e0 = ((int)__builtin_amdgcn_readlane(scan_incl(es), 63) - 1638) >> 13;
    e0 = e0 < 0 ? 0 : (e0 > 13 ? 13 : e0);
    const unsigned t0 = __builtin_amdgcn_readlane(te[0], e0), t1 = __builtin_amdgcn_readlane(te[1], e0);
    const unsigned t2 = __builtin_amdgcn_readlane(te[2], e0), t3 = __builtin_amdgcn_readlane(te[3], e0);
    unsigned c[16], bad = 0, nb[2];
    unsigned long long str[2];
#pragma unroll
    for (int h = 0; h < 2; ++h) {
        unsigned v[8], n[8];
#pragma unroll
        for (int k = 0; k < 8; ++k) {
            const unsigned x = d[8 * h + k];
            const unsigned e = (x >> 3) & 0x0f0f0f0fu;
            const unsigned inf = __builtin_amdgcn_perm(t1, t0, e) ^ __builtin_amdgcn_perm(t3, t2, e ^ 0x08080808u);
            c[8 * h + k] = (x & 0x87878787u) | (inf & 0x18181818u);
            bad |= inf;
            const unsigned sh = (inf >> 5) & 0x03030303u;
            const unsigned sy = inf & 0x07070707u;
            const u16x2 up = (__builtin_bit_cast(u16x2, sy) >> (unsigned short)8) << (__builtin_bit_cast(u16x2, sh) & (unsigned short)15);
            const unsigned m = (sy & 0x00ff00ffu) | __builtin_bit_cast(unsigned, up);
            v[k] = (m & 0xffffu) | ((m >> 16) << ((sh + (sh >> 8)) & 31u));
            n[k] = __builtin_amdgcn_sad_u8(sh, 0u, 0u);
        }
        unsigned long long acc = 0;
        unsigned off = 0;
#pragma unroll
        for (int k = 0; k < 8; k += 2) {
            acc |= (unsigned long long)(v[k] | (v[k + 1] << n[k])) << (off & 63u);
            off += n[k] + n[k + 1];
        }
        str[h] = acc; nb[h] = off;
    }
#pragma unroll
    for (int q = 0; q < 4; ++q) {
        const unsigned z = c[4 * q + 3];
        const unsigned p0 = c[4 * q] | ((z << 5) & 0x60606060u);
        const unsigned p1 = c[4 * q + 1] | ((z << 3) & 0x60606060u);
        const unsigned p2 = c[4 * q + 2] | ((z << 1) & 0x20202020u) | ((z >> 1) & 0x40404040u);
        o[(3 * q) >> 2][(3 * q) & 3] = p0; o[(3 * q + 1) >> 2][(3 * q + 1) & 3] = p1; o[(3 * q + 2) >> 2][(3 * q + 2) & 3] = p2;
    }
    const unsigned tot = nb[0] + nb[1];
    const unsigned inc = scan_incl(tot);
    const unsigned bits = __builtin_amdgcn_readlane(inc, 63);
    const bool fail = (bad & 0x80808080u) || nb[0] > 60u || nb[1] > 60u;
    if (__builtin_amdgcn_ballot_w64(fail) != 0 || bits > 8192u - 128u) return 0u;
#pragma unroll
    for (int h = 0; h < 2; ++h) {
        const unsigned ob = inc - tot + (h ? nb[0] : 0u);
        unsigned* wp = (unsigned*)(scr + (ob >> 5) * 4);
        const unsigned long long a = (unsigned long long)(unsigned)str[h] << (ob & 31u);
        const unsigned long long b = (unsigned long long)(unsigned)(str[h] >> 32) << (ob & 31u);
        __hip_atomic_fetch_or(wp, (unsigned)a, __ATOMIC_RELAXED, __HIP_MEMORY_SCOPE_WORKGROUP);
        __hip_atomic_fetch_or(wp + 1, (unsigned)(a >> 32) | (unsigned)b, __ATOMIC_RELAXED, __HIP_MEMORY_SCOPE_WORKGROUP);
        __hip_atomic_fetch_or(wp + 2, (unsigned)(b >> 32), __ATOMIC_RELAXED, __HIP_MEMORY_SCOPE_WORKGROUP);
    }
    asm volatile("s_waitcnt lgkmcnt(0)" ::: "memory");
    o[3] = *(const u32x4*)(scr + lane * 16);
    const unsigned nl = (bits + 127u) >> 7;
    return ((nl ? nl : 1u) << 4) | (unsigned)e0;
}
__device__ __forceinline__ void decode_unit(const u32x4 (&pk)[3], const unsigned char* slot, unsigned e0, unsigned (&d)[16]) {
    unsigned p[12];
#pragma unroll
    for (int q = 0; q < 12; ++q) p[q] = pk[q >> 2][q & 3];
#pragma unroll
    for (int q = 0; q < 4; ++q) {
        d[4 * q] = p[3 * q] & 0x9f9f9f9fu; d[4 * q + 1] = p[3 * q + 1] & 0x9f9f9f9fu; d[4 * q + 2] = p[3 * q + 2] & 0x9f9f9f9fu;
        d[4 * q + 3] = ((p[3 * q] >> 5) & 0x03030303u) | ((p[3 * q + 1] >> 3) & 0x0c0c0c0cu) |
                       ((p[3 * q + 2] >> 1) & 0x10101010u) | ((p[3 * q + 2] << 1) & 0x80808080u);
    }
    unsigned f[16], c[2][9];
    c[0][0] = c[1][0] = 0;
#pragma unroll
    for (int i = 0; i < 16; ++i) {
        f[i] = ~(d[i] | (d[i] >> 1)) & 0x08080808u;
        c[i >> 3][(i & 7) + 1] = __builtin_popcount(f[i]) + c[i >> 3][i & 7];
    }
    const unsigned n = c[0][8] + c[1][8];
    const unsigned o0 = (scan_incl(n) - n) * 3 + 30, o1 = o0 + c[0][8] * 3;
    const unsigned bias = (e0 - 6u) * 0x08080808u;
    unsigned y[16];
#pragma unroll
    for (int h = 0; h < 2; ++h) {
        const unsigned o = h ? o1 : o0;
        const unsigned* wp = (const unsigned*)(slot + 3068 + ((o >> 5) & 255u) * 4);
        const unsigned w0 = wp[0], w1 = wp[1], w2 = wp[2];
        const u64 win = __builtin_amdgcn_alignbit(w1, w0, o) | ((u64)__builtin_amdgcn_alignbit(w2, w1, o) << 32);
#pragma unroll
        for (int i = 0; i < 8; ++i)
            y[8 * h + i] = *(const LAS unsigned*)(size_t)((unsigned)(win >> ((c[h][i] * 3) & 63u)) & 0x3ffcu);
    }
#pragma unroll
    for (int i = 0; i < 16; ++i) {
        const unsigned sel = __umul24(f[i], 0x202020u) + (f[i] >> 1);
        d[i] = d[i] + __builtin_amdgcn_perm(y[i], 0x28282828u, sel) + bias;
    }
}

__global__ void __launch_bounds__(THREADS, 3) ag_gemm(Params p) {
    extern __shared__ __attribute__((aligned(16))) unsigned char smem[];
    const int lane = threadIdx.x & 63;
    const int wave = __builtin_amdgcn_readfirstlane(threadIdx.x >> 6);
    const unsigned j = 1 + blockIdx.x % (NR - 1);
    unsigned char* const myws = p.ws[0];
    unsigned char* const pws = p.ws[j];
    const unsigned char* const xp = p.x;
    const unsigned rank = p.rank;
    asm volatile("" :: "s"(myws), "s"(pws), "s"(xp), "s"(rank));
    const unsigned lds0 = (unsigned)(size_t)(LAS unsigned char*)smem;
    unsigned* const cnt = (unsigned*)(myws + WS_CNT + (size_t)(blockIdx.x * 16 + wave) * 32);
    const unsigned sv = lane * 16;
    STAMP(0);

    if (__builtin_expect(blockIdx.x < GEMM0, 1)) {
        constexpr unsigned POS_LAST = 1u << 29, POS_UNIT = 1u << 30, POS_END = 1u << 31;
        const unsigned pk = blockIdx.x / (NR - 1);
        const unsigned lv = (lane & 15) * KD + (lane >> 4) * 16;
        unsigned char* const w_pos = smem + RING * 4096;
        unsigned char* const w_meta = w_pos + 4096;
        unsigned char* const w_cons = w_meta + NU * 4;
        unsigned char* const w_told = w_cons + 4;
        constexpr unsigned NDRAW = NU - NFIX + PPP * NFEED;
        const bool feeder = wave >= 1 && wave <= NFEED, packer = wave >= NFEED + 3;
        unsigned* const ctr = feeder ? (unsigned*)(myws + WS_TICKET + j * 128) : cnt;
        const unsigned n0 = packer ? NRAW + wave - (NFEED + 3) : wave;
        unsigned u = unit_at(pk + PPP * n0);
        if (wave < NRAW || packer) {
            const unsigned char* sb = xp + (size_t)(u % NB16) * 16 * KD + (u / NB16) * CHK;
#pragma unroll
            for (int q = 0; q < 4; ++q) glds16(sb + q * 64, lv, lds0 + n0 * 4096 + q * 1024);
        } else if (wave > NFEED) {
#pragma unroll
            for (int q = 0; q < 4; ++q) *(LAS u32x4*)(w_pos + ((wave - NFEED - 1) * 4 + q) * 1024 + sv) = u32x4{0u, 0u, 0u, 0u};
            wait_lds();
        }
        if (j >= (unsigned)NR) return;
        if (packer) {
            unsigned xd[16];
            const u32x4 te = p.tab[lane & 15];
            __builtin_amdgcn_s_barrier();
            wait_vm<0>();
            unsigned char* slot = smem + n0 * 4096;
#pragma unroll
            for (int q = 0; q < 4; ++q) {
                const u32x4 v = *(const u32x4*)(slot + q * 1024 + sv);
                xd[4 * q] = v[0]; xd[4 * q + 1] = v[1]; xd[4 * q + 2] = v[2]; xd[4 * q + 3] = v[3];
            }
            u32x4 o[4];
            const unsigned m = encode_unit(xd, te, slot + 3072, lane, o);
            if (m) { *(u32x4*)(slot + sv) = o[0]; *(u32x4*)(slot + 1024 + sv) = o[1]; *(u32x4*)(slot + 2048 + sv) = o[2]; }
            else *(u32x4*)(slot + 3072 + sv) = u32x4{xd[12], xd[13], xd[14], xd[15]};
            lds_set(w_pos + n0 * 4, POS_UNIT | (u << 12) | (m ? (64u - (m >> 4)) | ((m & 15u) << 6) : INFO_RAW << 6));
            STAMP(2);
            STAMP_OUT();
            return;
        }
        unsigned cv = 0;
        draw(cv, ctr, 0u, 1u);
        const u64 tstart = __builtin_amdgcn_s_memrealtime();
        __builtin_amdgcn_s_barrier();
        unsigned t = take<0>(cv);
        if (feeder) {
            unsigned n = wave, m = 0;
            bool have = true;
            while (true) {
                if (n < NRAW || n >= NPOS0)
                    lds_set(w_pos + n * 4, POS_UNIT | (have && NFIX + t % NDRAW >= NU ? POS_LAST : 0u) | (u << 12) |
                                           (m ? (64u - (m >> 4)) | ((m & 15u) << 6) : INFO_RAW << 6));
                n = n < NPOS0 ? NPOS0 + n - 1 : n + NFEED;
                if (!have || NFIX + t % NDRAW < NU)
                    while (n >= __builtin_amdgcn_readfirstlane(lds_get(w_cons)) + (have ? DEPTH : DEPTH2)) __builtin_amdgcn_s_sleep(1);
                unsigned tv = 0;
                if (!have) { draw(tv, ctr, 0u, 1u); t = take<0>(tv); }
                const unsigned o = NFIX + t % NDRAW;
                if (o >= (unsigned)NU) { lds_set(w_pos + n * 4, POS_END); return; }
                u = unit_at(o);
                m = __builtin_amdgcn_readfirstlane(lds_get(w_meta + o * 4));
                if (m == 0 && o >= ENC_LO && o < ENC_HI) {
                    const unsigned e = __builtin_amdgcn_readfirstlane(__hip_atomic_load(
                        (unsigned*)(myws + WS_META + (size_t)o * MSTR), __ATOMIC_RELAXED, __HIP_MEMORY_SCOPE_SYSTEM));
                    if ((e >> 12) == ((t / NDRAW + 1) & 0xfffffu)) m = e & 0xfffu;
                    asm volatile("" ::: "memory");
                }
                if ((m >> 4) < 1u || (m >> 4) > 64u) m = 0;
                have = o < LATE;
                draw(tv, ctr, 0u, have ? 1u : 0u);
                const unsigned char* sb = m ? myws + WS_ENC + (size_t)u * 4096
                                            : xp + (size_t)(u % NB16) * 16 * KD + (u / NB16) * CHK;
                const unsigned vo = m ? sv : lv, st = m ? 1024 : 64;
#pragma unroll
                for (int q = 0; q < 4; ++q) glds16_sys(sb + q * st, vo, lds0 + (n % RING) * 4096 + q * 1024);
                t = take<0>(tv);
            }
        }
        const unsigned old = t, call = old + 1;
        if (__builtin_expect(wave == 0, 1)) {
            unsigned char* dst = pws + WS_SLOT + (size_t)((old & 1) * NR + rank) * SLOT_BYTES;
            unsigned char* const cons = w_cons, * const told = w_told;
            __builtin_amdgcn_s_setprio(3);
            STAMP(1);
            constexpr int FJ = (3 + WIN) % 4, FB = (3 + WIN) / 4;
            static_assert(FB <= 3, "window");
            unsigned n = 0, live = (1u << NFEED) - 1u, dsc = POS_UNIT | (u << 12) | (INFO_RAW << 6);
            unsigned e1 = 0, e2 = 0, e3 = 0;
            unsigned d1 = 0, d2 = 0;
            auto look = [&]() { return __builtin_amdgcn_readfirstlane(lds_get(w_pos + n * 4)); };
            auto acquire = [&]() {
                while (live) {
                    if ((live >> (n % NFEED)) & 1u) {
                        dsc = look();
                        for (int k = 0; k < SPIN0 && !dsc; ++k) dsc = look();
                        if (!dsc) {
                            wait_vm<4>(); lds_set(told, e2); e3 = e2;
                            dsc = look();
                            if (!dsc) {
                                wait_vm<0>(); lds_set(told, e1); e3 = e2 = e1;
                                do dsc = look(); while (!dsc);
                            }
                        }
                        if (dsc & POS_UNIT) return true;
                        live &= ~(1u << (n % NFEED));
                    }
                    n += 1;
                    lds_set(cons, n);
                }
                return false;
            };
            u32x4 r[4], q[4];
            auto grab = [&](u32x4 (&z)[4]) {
#pragma unroll
                for (int k = 0; k < 4; ++k) z[k] = *(const u32x4*)(smem + (n % RING) * 4096 + k * 1024 + sv);
            };
            auto send = [&](u32x4 (&a)[4], u32x4 (&z)[4]) {
                const unsigned vo = (dsc & 0x1ff000u) | sv;
                const unsigned drop = dsc;
                const unsigned e0 = n + 1, rep = FB == 1 ? e1 : (FB == 2 ? e2 : e3);
                n += 1;
                unsigned w = lds_get(w_pos + n * 4);
                const bool pace = !(dsc & POS_LAST);
                auto point = [&](int jj) { if (jj == FJ) { wait_vm<WIN - 1>(); lds_set(told, rep); } else if (pace) wait_vm<WIN - 1>(); };
                d2 = d1; d1 = dsc;
                point(0); store_sys<0>(a[0], dst, vo);
                point(1); store_sys<1024>(a[1], dst, vo);
                asm volatile("" : "+v"(w));
                dsc = __builtin_amdgcn_readfirstlane(w);
                bool ok = (dsc & POS_UNIT) != 0;
                if (ok) grab(z);
                point(2); store_sys<2048>(a[2], dst, vo);
                point(3); store_sys_short<3072>(a[3], dst, vo, drop);
                e3 = e2; e2 = e1; e1 = e0;
                if (!ok) {
                    STAMP(3);
                    ok = acquire();
                    if (ok) grab(z);
                }
                lds_set(cons, n + 1);
                return ok;
            };
            STAMP(5);
            grab(r); lds_set(w_pos, dsc); lds_set(cons, 1u);
            while (send(r, q) && send(q, r)) {}
            const unsigned u1 = (d1 >> 12) & 0x1ffu, u2 = (d2 >> 12) & 0x1ffu;
            if (DIRECT && FB == 2 && e3 < e2 && e2 < e1 && u1 >= GC * NB16 && u2 >= GC * NB16) {
                unsigned char* fl = pws + WS_FLAG + (size_t)((old & 1) * NR + rank) * (NB16 * 128);
                lds_set(told, e3 | 0x80000000u);
                wait_vm<4>();
                put_pair(fl + (u2 % NB16) * 128 + (u2 / NB16) * 8, ((d2 >> 6) & 15u) << ((u2 & 7u) * 4), call);
                wait_vm<1>();
                put_pair(fl + (u1 % NB16) * 128 + (u1 / NB16) * 8, ((d1 >> 6) & 15u) << ((u1 & 7u) * 4), call);
            } else {
                wait_vm<4>(); lds_set(told, e2);
                wait_vm<0>(); lds_set(told, e1 | 0x80000000u);
            }
            STAMP(2);
            STAMP_OUT();
            return;
        }
        if (wave == NFEED + 2) {
            constexpr int NQ = (ENC_HI - ENC_LO + 63) / 64;
            const __amdgpu_buffer_rsrc_t mr = __builtin_amdgcn_make_buffer_rsrc(myws + WS_META, 0, ENC_HI * MSTR, 0x00020000);
            while (__builtin_amdgcn_s_memrealtime() - tstart < CDELAY) __builtin_amdgcn_s_sleep(4);
            while (true) {
                unsigned e[NQ];
#pragma unroll
                for (int q = 0; q < NQ; ++q) e[q] = __builtin_amdgcn_raw_buffer_load_b32(mr, (ENC_LO + q * 64 + lane) * MSTR, 0, 17);
                bool ok = true;
#pragma unroll
                for (int q = 0; q < NQ; ++q) {
                    const unsigned u = ENC_LO + q * 64 + lane;
                    if ((e[q] >> 12) == (call & 0xfffffu) && u < (unsigned)ENC_HI) *(LAS unsigned*)(w_meta + u * 4) = (e[q] & 0xfffu) ? (e[q] & 0xfffu) : 1u;
                    else if (u < (unsigned)ENC_HI) ok = false;
                }
                if (__builtin_amdgcn_ballot_w64(!ok) == 0 || (__builtin_amdgcn_readfirstlane(lds_get(w_told)) >> 31)) break;
                __builtin_amdgcn_s_sleep(CSLEEP);
            }
            return;
        }
        unsigned char* fl = pws + WS_FLAG + (size_t)((old & 1) * NR + rank) * (NB16 * 128);
        unsigned char* gfl = pws + WS_GFLAG + (size_t)((old & 1) * NR + rank) * (4 * 128);
        unsigned char* gct = myws + WS_GCNT + (size_t)j * (NCH * 4 * 64);
        unsigned seen = 0;
        while (true) {
            const unsigned td = __builtin_amdgcn_readfirstlane(lds_get(w_told));
            const unsigned lim = td & 0x7fffffffu, n = seen + lane;
            if (seen >= lim) {
                if (td >> 31) break;
                __builtin_amdgcn_s_sleep(1);
                continue;
            }
            const unsigned dsc = n < lim ? lds_get(w_pos + n * 4) : 0u;
            if (dsc & POS_UNIT) {
                const unsigned u = (dsc >> 12) & 0x1ffu, nib = ((dsc >> 6) & 15u) << ((u & 7u) * 4);
                if (u >= GC * NB16) {
                    __hip_atomic_store((u64*)(fl + (u % NB16) * 128 + (u / NB16) * 8), ((u64)call << 32) | nib,
                                       __ATOMIC_RELAXED, __HIP_MEMORY_SCOPE_SYSTEM);
                } else {
                    u64* gc = (u64*)(gct + (u >> 3) * 64);
                    const u64 mine = (1ull << 32) | nib;
                    const u64 tot = __hip_atomic_fetch_add(gc, mine, __ATOMIC_RELAXED, __HIP_MEMORY_SCOPE_AGENT) + mine;
                    if ((tot >> 32) == 8u) {
                        __hip_atomic_store((u64*)(gfl + ((u >> 3) & 3u) * 128 + (u / NB16) * 8),
                                           ((u64)call << 32) | (unsigned)tot, __ATOMIC_RELAXED, __HIP_MEMORY_SCOPE_SYSTEM);
                        __hip_atomic_store(gc, 0ull, __ATOMIC_RELAXED, __HIP_MEMORY_SCOPE_AGENT);
                    }
                }
            }
            seen = seen + 64 < lim ? seen + 64 : lim;
        }
        STAMP(2);
        STAMP_OUT();
        return;
    }

    if (lds0 != 0) return;
    const int g = blockIdx.x - GEMM0;
    const int cb = g & 3, rg = g >> 2;
    static_assert(7 * NB16 == 4 * NRW + 24 && NB16 == 26 + 6, "block deal");
    const int nloc = rg < 24 ? 0 : (rg < 44 ? 1 : 2), S = rg < 44 ? 5 : 6;
    auto block_of = [&](unsigned k, unsigned& src, unsigned& b) {
        const unsigned t = rg + (k - nloc) * NRW;
        const bool own = k < (unsigned)nloc;
        src = own ? 0u : 1u + t % 7u;
        b = own ? (k ? rg - 18 : rg - 24) : t / 7u;
    };
    unsigned char* const w_item = smem + LDS_W;
    unsigned char* const w_land = w_item + 96;
    unsigned char* const w_got = w_item + 104;
    unsigned char* const w_tab = w_item + 108;
    unsigned char* const w_next = w_item + 112;
    unsigned char* const w_used = w_item + 128;
    unsigned char* const w_read = w_item + 144;
    if (threadIdx.x < 64) *(LAS unsigned*)(w_item + threadIdx.x * 4) = 0u;
    wait_lds();
    __builtin_amdgcn_s_barrier();

    if (wave >= 4 + NLOAD) {
        const int h = wave - 4 - NLOAD;
        const unsigned wv0 = (h * (256 / NSTG) + (lane >> 3)) * NL + cb * 128 + (((lane & 7) ^ ((lane >> 4) & 3)) << 4);
        const unsigned wv1 = wv0 ^ 64;
        while (__builtin_amdgcn_readfirstlane(lds_get(w_got)) < (unsigned)NLOAD) __builtin_amdgcn_s_sleep(2);
        int slot = 0;
#pragma unroll 1
        for (int c = 0; c < NCH; ++c) {
            if (c >= NWS) while (__builtin_amdgcn_ballot_w64(lds_get(w_read + (lane & 3) * 4) + (NWS - 1) < (unsigned)c) != 0) __builtin_amdgcn_s_sleep(2);
            const unsigned char* sb = p.w + (size_t)c * (CHK * NL);
            const unsigned ld = lds0 + LDS_S + slot * STAGE + h * (STAGE / NSTG);
#pragma unroll
            for (int i = 0; i < 32 / NSTG; ++i) glds16(sb + i * 8 * NL, (i & 2) ? wv1 : wv0, ld + i * 1024);
            wait_vm<0>();
            lds_set(w_land + h * 4, c + 1);
            slot = slot == NWS - 1 ? 0 : slot + 1;
        }
        return;
    }
    if (wave >= 4) {
        const int d = wave - 4;
        unsigned cv = 0;
        draw(cv, cnt, 0u, 1u);
        const unsigned eo = ENC_LO + (unsigned)(d * NGEMM + g), eu = unit_at(eo) % NU;
        const bool enc = eo < (unsigned)ENC_HI;
        unsigned xd[16];
        const u32x4 te = p.tab[lane & 15];
        if (enc) {
            const unsigned char* xs = xp + (size_t)((eu % NB16) * 16 + (lane & 15)) * KD + (eu / NB16) * CHK + (lane >> 4) * 16;
#pragma unroll
            for (int q = 0; q < 4; ++q) {
                const u32x4 v = *(const u32x4*)(xs + q * 64);
                xd[4 * q] = v[0]; xd[4 * q + 1] = v[1]; xd[4 * q + 2] = v[2]; xd[4 * q + 3] = v[3];
            }
        }
        const unsigned old = take<0>(cv), call = old + 1;
        if (lane == 0) __hip_atomic_fetch_add((LAS unsigned*)w_got, 1u, __ATOMIC_RELAXED, __HIP_MEMORY_SCOPE_WORKGROUP);
        if (enc) {
            u32x4 o[4];
            const unsigned m = encode_unit(xd, te, smem + LDS_A + d * 4096, lane, o);
            if (m) {
                unsigned char* eb = myws + WS_ENC + (size_t)eu * 4096;
                store_sys<0>(o[0], eb, sv); store_sys<1024>(o[1], eb, sv);
                store_sys<2048>(o[2], eb, sv); store_sys<3072>(o[3], eb, sv);
            }
            STAMP(20);
            wait_vm<0>();
            put_word(myws + WS_META + (size_t)eo * MSTR, 0u, (call << 12) | m);
            STAMP(21);
        }
        while (__builtin_amdgcn_readfirstlane(lds_get(w_tab)) < 4u) __builtin_amdgcn_s_sleep(2);
        unsigned* tmo = (unsigned*)(myws + WS_TMO);
        const unsigned par = (old & 1) * NR;
        const unsigned lv = (lane & 15) * KD + (lane >> 4) * 16;
        const unsigned nitem = S * NCH;
        unsigned dead = 0;
        unsigned o_unit, o_flag, o_gflag, o_nib;
        {
            unsigned src, b;
            block_of(lane < S ? lane : 0, src, b);
            const unsigned s = par + (rank + src) % NR;
            o_unit = src ? (unsigned)WS_SLOT + s * (unsigned)SLOT_BYTES + b * 4096 : b * 16 * KD;
            o_flag = (unsigned)WS_FLAG + (s * NB16 + b) * 128;
            o_gflag = (unsigned)WS_GFLAG + (s * 4 + (b >> 3)) * 128;
            o_nib = (b & 7u) * 4;
        }
#pragma unroll 1
        for (unsigned i = d; i < nitem;) {
            const unsigned sc = S == SL ? i / (2 * SL) : i / (2 * SL - 2), r = i - sc * 2 * S;
            const unsigned k = r >> 1, c = 2 * sc + (r & 1), sl = i % ARING;
            const bool own = k < (unsigned)nloc;
            unsigned info = INFO_RAW;
            while (__builtin_amdgcn_ballot_w64(lds_get(w_used + (lane & 3) * 4) + ARING <= i) != 0) __builtin_amdgcn_s_sleep(2);
            if (!own && !dead) {
                const u64* fp = (const u64*)(myws + __builtin_amdgcn_readlane(c < GC ? o_gflag : o_flag, k)) + c;
                const u64 t0 = __builtin_amdgcn_s_memrealtime();
                unsigned it = 0;
                while (true) {
                    const u64 fv = __hip_atomic_load(fp, __ATOMIC_RELAXED, __HIP_MEMORY_SCOPE_SYSTEM);
                    info = (__builtin_amdgcn_readfirstlane((unsigned)fv) >> __builtin_amdgcn_readlane(o_nib, k)) & 15u;
                    if (__builtin_amdgcn_readfirstlane((unsigned)(fv >> 32)) == call) break;
                    if ((++it & 255) == 0) {
                        const unsigned tv = __builtin_amdgcn_readfirstlane(
                            __hip_atomic_load(tmo, __ATOMIC_RELAXED, __HIP_MEMORY_SCOPE_AGENT));
                        if (tv == call || __builtin_amdgcn_s_memrealtime() - t0 > DEADLINE_TICKS) {
                            __hip_atomic_store(tmo, call, __ATOMIC_RELAXED, __HIP_MEMORY_SCOPE_AGENT);
                            dead = 0x80000000u;
                            break;
                        }
                    }
                    __builtin_amdgcn_s_sleep(POLL_SLEEP);
                }
                asm volatile("" ::: "memory");
            }
            STAMP(32 + c);
            if (!dead) {
                const unsigned ou = __builtin_amdgcn_readlane(o_unit, k);
                const unsigned char* sb = own ? xp + ou + c * CHK : myws + ou + c * (NB16 * 4096);
                const unsigned vo = own ? lv : sv, st = own ? 64 : 1024;
#pragma unroll
                for (int q = 0; q < 4; ++q) glds16_sys(sb + q * st, vo, lds0 + LDS_A + sl * 4096 + q * 1024);
            }
            if (c >= NCH - 2 && __builtin_amdgcn_readfirstlane(
                    __hip_atomic_load(tmo, __ATOMIC_RELAXED, __HIP_MEMORY_SCOPE_AGENT)) == call) dead = 0x80000000u;
            wait_vm<0>();
            STAMP(48 + c);
            if (WARM && c + WARM < NCH && !own)
                glds4_sys(myws + __builtin_amdgcn_readlane(o_unit, k) + (c + WARM) * (NB16 * 4096), cb * 1024 + (lane & 7) * 128,
                          lds0 + LDS_P + d * 256);
            if (info <= 13u && !dead) {
                unsigned char* unit = smem + LDS_A + sl * 4096;
                u32x4 pk[3];
#pragma unroll
                for (int q = 0; q < 3; ++q) pk[q] = *(const u32x4*)(unit + q * 1024 + sv);
                unsigned v[16];
                decode_unit(pk, unit, info, v);
#pragma unroll
                for (int q = 0; q < 4; ++q)
                    *(u32x4*)(unit + q * 1024 + sv) = u32x4{v[4 * q], v[4 * q + 1], v[4 * q + 2], v[4 * q + 3]};
            }
            lds_set(w_item + sl * 4, (i + 1) | dead);
            STAMP(2 + c);
            unsigned nx = 0;
            if (lane == 0) nx = __hip_atomic_fetch_add((LAS unsigned*)w_next, 1u, __ATOMIC_RELAXED, __HIP_MEMORY_SCOPE_WORKGROUP);
            i = NLOAD + __builtin_amdgcn_readfirstlane(nx);
        }
        STAMP_OUT();
        return;
    }

    {
        const unsigned b0 = symbol_entry(threadIdx.x), b1 = symbol_entry(threadIdx.x + 256);
#pragma unroll
        for (unsigned x = 0; x < 16; ++x)
            *(LAS unsigned*)(smem + LDS_T + x * 1024 + threadIdx.x * 4) = ((x & 1) ? b1 : b0) | symbol_entry((x >> 1) << 9);
    }
    asm volatile("" ::: "memory");
    if (lane == 0) __hip_atomic_fetch_add((LAS unsigned*)w_tab, 1u, __ATOMIC_RELAXED, __HIP_MEMORY_SCOPE_WORKGROUP);
    __builtin_amdgcn_s_setprio(MPRIO);
    const float scale = p.sx[0] * p.sw[0];
    const __amdgpu_buffer_rsrc_t orsrc = __builtin_amdgcn_make_buffer_rsrc((void*)p.out, 0, NR * MS * NL * 4, 0x00020000);
    const int G = lane >> 4, i16 = lane & 15;
    const int ldsb = ((16 * G + (i16 >> 1)) * 128 + (((i16 >> 2) & 3) | ((G & 1) << 2)) * 16 + 8 * (i16 & 1)) ^ (wave << 5);
    const int ocol = ((i16 & 7) * NL + cb * 128 + wave * 32 + (i16 >> 3) * 16 + G * 4) * 4;
    unsigned ctl = 0;
    auto wait_ctl = [&](int l, unsigned v) {
        unsigned x = __builtin_amdgcn_readlane(ctl, l);
        while ((x & 0x7fffffffu) <= v) {
            ctl = lds_get(w_item + (lane & 31) * 4);
            x = __builtin_amdgcn_readlane(ctl, l);
            if ((x & 0x7fffffffu) <= v) __builtin_amdgcn_s_sleep(1);
        }
        return x;
    };
    i32x8 wf[2][2], wl[2][2];
    auto read_w = [&](i32x8 (&z)[2][2], int slot) {
#pragma unroll
        for (int ks = 0; ks < 2; ++ks)
#pragma unroll
            for (int n = 0; n < 2; ++n) {
                const unsigned char* lp = smem + LDS_S + slot * STAGE + ks * 16384 + (ldsb ^ (n << 4));
                i32x2 t0 = __builtin_amdgcn_ds_read_tr8_b64_v2i32((LAS i32x2*)(lp));
                i32x2 t1 = __builtin_amdgcn_ds_read_tr8_b64_v2i32((LAS i32x2*)(lp + 1024));
                i32x2 t2 = __builtin_amdgcn_ds_read_tr8_b64_v2i32((LAS i32x2*)(lp + 8192));
                i32x2 t3 = __builtin_amdgcn_ds_read_tr8_b64_v2i32((LAS i32x2*)(lp + 8192 + 1024));
                z[ks][n] = i32x8{t0[0], t0[1], t1[0], t1[1], t2[0], t2[1], t3[0], t3[1]};
            }
    };
    f32x4 acc[SL][2];
#pragma unroll
    for (int k = 0; k < SL; ++k) acc[k][0] = acc[k][1] = f32x4{0.f, 0.f, 0.f, 0.f};
    asm volatile("" :: "v"(scale) : "memory");
    STAMP(1);
    unsigned i = 0, as = 0, bad = 0;
    unsigned rowoff;
    {
        unsigned src, b;
        block_of(lane < S ? lane : 0, src, b);
        rowoff = (((rank + src) % NR) * MS + b * 16) * (NL * 4);
        asm volatile("" : "+v"(rowoff));
        if (WARMO) {
            const unsigned z = __builtin_amdgcn_raw_buffer_load_b32(orsrc, rowoff + cb * 512, 0, 17);
            asm volatile("" :: "v"(z));
        }
    }
    u32x4 a[2][4];
    auto read_a = [&](u32x4 (&z)[4], unsigned v) {
        bad |= v >> 31;
        const unsigned char* unit = smem + LDS_A + as * 4096 + sv;
#pragma unroll
        for (int q = 0; q < 4; ++q) z[q] = *(const u32x4*)(unit + q * 1024);
        ++i; as = as + 1 == ARING ? 0 : as + 1;
        lds_set(w_used + wave * 4, i);
    };
    auto mul = [&](f32x4 (&ac)[2], const i32x8 (&w)[2][2], const u32x4 (&z)[4]) {
        const i32x8 xa0 = {(int)z[0][0], (int)z[0][1], (int)z[0][2], (int)z[0][3], (int)z[1][0], (int)z[1][1], (int)z[1][2], (int)z[1][3]};
        const i32x8 xa1 = {(int)z[2][0], (int)z[2][1], (int)z[2][2], (int)z[2][3], (int)z[3][0], (int)z[3][1], (int)z[3][2], (int)z[3][3]};
        ac[0] = __builtin_amdgcn_mfma_scale_f32_16x16x128_f8f6f4(w[0][0], xa0, ac[0], 0, 0, 0, 0, 0, 0);
        ac[1] = __builtin_amdgcn_mfma_scale_f32_16x16x128_f8f6f4(w[0][1], xa0, ac[1], 0, 0, 0, 0, 0, 0);
        ac[0] = __builtin_amdgcn_mfma_scale_f32_16x16x128_f8f6f4(w[1][0], xa1, ac[0], 0, 0, 0, 0, 0, 0);
        ac[1] = __builtin_amdgcn_mfma_scale_f32_16x16x128_f8f6f4(w[1][1], xa1, ac[1], 0, 0, 0, 0, 0, 0);
    };
#pragma unroll 1
    for (int c = 0; c < NCH; c += 2) {
        wait_ctl(24, c + 1); if (NSTG > 1) wait_ctl(25, c + 1);
        read_w(wf, 0);
        read_w(wl, 1);
        lds_set(w_read + wave * 4, c + 2);
#pragma unroll
        for (int k = 0; k < SL; ++k) {
            if (k < SL - 1 || S == SL) {
                read_a(a[0], wait_ctl(as, i));
                read_a(a[1], wait_ctl(as, i));
                if (c == NCH - 2) STAMP(32 + k);
                __builtin_amdgcn_sched_barrier(0);
                mul(acc[k], wf, a[0]);
                mul(acc[k], wl, a[1]);
                __builtin_amdgcn_sched_barrier(0);
                if (c == NCH - 2) {
                    int o1 = ocol + __builtin_amdgcn_readlane(rowoff, k), o2 = o1 + 8 * NL * 4;
                    asm volatile("" : "+v"(o1), "+v"(o2));
                    const float sc = bad ? __builtin_nanf("") : scale;
                    const u32x4 lo = __builtin_bit_cast(u32x4, acc[k][0] * sc), hi = __builtin_bit_cast(u32x4, acc[k][1] * sc);
                    u32x4 x, y;
#pragma unroll
                    for (int e = 0; e < 4; ++e) {
                        x[e] = __builtin_amdgcn_update_dpp(lo[e], hi[e], 0x128, 0xf, 0xc, false);
                        y[e] = __builtin_amdgcn_update_dpp(hi[e], lo[e], 0x128, 0xf, 0x3, false);
                    }
                    __builtin_amdgcn_raw_buffer_store_b128(x, orsrc, o1, 0, OUT_AUX);
                    __builtin_amdgcn_raw_buffer_store_b128(y, orsrc, o2, 0, OUT_AUX);
                }
            }
        }
        STAMP(2 + c); STAMP(3 + c);
    }
}

extern "C" void kernel_launch(int rank, int nranks, ncclComm_t comm,
                              hipStream_t stream,
                              void* const* d_in, const int* in_sizes, int n_in,
                              void* d_out, int out_size,
                              const int* in_dtypes, int out_dtype,
                              void* const* d_out_all,
                              void* const* d_ws_all, size_t ws_bytes) {
    (void)comm; (void)in_dtypes; (void)out_dtype; (void)d_out_all;
    if (nranks != NR || (unsigned)rank >= (unsigned)NR || n_in != 4 || in_sizes[0] != MS * KD ||
        in_sizes[1] != KD * NL || out_size != NR * MS * NL || ws_bytes < WS_NEED) {
        fprintf(stderr, "unexpected problem shape\n");
        return;
    }
    Params p;
    for (int j = 0; j < NR; ++j) p.ws[j] = (unsigned char*)d_ws_all[(rank + j) % NR];
    p.x = (const unsigned char*)d_in[0];
    p.w = (const unsigned char*)d_in[1];
    p.sx = (const float*)d_in[2];
    p.sw = (const float*)d_in[3];
    p.out = (float*)d_out;
    p.rank = rank;
    p.pad = 0;
    for (int e0 = 0; e0 < 16; ++e0) {
        unsigned t[4];
        encode_table(e0 < 13 ? e0 : 13, t);
        p.tab[e0] = u32x4{t[0], t[1], t[2], t[3]};
    }
    HIP_CHECK(hipFuncSetAttribute((const void*)ag_gemm, hipFuncAttributeMaxDynamicSharedMemorySize, LDS_BYTES));
    hipLaunchKernelGGL(ag_gemm, dim3(GRIDX), dim3(THREADS), LDS_BYTES, stream, p);
    HIP_CHECK(hipGetLastError());
}
```

```cpp
#include <hip/hip_runtime.h>
#include <rccl/rccl.h>
#include <cstdio>

#define HIP_CHECK(expr) do { hipError_t _e = (expr); if (_e != hipSuccess) { \
    fprintf(stderr, "HIP error %s at %s:%d\n", hipGetErrorString(_e), __FILE__, __LINE__); \
    return; } } while (0)

typedef unsigned u32x4 __attribute__((ext_vector_type(4)));
typedef unsigned u32x2 __attribute__((ext_vector_type(2)));
typedef int i32x2 __attribute__((ext_vector_type(2)));
typedef int i32x8 __attribute__((ext_vector_type(8)));
typedef float f32x4 __attribute__((ext_vector_type(4)));
typedef unsigned long long u64;
#define LAS __attribute__((address_space(3)))

#define WIN 6
#define DEPTH 5
#define NRAW 4
#define ENC_HI 480
#define RUSH 112
#define WARM 12

constexpr int NR = 8;
constexpr int KD = 4096;
constexpr int MS = 512;
constexpr int NL = 512;
constexpr int NB16 = MS / 16;
constexpr int CHK = 256;
constexpr int NCH = KD / CHK;
constexpr int NU = NB16 * NCH;
constexpr int PPP = 8;
constexpr int NFEED = 3;
constexpr int NLOAD = 7;
constexpr int NWAVE = 4 + NLOAD + 1;
constexpr int RING = 16;
constexpr int GEMM0 = 7 * PPP;
constexpr unsigned NPOS0 = NRAW + NWAVE - (NFEED + 2);
constexpr unsigned NFIX = PPP * NPOS0;
constexpr unsigned ENC_LO = NFIX;
constexpr int NRW = 50;
constexpr int SL = 6;
constexpr int NGEMM = 4 * NRW;
constexpr int GRIDX = GEMM0 + NGEMM;
constexpr int THREADS = 64 * NWAVE;
constexpr int STAGE = CHK * 128;
constexpr int ARING = 19;
constexpr int LDS_T = 0;
constexpr int LDS_X = 3072;
constexpr int LDS_S = 16384;
constexpr int LDS_A = LDS_S + 2 * STAGE;
constexpr int LDS_W = LDS_A + ARING * 4096 + 64;
constexpr int LDS_P = LDS_W + 256;
constexpr int LDS_BYTES = LDS_P + 8 * 256;
static_assert(GRIDX <= 256 && LDS_BYTES <= 163840 && ENC_HI - ENC_LO <= 4 * NGEMM, "resident grid");
static_assert(DEPTH >= 1 && DEPTH <= RING && NRAW <= NFEED + 1 && NPOS0 > NFEED && NPOS0 <= RING, "push geometry");
static_assert(LDS_X + NLOAD * 1600 <= LDS_S && ARING <= 24 && RING * 4096 + 7168 <= LDS_BYTES, "LDS map");

__device__ __forceinline__ unsigned unit_at(unsigned q) {
    return (q & ~63u) + (q & 1u) * NB16 + ((q & 63u) >> 1);
}

constexpr int FS = 32, MSTR = 32;
constexpr size_t WS_FLAG = 1536 << 10;
constexpr size_t WS_TMO = 64 << 10;
constexpr size_t WS_IDLE = 0;
constexpr size_t WS_TICKET = 96 << 10;
constexpr size_t WS_CNT = 128 << 10;
constexpr size_t WS_META = 256 << 10;
constexpr size_t WS_TRACE = 512 << 10;
constexpr size_t SLOT_BYTES = (size_t)MS * KD;
constexpr size_t WS_ENC = 2 << 20;
constexpr size_t WS_SLOT = 4 << 20;
constexpr size_t WS_NEED = WS_SLOT + 2 * NR * SLOT_BYTES;
constexpr unsigned INFO_RAW = 15u;
constexpr u64 DEADLINE_TICKS = 100000000ull;

struct Params {
    unsigned char* ws[NR];
    const unsigned char* x;
    const unsigned char* w;
    const float* sx;
    const float* sw;
    float* out;
    int rank;
    int pad;
    u32x4 tab[16];
};

#define STAMP(i) do {} while (0)
#define STAMP_OUT() do {} while (0)

template <int N> __device__ __forceinline__ void wait_vm() {
    asm volatile("s_waitcnt vmcnt(%0)" :: "i"(N) : "memory");
}
__device__ __forceinline__ void wait_lds() { asm volatile("s_waitcnt lgkmcnt(0)" ::: "memory"); }
__device__ __forceinline__ void glds16(const unsigned char* sbase, unsigned voff, unsigned lds) {
    unsigned keep;
    asm volatile("s_mov_b32 %0, m0\n\ts_mov_b32 m0, %3\n\ts_nop 0\n\tglobal_load_lds_dwordx4 %1, %2\n\ts_mov_b32 m0, %0"
                 : "=&s"(keep) : "v"(voff), "s"(sbase), "s"(lds) : "memory");
}
__device__ __forceinline__ void glds16_sys(const unsigned char* sbase, unsigned voff, unsigned lds) {
    unsigned keep;
    asm volatile("s_mov_b32 %0, m0\n\ts_mov_b32 m0, %3\n\ts_nop 0\n\tglobal_load_lds_dwordx4 %1, %2 sc0 sc1\n\ts_mov_b32 m0, %0"
                 : "=&s"(keep) : "v"(voff), "s"(sbase), "s"(lds) : "memory");
}
__device__ __forceinline__ void glds4_sys(const unsigned char* sbase, unsigned voff, unsigned lds) {
    unsigned keep;
    asm volatile("s_mov_b32 %0, m0\n\ts_mov_b32 m0, %3\n\ts_nop 0\n\tglobal_load_lds_dword %1, %2 sc0 sc1\n\ts_mov_b32 m0, %0"
                 : "=&s"(keep) : "v"(voff), "s"(sbase), "s"(lds) : "memory");
}
template <int OFF> __device__ __forceinline__ void store_sys(const u32x4& d, unsigned char* sbase, unsigned voff) {
    asm volatile("global_store_dwordx4 %1, %0, %2 offset:%3 sc0 sc1\n\ts_nop 1" :: "v"(d), "v"(voff), "s"(sbase), "i"(OFF) : "memory");
}
template <int OFF> __device__ __forceinline__ void store_sys_short(const u32x4& d, unsigned char* sbase, unsigned voff, unsigned drop) {
    unsigned t;
    asm volatile("s_nop 0\n\tv_readfirstlane_b32 %0, %5\n\ts_lshr_b64 exec, -1, %0\n\t"
                 "global_store_dwordx4 %2, %1, %3 offset:%4 sc0 sc1\n\ts_mov_b64 exec, -1\n\ts_nop 0"
                 : "=&s"(t) : "v"(d), "v"(voff), "s"(sbase), "i"(OFF), "v"(drop) : "memory", "scc");
}
__device__ __forceinline__ void put_word(unsigned char* sbase, unsigned zero, unsigned val) {
    asm volatile("s_mov_b64 exec, 1\n\tglobal_store_dword %0, %1, %2 sc0 sc1\n\ts_mov_b64 exec, -1\n\ts_nop 0"
                 :: "v"(zero), "v"(val), "s"(sbase) : "memory");
}
__device__ __forceinline__ void put_flag(unsigned char* sbase, unsigned voff, unsigned lo, unsigned hi) {
    const u64 v = ((u64)hi << 32) | lo;
    asm volatile("s_mov_b64 exec, %3\n\tglobal_store_dwordx2 %0, %1, %2 sc0 sc1\n\ts_mov_b64 exec, -1\n\ts_nop 0"
                 :: "v"(voff), "v"(v), "s"(sbase), "i"((1 << (FS / 8)) - 1) : "memory");
}
__device__ __forceinline__ void draw(unsigned& t, unsigned* ctr, unsigned zero, unsigned one) {
    asm volatile("s_mov_b64 exec, 1\n\tglobal_atomic_add %0, %1, %2, %3 sc0\n\ts_mov_b64 exec, -1"
                 : "+v"(t) : "v"(zero), "v"(one), "s"(ctr) : "memory");
}
template <int N> __device__ __forceinline__ unsigned take(unsigned& t) {
    asm volatile("s_waitcnt vmcnt(%1) ; take" : "+v"(t) : "i"(N) : "memory");
    return __builtin_amdgcn_readfirstlane(t);
}
__device__ __forceinline__ unsigned lds_get(unsigned char* a) {
    const unsigned v = __hip_atomic_load((LAS unsigned*)a, __ATOMIC_RELAXED, __HIP_MEMORY_SCOPE_WORKGROUP);
    asm volatile("" ::: "memory");
    return v;
}
__device__ __forceinline__ void lds_set(unsigned char* a, unsigned v) {
    wait_lds();
    __hip_atomic_store((LAS unsigned*)a, v, __ATOMIC_RELAXED, __HIP_MEMORY_SCOPE_WORKGROUP);
}

static_assert(WIN >= 5 && WIN <= 8, "window");
constexpr int FJ = (3 + WIN) % 4;
constexpr int window_n(int j) {
    int c = WIN - 1;
    for (int q = j - WIN + 1; q < j; ++q) c += (q + 8) % 4 == FJ;
    return c;
}

__device__ __forceinline__ unsigned scan_incl(unsigned v) {
    v += __builtin_amdgcn_update_dpp(0u, v, 0x111, 0xf, 0xf, false);
    v += __builtin_amdgcn_update_dpp(0u, v, 0x112, 0xf, 0xf, false);
    v += __builtin_amdgcn_update_dpp(0u, v, 0x114, 0xf, 0xf, false);
    v += __builtin_amdgcn_update_dpp(0u, v, 0x118, 0xf, 0xf, false);
    v += __builtin_amdgcn_update_dpp(0u, v, 0x142, 0xa, 0xf, false);
    v += __builtin_amdgcn_update_dpp(0u, v, 0x143, 0xc, 0xf, false);
    return v;
}
__device__ __forceinline__ void symbol_entries(unsigned x, unsigned& lo, unsigned& sel, unsigned& two) {
    lo = sel = two = 0;
    unsigned r = 4;
#pragma unroll
    for (unsigned i = 0; i < 4; ++i) {
        const unsigned y = (x >> (2 * i)) & 3u;
        lo |= ((0x40182000u >> (8 * y)) & 0xffu) << (8 * i);
        sel |= (y ? i : r) << (8 * i);
        r += y == 0;
        two |= ((0x48000810u >> (8 * y)) & 0xffu) << (8 * i);
    }
}
__host__ __device__ inline void encode_table(int e0, unsigned* t) {
    unsigned a[16];
#pragma unroll
    for (int e = 0; e < 16; ++e) {
        const int q = e - e0;
        a[e] = (q >= 0 && q <= 2) ? (unsigned)((q + 1) << 3) : (q == -1 || q == -2) ? (unsigned)(4 - q) : q == 3 ? 7u
             : (q <= -3 && q >= -5) ? (unsigned)(0x84 | ((-3 - q) << 5)) : q == 4 ? 0xe4u : 0x0cu;
    }
    auto sg = [](unsigned v) { return (v & 0x80u) ? 0xffu : 0u; };
#pragma unroll
    for (int i = 5; i < 8; ++i) { a[i] ^= 0xffu; a[8 + i] ^= 0xffu; }
    a[3] ^= sg(a[15]); a[2] ^= sg(a[13]); a[11] ^= sg(a[7]); a[10] ^= sg(a[5]);
    a[1] ^= sg(a[11]); a[9] ^= sg(a[3]);
    a[0] ^= sg(a[9]); a[8] ^= sg(a[1]);
#pragma unroll
    for (int i = 0; i < 4; ++i) t[i] = a[4 * i] | (a[4 * i + 1] << 8) | (a[4 * i + 2] << 16) | (a[4 * i + 3] << 24);
}
__device__ __forceinline__ unsigned encode_unit(const unsigned (&d)[16], const u32x4& te, unsigned char* scr,
                                                int lane, u32x4 (&o)[4]) {
    *(u32x4*)(scr + lane * 16) = u32x4{0u, 0u, 0u, 0u};
    unsigned es = 0;
#pragma unroll
    for (int i = 0; i < 16; i += 5) es = __builtin_amdgcn_sad_u8(d[i] & 0x78787878u, 0u, es);
    int e0 = ((int)__builtin_amdgcn_readlane(scan_incl(es), 63) - 1638) >> 13;
    e0 = e0 < 0 ? 0 : (e0 > 13 ? 13 : e0);
    const unsigned t0 = __builtin_amdgcn_readlane(te[0], e0), t1 = __builtin_amdgcn_readlane(te[1], e0);
    const unsigned t2 = __builtin_amdgcn_readlane(te[2], e0), t3 = __builtin_amdgcn_readlane(te[3], e0);
    unsigned c[16], bad = 0;
    unsigned s1[4], n1[4], s2[4], n2[4];
#pragma unroll
    for (int h = 0; h < 4; ++h) {
        unsigned v[4], n[4], w[4], k2[4];
#pragma unroll
        for (int k = 0; k < 4; ++k) {
            const unsigned x = d[4 * h + k];
            const unsigned e = (x >> 3) & 0x0f0f0f0fu;
            const unsigned inf = __builtin_amdgcn_perm(t1, t0, e) ^ __builtin_amdgcn_perm(t3, t2, e ^ 0x08080808u);
            c[4 * h + k] = (x & 0x87878787u) | (inf & 0x18181818u);
            bad |= inf & (inf >> 1);
            const unsigned f1 = inf & 0x04040404u, f2 = inf & 0x80808080u;
            v[k] = __builtin_amdgcn_udot4(inf & 0x03030303u, __builtin_amdgcn_perm(0u, 0x40100401u, f1 * 0x404040u), 0u, false);
            w[k] = __builtin_amdgcn_udot4((inf >> 5) & 0x03030303u, __builtin_amdgcn_perm(0u, 0x40100401u, f2 * 0x20202u), 0u, false);
            n[k] = __builtin_popcount(f1) + (k ? n[k - 1] : 0u);
            k2[k] = __builtin_popcount(f2) + (k ? k2[k - 1] : 0u);
        }
        s1[h] = v[0] | (v[1] << (2 * n[0])) | (v[2] << (2 * n[1])) | (v[3] << (2 * n[2]));
        s2[h] = w[0] | (w[1] << (2 * k2[0])) | (w[2] << (2 * k2[1])) | (w[3] << (2 * k2[2]));
        n1[h] = 2 * n[3]; n2[h] = 2 * k2[3];
    }
#pragma unroll
    for (int q = 0; q < 4; ++q) {
        const unsigned z = c[4 * q + 3];
        const unsigned p0 = c[4 * q] | ((z << 5) & 0x60606060u);
        const unsigned p1 = c[4 * q + 1] | ((z << 3) & 0x60606060u);
        const unsigned p2 = c[4 * q + 2] | ((z << 1) & 0x20202020u) | ((z >> 1) & 0x40404040u);
        o[(3 * q) >> 2][(3 * q) & 3] = p0; o[(3 * q + 1) >> 2][(3 * q + 1) & 3] = p1; o[(3 * q + 2) >> 2][(3 * q + 2) & 3] = p2;
    }
    const unsigned m1 = n1[0] + n1[1], m2 = n2[0] + n2[1] + n2[2] + n2[3];
    const unsigned tot = (m1 + n1[2] + n1[3]) | (m2 << 16);
    const unsigned inc = scan_incl(tot), sum = __builtin_amdgcn_readlane(inc, 63);
    const unsigned at2 = 16u + (sum & 0xffffu), bits = at2 + (sum >> 16);
    if (__builtin_amdgcn_ballot_w64((bad & 0x04040404u) || m2 > 64u) != 0 || at2 > 16u + 64u * 48u || bits > 8192u - 128u) return 0u;
    auto put = [&](u64 str, unsigned ob) {
        unsigned* wp = (unsigned*)(scr + (ob >> 5) * 4);
        const u64 a = (u64)(unsigned)str << (ob & 31u), b = (u64)(unsigned)(str >> 32) << (ob & 31u);
        __hip_atomic_fetch_or(wp, (unsigned)a, __ATOMIC_RELAXED, __HIP_MEMORY_SCOPE_WORKGROUP);
        __hip_atomic_fetch_or(wp + 1, (unsigned)(a >> 32) | (unsigned)b, __ATOMIC_RELAXED, __HIP_MEMORY_SCOPE_WORKGROUP);
        __hip_atomic_fetch_or(wp + 2, (unsigned)(b >> 32), __ATOMIC_RELAXED, __HIP_MEMORY_SCOPE_WORKGROUP);
    };
    const unsigned ex = inc - tot, o1 = 16u + (ex & 0xffffu);
    put(s1[0] | ((u64)s1[1] << n1[0]), o1);
    put(s1[2] | ((u64)s1[3] << n1[2]), o1 + m1);
    put(s2[0] | ((u64)s2[1] << n2[0]) | ((u64)s2[2] << ((n2[0] + n2[1]) & 63u)) | ((u64)s2[3] << ((m2 - n2[3]) & 63u)), at2 + (ex >> 16));
    if (lane == 0) __hip_atomic_fetch_or((unsigned*)scr, at2, __ATOMIC_RELAXED, __HIP_MEMORY_SCOPE_WORKGROUP);
    asm volatile("" ::: "memory");
    {
        const unsigned g0 = 16u + lane * 48u;
        const unsigned* gp = (const unsigned*)(scr + ((g0 >> 3) & ~3u));
        u64 g = (gp[0] | ((u64)gp[1] << 32)) >> (g0 & 16u);
        const int left = (int)at2 - (int)g0;
        g |= ~0ull << (left <= 0 ? 0 : (left < 48 ? left : 48));
        if (__builtin_amdgcn_ballot_w64(__builtin_popcountll(~(g | (g >> 1)) & 0x5555555555555555ull) > 15) != 0) return 0u;
    }
    asm volatile("s_waitcnt lgkmcnt(0)" ::: "memory");
    o[3] = *(const u32x4*)(scr + lane * 16);
    const unsigned nl = ((bits + 255u) >> 8) * 2u;
    return (nl << 4) | (unsigned)e0;
}
__device__ __forceinline__ void decode_unit(const unsigned char* slot, unsigned char* ex, int lane, unsigned e0, unsigned (&d)[16]) {
    const unsigned g0 = 2u + lane * 6u;
    const unsigned* gp = (const unsigned*)(slot + 3072 + (g0 & ~3u));
    const unsigned ga = gp[0], gb = gp[1], at2 = *(const unsigned*)(slot + 3072) & 0xffffu;
    unsigned p[12];
#pragma unroll
    for (int q = 0; q < 3; ++q) {
        const u32x4 v = *(const u32x4*)(slot + q * 1024 + lane * 16);
        p[4 * q] = v[0]; p[4 * q + 1] = v[1]; p[4 * q + 2] = v[2]; p[4 * q + 3] = v[3];
    }
    const u64 g = (ga | ((u64)gb << 32)) >> ((g0 & 2u) * 8);
    const unsigned gl = (unsigned)g, gh = (unsigned)(g >> 32);
    u32x2 ta[6];
#pragma unroll
    for (int i = 0; i < 6; ++i) {
        const unsigned w = i < 4 ? gl : gh, sh = 8 * (i & 3);
        ta[i] = *(const LAS u32x2*)(size_t)((sh ? w >> (sh - 3) : w << 3) & 0x7f8u);
    }
    const unsigned zl = (~(gl | (gl >> 1)) & 0x55555555u) * 3u, zh = (~(gh | (gh >> 1)) & 0x5555u) * 3u;
    unsigned z[7];
    z[0] = 0; z[1] = __builtin_popcount(zl & 0xffu); z[2] = __builtin_popcount(zl & 0xffffu); z[3] = __builtin_popcount(zl & 0xffffffu);
    z[4] = __builtin_popcount(zl); z[5] = __builtin_popcount(zh & 0xffu) + z[4]; z[6] = __builtin_popcount(zh) + z[4];
#pragma unroll
    for (int q = 0; q < 4; ++q) {
        d[4 * q] = p[3 * q] & 0x9f9f9f9fu; d[4 * q + 1] = p[3 * q + 1] & 0x9f9f9f9fu; d[4 * q + 2] = p[3 * q + 2] & 0x9f9f9f9fu;
        d[4 * q + 3] = ((p[3 * q] >> 5) & 0x03030303u) | ((p[3 * q + 1] >> 3) & 0x0c0c0c0cu) |
                       ((p[3 * q + 2] >> 1) & 0x10101010u) | ((p[3 * q + 2] << 1) & 0x80808080u);
    }
    const unsigned o2 = at2 + scan_incl(z[6]) - z[6] - 2u;
    const unsigned* wp = (const unsigned*)(slot + 3072 + ((o2 >> 5) & 255u) * 4);
    const unsigned w0 = wp[0], w1 = wp[1];
    unsigned f[16], c[17];
    c[0] = 0;
#pragma unroll
    for (int i = 0; i < 16; ++i) {
        f[i] = ~(d[i] | (d[i] >> 1)) & 0x08080808u;
        c[i + 1] = __builtin_popcount(f[i]) + c[i];
    }
    const unsigned win = __builtin_amdgcn_alignbit(w1, w0, o2);
    unsigned tb[6];
#pragma unroll
    for (int i = 0; i < 6; ++i) tb[i] = *(const LAS unsigned*)(size_t)(2048u + ((win >> (z[i] & 31u)) & 0x3fcu));
    const unsigned base = (unsigned)(size_t)(LAS unsigned char*)ex + scan_incl(c[16]) - c[16];
#pragma unroll
    for (int i = 0; i < 6; i += 2)
        *(u32x2*)(ex + lane * 24 + i * 4) = u32x2{__builtin_amdgcn_perm(tb[i], ta[i][0], ta[i][1]),
                                                  __builtin_amdgcn_perm(tb[i + 1], ta[i + 1][0], ta[i + 1][1])};
    asm volatile("" ::: "memory");
    unsigned ya[16], yb[16];
#pragma unroll
    for (int i = 0; i < 16; ++i) {
        const LAS unsigned* yp = (const LAS unsigned*)(size_t)((base + c[i]) & ~3u);
        ya[i] = yp[0]; yb[i] = yp[1];
    }
    const unsigned bias = (e0 - 5u) * 0x08080808u;
#pragma unroll
    for (int i = 0; i < 16; ++i) {
        const unsigned y = __builtin_amdgcn_alignbyte(yb[i], ya[i], base + c[i]);
        const unsigned sel = __umul24(f[i], 0x202020u) + (f[i] >> 1);
        d[i] = d[i] + __builtin_amdgcn_perm(y, 0x20202020u, sel) + bias;
    }
}

__global__ void __launch_bounds__(THREADS, 3) ag_gemm(Params p) {
    extern __shared__ __attribute__((aligned(16))) unsigned char smem[];
    const int lane = threadIdx.x & 63;
    const int wave = __builtin_amdgcn_readfirstlane(threadIdx.x >> 6);
    const unsigned j = 1 + blockIdx.x % (NR - 1);
    unsigned char* const myws = p.ws[0];
    unsigned char* const pws = p.ws[j];
    const unsigned char* const xp = p.x;
    const unsigned rank = p.rank;
    asm volatile("" :: "s"(myws), "s"(pws), "s"(xp), "s"(rank));
    const unsigned lds0 = (unsigned)(size_t)(LAS unsigned char*)smem;
    unsigned* const cnt = (unsigned*)(myws + WS_CNT + (size_t)(blockIdx.x * 16 + wave) * 32);
    const unsigned sv = lane * 16;
    STAMP(0);

    if (__builtin_expect(blockIdx.x < GEMM0, 1)) {
        constexpr unsigned POS_RUSH = 1u << 29, POS_UNIT = 1u << 30, POS_END = 1u << 31;
        const unsigned pk = blockIdx.x / (NR - 1);
        const unsigned lv = (lane & 15) * KD + (lane >> 4) * 16;
        unsigned char* const w_pos = smem + RING * 4096;
        unsigned char* const w_meta = w_pos + 4096;
        unsigned char* const w_cons = w_meta + NU * 4;
        unsigned char* const w_done = w_cons + 4;
        unsigned* const tick = (unsigned*)(myws + WS_TICKET + j * 128);
        const bool feeder = wave >= 1 && wave <= NFEED, packer = wave >= NFEED + 2;
        unsigned* const ctr = feeder ? tick : cnt;
        const unsigned n0 = packer ? NRAW + wave - (NFEED + 2) : wave;
        unsigned u = unit_at(pk + PPP * n0);
        unsigned cv = 0;
        draw(cv, ctr, 0u, 1u);
        if (wave < NRAW || packer) {
            const unsigned char* sb = xp + (size_t)(u % NB16) * 16 * KD + (u / NB16) * CHK;
#pragma unroll
            for (int q = 0; q < 4; ++q) glds16(sb + q * 64, lv, lds0 + n0 * 4096 + q * 1024);
        } else if (wave > NFEED) {
#pragma unroll
            for (int q = 0; q < 7; ++q) *(LAS u32x4*)(w_pos + q * 1024 + sv) = u32x4{0u, 0u, 0u, 0u};
            wait_lds();
        }
        u32x4 te = {0u, 0u, 0u, 0u};
        if (packer) te = p.tab[lane & 15];
        const u64 tstart = __builtin_amdgcn_s_memrealtime();
        STAMP(6);
        __builtin_amdgcn_s_barrier();
        unsigned t = take<0>(cv);
        if (j >= (unsigned)NR) return;
        if (__builtin_expect(packer, 0)) {
            unsigned xd[16];
            unsigned char* slot = smem + n0 * 4096;
#pragma unroll
            for (int q = 0; q < 4; ++q) {
                const u32x4 v = *(const u32x4*)(slot + q * 1024 + sv);
                xd[4 * q] = v[0]; xd[4 * q + 1] = v[1]; xd[4 * q + 2] = v[2]; xd[4 * q + 3] = v[3];
            }
            u32x4 o[4];
            const unsigned m = encode_unit(xd, te, slot + 3072, lane, o);
            if (m) { *(u32x4*)(slot + sv) = o[0]; *(u32x4*)(slot + 1024 + sv) = o[1]; *(u32x4*)(slot + 2048 + sv) = o[2]; }
            else *(u32x4*)(slot + 3072 + sv) = u32x4{xd[12], xd[13], xd[14], xd[15]};
            lds_set(w_pos + n0 * 4, POS_UNIT | (u << 12) | (m ? (64u - (m >> 4)) | ((m & 15u) << 6) : INFO_RAW << 6));
            STAMP(2);
            STAMP_OUT();
            return;
        }
        if (feeder) {
            unsigned n = wave, m = 0, rush = 0;
            while (true) {
                if (n < NRAW || n >= NPOS0)
                    lds_set(w_pos + n * 4, POS_UNIT | rush | (u << 12) |
                                           (m ? (64u - (m >> 4)) | ((m & 15u) << 6) : INFO_RAW << 6));
                n = n < NPOS0 ? NPOS0 + n - 1 : n + NFEED;
                const unsigned o = NFIX + t % NU;
                if (o >= (unsigned)NU) { lds_set(w_pos + n * 4, POS_END); return; }
                while (n >= __builtin_amdgcn_readfirstlane(lds_get(w_cons)) + DEPTH) __builtin_amdgcn_s_sleep(1);
                u = unit_at(o);
                rush = o >= NU - RUSH && n * PPP + 4 < o ? POS_RUSH : 0u;
                m = __builtin_amdgcn_readfirstlane(lds_get(w_meta + o * 4));
                if (m == 0 && o >= ENC_LO && o < ENC_HI) {
                    const unsigned e = __builtin_amdgcn_readfirstlane(__hip_atomic_load(
                        (unsigned*)(myws + WS_META + (size_t)o * MSTR), __ATOMIC_RELAXED, __HIP_MEMORY_SCOPE_SYSTEM));
                    if ((e >> 12) == ((t / NU + 1) & 0xfffffu)) m = e & 0xfffu;
                    asm volatile("" ::: "memory");
                }
                if ((m >> 4) < 1u || (m >> 4) > 64u) m = 0;
                unsigned tv = 0;
                draw(tv, ctr, 0u, 1u);
                const unsigned char* sb = m ? myws + WS_ENC + (size_t)u * 4096
                                            : xp + (size_t)(u % NB16) * 16 * KD + (u / NB16) * CHK;
                const unsigned vo = m ? sv : lv, st = m ? 1024 : 64;
#pragma unroll
                for (int q = 0; q < 4; ++q) glds16_sys(sb + q * st, vo, lds0 + (n % RING) * 4096 + q * 1024);
                t = take<0>(tv);
            }
        }
        const unsigned old = t, call = old + 1;
        if (__builtin_expect(wave == 0, 1)) {
            unsigned char* const dst = pws + WS_SLOT + (size_t)((old & 1) * NR + rank) * SLOT_BYTES;
            unsigned char* const fl = pws + WS_FLAG + (size_t)((old & 1) * NR + rank) * (NU * FS);
            unsigned char* const idle = pws + WS_IDLE + (rank * 64 + blockIdx.x) * 128;
            __builtin_amdgcn_s_setprio(3);
            STAMP(1);
            unsigned n = 0, live = (1u << NFEED) - 1u, dsc = POS_UNIT | (u << 12) | (INFO_RAW << 6);
            unsigned d1 = 0, d2 = 0;
            auto flag = [&](unsigned dd) {
                const unsigned h = __builtin_amdgcn_readfirstlane(dd), uu = (h >> 12) & 0x1ffu;
                put_flag(h ? fl + ((uu % NB16) * NCH + uu / NB16) * FS : idle, lane * 8, (h >> 6) & 15u, call);
            };
            auto look = [&]() { return __builtin_amdgcn_readfirstlane(lds_get(w_pos + n * 4)); };
            auto acquire = [&]() {
                while (live) {
                    if ((live >> (n % NFEED)) & 1u) {
                        dsc = look();
                        for (int k = 0; k < 6 && !dsc; ++k) dsc = look();
                        if (!dsc) {
                            wait_vm<5>(); flag(d2); d2 = 0;
                            dsc = look();
                            if (!dsc) {
                                wait_vm<1>(); flag(d1); d1 = 0;
                                do dsc = look(); while (!dsc);
                            }
                        }
                        if (dsc & POS_UNIT) return true;
                        live &= ~(1u << (n % NFEED));
                    }
                    n += 1;
                    lds_set(w_cons, n);
                }
                return false;
            };
            u32x4 r[4], q[4];
            auto grab = [&](u32x4 (&z)[4]) {
#pragma unroll
                for (int k = 0; k < 4; ++k) z[k] = *(const u32x4*)(smem + (n % RING) * 4096 + k * 1024 + sv);
            };
            auto send = [&](u32x4 (&a)[4], u32x4 (&z)[4]) {
                const unsigned vo = (dsc & 0x1ff000u) | sv, d0 = dsc;
                n += 1;
                unsigned w = lds_get(w_pos + n * 4);
                const bool pace = !(dsc & POS_RUSH);
                if (FJ == 0) { wait_vm<window_n(0)>(); flag(d2); } else if (pace) wait_vm<window_n(0)>();
                store_sys<0>(a[0], dst, vo);
                if (FJ == 1) { wait_vm<window_n(1)>(); flag(d2); } else if (pace) wait_vm<window_n(1)>();
                store_sys<1024>(a[1], dst, vo);
                asm volatile("" : "+v"(w));
                dsc = __builtin_amdgcn_readfirstlane(w);
                bool ok = (dsc & POS_UNIT) != 0;
                if (ok) grab(z);
                if (FJ == 2) { wait_vm<window_n(2)>(); flag(d2); } else if (pace) wait_vm<window_n(2)>();
                store_sys<2048>(a[2], dst, vo);
                if (FJ == 3) { wait_vm<window_n(3)>(); flag(d2); }
                store_sys_short<3072>(a[3], dst, vo, d0);
                d2 = d1; d1 = d0;
                if (!ok) {
                    STAMP(3);
                    ok = acquire();
                    if (ok) grab(z);
                }
                lds_set(w_cons, n + 1);
                return ok;
            };
            STAMP(5);
            grab(r); lds_set(w_cons, 1u);
            while (send(r, q) && send(q, r)) {}
            wait_vm<5>(); flag(d2);
            wait_vm<1>(); flag(d1);
            lds_set(w_done, 1u);
            if (lane == 0) __hip_atomic_fetch_add(tick, NPOS0 - NFEED, __ATOMIC_RELAXED, __HIP_MEMORY_SCOPE_AGENT);
            STAMP(2);
            STAMP_OUT();
            return;
        }
        constexpr int NQ = (ENC_HI - ENC_LO + 63) / 64;
        const __amdgpu_buffer_rsrc_t mr = __builtin_amdgcn_make_buffer_rsrc(myws + WS_META, 0, ENC_HI * MSTR, 0x00020000);
        while (__builtin_amdgcn_s_memrealtime() - tstart < 450) __builtin_amdgcn_s_sleep(4);
        while (true) {
            unsigned e[NQ];
#pragma unroll
            for (int q = 0; q < NQ; ++q) e[q] = __builtin_amdgcn_raw_buffer_load_b32(mr, (ENC_LO + q * 64 + lane) * MSTR, 0, 17);
            bool ok = true;
#pragma unroll
            for (int q = 0; q < NQ; ++q) {
                const unsigned o = ENC_LO + q * 64 + lane;
                if ((e[q] >> 12) == (call & 0xfffffu) && o < (unsigned)ENC_HI) *(LAS unsigned*)(w_meta + o * 4) = (e[q] & 0xfffu) ? (e[q] & 0xfffu) : 1u;
                else if (o < (unsigned)ENC_HI) ok = false;
            }
            if (__builtin_amdgcn_ballot_w64(!ok) == 0 || __builtin_amdgcn_readfirstlane(lds_get(w_done))) break;
            __builtin_amdgcn_s_sleep(16);
        }
        return;
    }

    if (lds0 != 0) return;
    const int g = blockIdx.x - GEMM0;
    const int cb = g & 3, rg = g >> 2;
    static_assert(7 * NB16 == 4 * NRW + 24 && NB16 == 26 + 6, "block deal");
    const int nloc = rg < 24 ? 0 : (rg < 44 ? 1 : 2), S = rg < 44 ? 5 : 6;
    auto block_of = [&](unsigned k, unsigned& src, unsigned& b) {
        const unsigned t = rg + (k - nloc) * NRW;
        const bool own = k < (unsigned)nloc;
        src = own ? 0u : 1u + t % 7u;
        b = own ? (k ? rg - 18 : rg - 24) : t / 7u;
    };
    unsigned char* const w_item = smem + LDS_W;
    unsigned char* const w_land = w_item + 96;
    unsigned char* const w_got = w_item + 104;
    unsigned char* const w_tab = w_item + 108;
    unsigned char* const w_next = w_item + 112;
    unsigned char* const w_used = w_item + 128;
    unsigned char* const w_read = w_item + 144;
    if (threadIdx.x < 64) *(LAS unsigned*)(w_item + threadIdx.x * 4) = 0u;
    wait_lds();
    __builtin_amdgcn_s_barrier();

    if (wave == 4 + NLOAD) {
        const unsigned wv0 = (lane >> 3) * NL + cb * 128 + (((lane & 7) ^ ((lane >> 4) & 3)) << 4);
        const unsigned wv1 = wv0 ^ 64;
        while (__builtin_amdgcn_readfirstlane(lds_get(w_got)) < (unsigned)NLOAD) __builtin_amdgcn_s_sleep(2);
#pragma unroll 1
        for (int c = 0; c < NCH; ++c) {
            while (__builtin_amdgcn_ballot_w64(lds_get(w_read + (lane & 3) * 4) + 1 < (unsigned)c) != 0) __builtin_amdgcn_s_sleep(2);
            const unsigned char* sb = p.w + (size_t)c * (CHK * NL);
            const unsigned ld = lds0 + LDS_S + (c & 1) * STAGE;
#pragma unroll
            for (int i = 0; i < 32; ++i) glds16(sb + i * 8 * NL, (i & 2) ? wv1 : wv0, ld + i * 1024);
            wait_vm<0>();
            lds_set(w_land, c + 1);
        }
        return;
    }
    if (wave >= 4) {
        const int d = wave - 4;
        unsigned cv = 0;
        draw(cv, cnt, 0u, 1u);
        const unsigned eo = ENC_LO + (unsigned)(d * NGEMM + g), eu = unit_at(eo) % NU;
        const bool enc = eo < (unsigned)ENC_HI;
        unsigned xd[16];
        const u32x4 te = p.tab[lane & 15];
        if (enc) {
            const unsigned char* xs = xp + (size_t)((eu % NB16) * 16 + (lane & 15)) * KD + (eu / NB16) * CHK + (lane >> 4) * 16;
#pragma unroll
            for (int q = 0; q < 4; ++q) {
                const u32x4 v = *(const u32x4*)(xs + q * 64);
                xd[4 * q] = v[0]; xd[4 * q + 1] = v[1]; xd[4 * q + 2] = v[2]; xd[4 * q + 3] = v[3];
            }
        }
        const unsigned old = take<0>(cv), call = old + 1;
        if (lane == 0) __hip_atomic_fetch_add((LAS unsigned*)w_got, 1u, __ATOMIC_RELAXED, __HIP_MEMORY_SCOPE_WORKGROUP);
        if (enc) {
            u32x4 o[4];
            const unsigned m = encode_unit(xd, te, smem + LDS_A + d * 4096, lane, o);
            if (m) {
                unsigned char* eb = myws + WS_ENC + (size_t)eu * 4096;
                store_sys<0>(o[0], eb, sv); store_sys<1024>(o[1], eb, sv);
                store_sys<2048>(o[2], eb, sv); store_sys<3072>(o[3], eb, sv);
            }
            STAMP(20);
            wait_vm<0>();
            put_word(myws + WS_META + (size_t)eo * MSTR, 0u, (call << 12) | m);
            STAMP(21);
        }
        while (__builtin_amdgcn_readfirstlane(lds_get(w_tab)) < 4u) __builtin_amdgcn_s_sleep(2);
        unsigned* tmo = (unsigned*)(myws + WS_TMO);
        const unsigned par = (old & 1) * NR;
        const unsigned lv = (lane & 15) * KD + (lane >> 4) * 16;
        const unsigned nitem = S * NCH;
        unsigned dead = 0;
        unsigned o_unit, o_flag;
        {
            unsigned src, b;
            block_of(lane < S ? lane : 0, src, b);
            const unsigned s = par + (rank + src) % NR;
            o_unit = src ? (unsigned)WS_SLOT + s * (unsigned)SLOT_BYTES + b * 4096 : b * 16 * KD;
            o_flag = (unsigned)WS_FLAG + (s * NB16 + b) * (NCH * FS);
        }
#pragma unroll 1
        for (unsigned i = d; i < nitem;) {
            const unsigned sc = S == SL ? i / (2 * SL) : i / (2 * SL - 2), r = i - sc * 2 * S;
            const unsigned k = r >> 1, c = 2 * sc + (r & 1), sl = i % ARING;
            const bool own = k < (unsigned)nloc;
            unsigned info = INFO_RAW;
            while (__builtin_amdgcn_ballot_w64(lds_get(w_used + (lane & 3) * 4) + ARING <= i) != 0) __builtin_amdgcn_s_sleep(2);
            if (!own && !dead) {
                const u64* fp = (const u64*)(myws + __builtin_amdgcn_readlane(o_flag, k) + c * FS);
                const u64 t0 = __builtin_amdgcn_s_memrealtime();
                unsigned it = 0;
                while (true) {
                    const u64 fv = __hip_atomic_load(fp, __ATOMIC_RELAXED, __HIP_MEMORY_SCOPE_SYSTEM);
                    info = __builtin_amdgcn_readfirstlane((unsigned)fv);
                    if ((int)(__builtin_amdgcn_readfirstlane((unsigned)(fv >> 32)) - call) >= 0) break;
                    if ((++it & 255) == 0) {
                        const unsigned tv = __builtin_amdgcn_readfirstlane(
                            __hip_atomic_load(tmo, __ATOMIC_RELAXED, __HIP_MEMORY_SCOPE_AGENT));
                        if (tv == call || __builtin_amdgcn_s_memrealtime() - t0 > DEADLINE_TICKS) {
                            __hip_atomic_store(tmo, call, __ATOMIC_RELAXED, __HIP_MEMORY_SCOPE_AGENT);
                            dead = 0x80000000u;
                            break;
                        }
                    }
                    __builtin_amdgcn_s_sleep(0);
                }
                asm volatile("" ::: "memory");
            }
            STAMP(32 + c);
            if (!dead) {
                const unsigned ou = __builtin_amdgcn_readlane(o_unit, k);
                const unsigned char* sb = own ? xp + ou + c * CHK : myws + ou + c * (NB16 * 4096);
                const unsigned vo = own ? lv : sv, st = own ? 64 : 1024;
#pragma unroll
                for (int q = 0; q < 4; ++q) glds16_sys(sb + q * st, vo, lds0 + LDS_A + sl * 4096 + q * 1024);
            }
            if (c >= NCH - 2 && __builtin_amdgcn_readfirstlane(
                    __hip_atomic_load(tmo, __ATOMIC_RELAXED, __HIP_MEMORY_SCOPE_AGENT)) == call) dead = 0x80000000u;
            wait_vm<0>();
            STAMP(48 + c);
            if (c + WARM < NCH && !own)
                glds4_sys(myws + __builtin_amdgcn_readlane(o_unit, k) + (c + WARM) * (NB16 * 4096), cb * 1024 + (lane & 7) * 128,
                          lds0 + LDS_P + d * 256);
            if (info <= 13u && !dead) {
                unsigned char* unit = smem + LDS_A + sl * 4096;
                unsigned v[16];
                decode_unit(unit, smem + LDS_X + d * 1600, lane, info, v);
#pragma unroll
                for (int q = 0; q < 4; ++q)
                    *(u32x4*)(unit + q * 1024 + sv) = u32x4{v[4 * q], v[4 * q + 1], v[4 * q + 2], v[4 * q + 3]};
            }
            lds_set(w_item + sl * 4, (i + 1) | dead);
            STAMP(2 + c);
            unsigned nx = 0;
            if (lane == 0) nx = __hip_atomic_fetch_add((LAS unsigned*)w_next, 1u, __ATOMIC_RELAXED, __HIP_MEMORY_SCOPE_WORKGROUP);
            i = NLOAD + __builtin_amdgcn_readfirstlane(nx);
        }
        STAMP_OUT();
        return;
    }

    {
        unsigned lo, sel, two;
        symbol_entries(threadIdx.x, lo, sel, two);
        *(LAS u32x2*)(smem + LDS_T + threadIdx.x * 8) = u32x2{lo, sel};
        *(LAS unsigned*)(smem + LDS_T + 2048 + threadIdx.x * 4) = two;
    }
    wait_lds();
    if (lane == 0) __hip_atomic_fetch_add((LAS unsigned*)w_tab, 1u, __ATOMIC_RELAXED, __HIP_MEMORY_SCOPE_WORKGROUP);
    __builtin_amdgcn_s_setprio(2);
    const float scale = p.sx[0] * p.sw[0];
    const __amdgpu_buffer_rsrc_t orsrc = __builtin_amdgcn_make_buffer_rsrc((void*)p.out, 0, NR * MS * NL * 4, 0x00020000);
    const int G = lane >> 4, i16 = lane & 15;
    const int ldsb = ((16 * G + (i16 >> 1)) * 128 + (((i16 >> 2) & 3) | ((G & 1) << 2)) * 16 + 8 * (i16 & 1)) ^ (wave << 5);
    const int ocol = ((i16 & 7) * NL + cb * 128 + wave * 32 + (i16 >> 3) * 16 + G * 4) * 4;
    unsigned ctl = 0;
    auto wait_ctl = [&](int l, unsigned v) {
        unsigned x = __builtin_amdgcn_readlane(ctl, l);
        while ((x & 0x7fffffffu) <= v) {
            ctl = lds_get(w_item + (lane & 31) * 4);
            x = __builtin_amdgcn_readlane(ctl, l);
            if ((x & 0x7fffffffu) <= v) __builtin_amdgcn_s_sleep(1);
        }
        return x;
    };
    i32x8 wf[2][2], wl[2][2];
    auto read_w = [&](i32x8 (&z)[2][2], int slot) {
#pragma unroll
        for (int ks = 0; ks < 2; ++ks)
#pragma unroll
            for (int n = 0; n < 2; ++n) {
                const unsigned char* lp = smem + LDS_S + slot * STAGE + ks * 16384 + (ldsb ^ (n << 4));
                i32x2 t0 = __builtin_amdgcn_ds_read_tr8_b64_v2i32((LAS i32x2*)(lp));
                i32x2 t1 = __builtin_amdgcn_ds_read_tr8_b64_v2i32((LAS i32x2*)(lp + 1024));
                i32x2 t2 = __builtin_amdgcn_ds_read_tr8_b64_v2i32((LAS i32x2*)(lp + 8192));
                i32x2 t3 = __builtin_amdgcn_ds_read_tr8_b64_v2i32((LAS i32x2*)(lp + 8192 + 1024));
                z[ks][n] = i32x8{t0[0], t0[1], t1[0], t1[1], t2[0], t2[1], t3[0], t3[1]};
            }
    };
    f32x4 acc[SL][2];
#pragma unroll
    for (int k = 0; k < SL; ++k) acc[k][0] = acc[k][1] = f32x4{0.f, 0.f, 0.f, 0.f};
    asm volatile("" :: "v"(scale) : "memory");
    STAMP(1);
    unsigned i = 0, as = 0, bad = 0;
    unsigned rowoff;
    {
        unsigned src, b;
        block_of(lane < S ? lane : 0, src, b);
        rowoff = (((rank + src) % NR) * MS + b * 16) * (NL * 4);
        asm volatile("" : "+v"(rowoff));
        const unsigned z = __builtin_amdgcn_raw_buffer_load_b32(orsrc, rowoff + cb * 512, 0, 17);
        asm volatile("" :: "v"(z));
    }
    u32x4 a[2][4];
    auto read_a = [&](u32x4 (&z)[4], unsigned v) {
        bad |= v >> 31;
        const unsigned char* unit = smem + LDS_A + as * 4096 + sv;
#pragma unroll
        for (int q = 0; q < 4; ++q) z[q] = *(const u32x4*)(unit + q * 1024);
        ++i; as = as + 1 == ARING ? 0 : as + 1;
    };
    auto mul = [&](f32x4 (&ac)[2], const i32x8 (&w)[2][2], const u32x4 (&z)[4]) {
        const i32x8 xa0 = {(int)z[0][0], (int)z[0][1], (int)z[0][2], (int)z[0][3], (int)z[1][0], (int)z[1][1], (int)z[1][2], (int)z[1][3]};
        const i32x8 xa1 = {(int)z[2][0], (int)z[2][1], (int)z[2][2], (int)z[2][3], (int)z[3][0], (int)z[3][1], (int)z[3][2], (int)z[3][3]};
        ac[0] = __builtin_amdgcn_mfma_scale_f32_16x16x128_f8f6f4(w[0][0], xa0, ac[0], 0, 0, 0, 0, 0, 0);
        ac[1] = __builtin_amdgcn_mfma_scale_f32_16x16x128_f8f6f4(w[0][1], xa0, ac[1], 0, 0, 0, 0, 0, 0);
        ac[0] = __builtin_amdgcn_mfma_scale_f32_16x16x128_f8f6f4(w[1][0], xa1, ac[0], 0, 0, 0, 0, 0, 0);
        ac[1] = __builtin_amdgcn_mfma_scale_f32_16x16x128_f8f6f4(w[1][1], xa1, ac[1], 0, 0, 0, 0, 0, 0);
    };
#pragma unroll 1
    for (int c = 0; c < NCH; c += 2) {
        wait_ctl(24, c + 1);
        read_w(wf, 0);
        read_w(wl, 1);
        lds_set(w_read + wave * 4, c + 2);
#pragma unroll
        for (int k = 0; k < SL; ++k) {
            if (k < SL - 1 || S == SL) {
                read_a(a[0], wait_ctl(as, i));
                read_a(a[1], wait_ctl(as, i));
                if (c == NCH - 2) STAMP(32 + k);
                __builtin_amdgcn_sched_barrier(0);
                mul(acc[k], wf, a[0]);
                mul(acc[k], wl, a[1]);
                __builtin_amdgcn_sched_barrier(0);
                lds_set(w_used + wave * 4, i);
                if (c == NCH - 2) {
                    int o1 = ocol + __builtin_amdgcn_readlane(rowoff, k), o2 = o1 + 8 * NL * 4;
                    asm volatile("" : "+v"(o1), "+v"(o2));
                    const float sc = bad ? __builtin_nanf("") : scale;
                    const u32x4 lo = __builtin_bit_cast(u32x4, acc[k][0] * sc), hi = __builtin_bit_cast(u32x4, acc[k][1] * sc);
                    u32x4 x, y;
#pragma unroll
                    for (int e = 0; e < 4; ++e) {
                        x[e] = __builtin_amdgcn_update_dpp(lo[e], hi[e], 0x128, 0xf, 0xc, false);
                        y[e] = __builtin_amdgcn_update_dpp(hi[e], lo[e], 0x128, 0xf, 0x3, false);
                    }
                    __builtin_amdgcn_raw_buffer_store_b128(x, orsrc, o1, 0, 17);
                    __builtin_amdgcn_raw_buffer_store_b128(y, orsrc, o2, 0, 17);
                }
            }
        }
        STAMP(2 + c); STAMP(3 + c);
    }
}

extern "C" void kernel_launch(int rank, int nranks, ncclComm_t comm,
                              hipStream_t stream,
                              void* const* d_in, const int* in_sizes, int n_in,
                              void* d_out, int out_size,
                              const int* in_dtypes, int out_dtype,
                              void* const* d_out_all,
                              void* const* d_ws_all, size_t ws_bytes) {
    (void)comm; (void)in_dtypes; (void)out_dtype; (void)d_out_all;
    if (nranks != NR || (unsigned)rank >= (unsigned)NR || n_in != 4 || in_sizes[0] != MS * KD ||
        in_sizes[1] != KD * NL || out_size != NR * MS * NL || ws_bytes < WS_NEED) {
        fprintf(stderr, "unexpected problem shape\n");
        return;
    }
    Params p;
    for (int j = 0; j < NR; ++j) p.ws[j] = (unsigned char*)d_ws_all[(rank + j) % NR];
    p.x = (const unsigned char*)d_in[0];
    p.w = (const unsigned char*)d_in[1];
    p.sx = (const float*)d_in[2];
    p.sw = (const float*)d_in[3];
    p.out = (float*)d_out;
    p.rank = rank;
    p.pad = 0;
    for (int e0 = 0; e0 < 16; ++e0) {
        unsigned t[4];
        encode_table(e0 < 13 ? e0 : 13, t);
        p.tab[e0] = u32x4{t[0], t[1], t[2], t[3]};
    }
    HIP_CHECK(hipFuncSetAttribute((const void*)ag_gemm, hipFuncAttributeMaxDynamicSharedMemorySize, LDS_BYTES));
    hipLaunchKernelGGL(ag_gemm, dim3(GRIDX), dim3(THREADS), LDS_BYTES, stream, p);
    HIP_CHECK(hipGetLastError());
}
```

```cpp
#include <hip/hip_runtime.h>
#include <rccl/rccl.h>
#include <cstdio>

#define HIP_CHECK(expr) do { hipError_t _e = (expr); if (_e != hipSuccess) { \
    fprintf(stderr, "HIP error %s at %s:%d\n", hipGetErrorString(_e), __FILE__, __LINE__); \
    return; } } while (0)

typedef unsigned u32x4 __attribute__((ext_vector_type(4)));
typedef unsigned u32x2 __attribute__((ext_vector_type(2)));
typedef int i32x2 __attribute__((ext_vector_type(2)));
typedef int i32x8 __attribute__((ext_vector_type(8)));
typedef float f32x4 __attribute__((ext_vector_type(4)));
typedef unsigned long long u64;
#define LAS __attribute__((address_space(3)))

constexpr int WIN = 6;
constexpr int DEPTH = 5;
constexpr int NRAW = 4;
constexpr int ENC_HI = 480;
constexpr int RUSH = 112;
constexpr int WARM = 12;

constexpr int NR = 8;
constexpr int KD = 4096;
constexpr int MS = 512;
constexpr int NL = 512;
constexpr int NB16 = MS / 16;
constexpr int CHK = 256;
constexpr int NCH = KD / CHK;
constexpr int NU = NB16 * NCH;
constexpr int PPP = 8;
constexpr int NFEED = 3;
constexpr int NLOAD = 7;
constexpr int NWAVE = 4 + NLOAD + 1;
constexpr int RING = 16;
constexpr int GEMM0 = 7 * PPP;
constexpr unsigned NPOS0 = NRAW + NWAVE - (NFEED + 2);
constexpr unsigned NFIX = PPP * NPOS0;
constexpr unsigned ENC_LO = NFIX;
constexpr int NRW = 50;
constexpr int SL = 6;
constexpr int NGEMM = 4 * NRW;
constexpr int GRIDX = GEMM0 + NGEMM;
constexpr int THREADS = 64 * NWAVE;
constexpr int STAGE = CHK * 128;
constexpr int ARING = 19;
constexpr int LDS_T = 0;
constexpr int LDS_X = 3072;
constexpr int LDS_S = 16384;
constexpr int LDS_A = LDS_S + 2 * STAGE;
constexpr int LDS_W = LDS_A + ARING * 4096 + 64;
constexpr int LDS_P = LDS_W + 256;
constexpr int LDS_BYTES = LDS_P + 8 * 256;
static_assert(GRIDX <= 256 && LDS_BYTES <= 163840 && ENC_HI - ENC_LO <= 4 * NGEMM, "resident grid");
static_assert(DEPTH >= 1 && DEPTH <= RING && NRAW <= NFEED + 1 && NPOS0 > NFEED && NPOS0 <= RING, "push geometry");
static_assert(LDS_X + NLOAD * 1600 <= LDS_S && ARING <= 24 && ARING >= 2 * SL && RING * 4096 + 7168 <= LDS_BYTES, "LDS map");

__device__ __forceinline__ unsigned unit_at(unsigned q) {
    return (q & ~63u) + (q & 1u) * NB16 + ((q & 63u) >> 1);
}

constexpr int FS = 32, MSTR = 32;
constexpr size_t WS_FLAG = 1536 << 10;
constexpr size_t WS_TMO = 64 << 10;
constexpr size_t WS_IDLE = 0;
constexpr size_t WS_TICKET = 96 << 10;
constexpr size_t WS_CNT = 128 << 10;
constexpr size_t WS_META = 256 << 10;
constexpr size_t SLOT_BYTES = (size_t)MS * KD;
constexpr size_t WS_ENC = 2 << 20;
constexpr size_t WS_SLOT = 4 << 20;
constexpr size_t WS_NEED = WS_SLOT + 2 * NR * SLOT_BYTES;
constexpr unsigned INFO_RAW = 15u;
constexpr u64 DEADLINE_TICKS = 100000000ull;

struct Params {
    unsigned char* ws[NR];
    const unsigned char* x;
    const unsigned char* w;
    const float* sx;
    const float* sw;
    float* out;
    int rank;
    int pad;
    u32x4 tab[16];
};

template <int N> __device__ __forceinline__ void wait_vm() {
    asm volatile("s_waitcnt vmcnt(%0)" :: "i"(N) : "memory");
}
__device__ __forceinline__ void wait_lds() { asm volatile("s_waitcnt lgkmcnt(0)" ::: "memory"); }
__device__ __forceinline__ void glds16(const unsigned char* sbase, unsigned voff, unsigned lds) {
    unsigned keep;
    asm volatile("s_mov_b32 %0, m0\n\ts_mov_b32 m0, %3\n\ts_nop 0\n\tglobal_load_lds_dwordx4 %1, %2\n\ts_mov_b32 m0, %0"
                 : "=&s"(keep) : "v"(voff), "s"(sbase), "s"(lds) : "memory");
}
__device__ __forceinline__ void glds16_sys(const unsigned char* sbase, unsigned voff, unsigned lds) {
    unsigned keep;
    asm volatile("s_mov_b32 %0, m0\n\ts_mov_b32 m0, %3\n\ts_nop 0\n\tglobal_load_lds_dwordx4 %1, %2 sc0 sc1\n\ts_mov_b32 m0, %0"
                 : "=&s"(keep) : "v"(voff), "s"(sbase), "s"(lds) : "memory");
}
__device__ __forceinline__ void glds4_sys(const unsigned char* sbase, unsigned voff, unsigned lds) {
    unsigned keep;
    asm volatile("s_mov_b32 %0, m0\n\ts_mov_b32 m0, %3\n\ts_nop 0\n\tglobal_load_lds_dword %1, %2 sc0 sc1\n\ts_mov_b32 m0, %0"
                 : "=&s"(keep) : "v"(voff), "s"(sbase), "s"(lds) : "memory");
}
template <int OFF> __device__ __forceinline__ void store_sys(const u32x4& d, unsigned char* sbase, unsigned voff) {
    asm volatile("global_store_dwordx4 %1, %0, %2 offset:%3 sc0 sc1\n\ts_nop 1" :: "v"(d), "v"(voff), "s"(sbase), "i"(OFF) : "memory");
}
template <int OFF> __device__ __forceinline__ void store_sys_short(const u32x4& d, unsigned char* sbase, unsigned voff, unsigned drop) {
    unsigned t;
    asm volatile("s_nop 0\n\tv_readfirstlane_b32 %0, %5\n\ts_lshr_b64 exec, -1, %0\n\t"
                 "global_store_dwordx4 %2, %1, %3 offset:%4 sc0 sc1\n\ts_mov_b64 exec, -1\n\ts_nop 0"
                 : "=&s"(t) : "v"(d), "v"(voff), "s"(sbase), "i"(OFF), "v"(drop) : "memory", "scc");
}
__device__ __forceinline__ void put_word(unsigned char* sbase, unsigned zero, unsigned val) {
    asm volatile("s_mov_b64 exec, 1\n\tglobal_store_dword %0, %1, %2 sc0 sc1\n\ts_mov_b64 exec, -1\n\ts_nop 0"
                 :: "v"(zero), "v"(val), "s"(sbase) : "memory");
}
__device__ __forceinline__ void put_flag(unsigned char* sbase, unsigned voff, unsigned lo, unsigned hi) {
    const u64 v = ((u64)hi << 32) | lo;
    asm volatile("s_mov_b64 exec, %3\n\tglobal_store_dwordx2 %0, %1, %2 sc0 sc1\n\ts_mov_b64 exec, -1\n\ts_nop 0"
                 :: "v"(voff), "v"(v), "s"(sbase), "i"((1 << (FS / 8)) - 1) : "memory");
}
__device__ __forceinline__ void draw(unsigned& t, unsigned* ctr, unsigned zero, unsigned one) {
    asm volatile("s_mov_b64 exec, 1\n\tglobal_atomic_add %0, %1, %2, %3 sc0\n\ts_mov_b64 exec, -1"
                 : "+v"(t) : "v"(zero), "v"(one), "s"(ctr) : "memory");
}
template <int N> __device__ __forceinline__ unsigned take(unsigned& t) {
    asm volatile("s_waitcnt vmcnt(%1) ; take" : "+v"(t) : "i"(N) : "memory");
    return __builtin_amdgcn_readfirstlane(t);
}
__device__ __forceinline__ unsigned lds_get(unsigned char* a) {
    const unsigned v = __hip_atomic_load((LAS unsigned*)a, __ATOMIC_RELAXED, __HIP_MEMORY_SCOPE_WORKGROUP);
    asm volatile("" ::: "memory");
    return v;
}
__device__ __forceinline__ void lds_set(unsigned char* a, unsigned v) {
    wait_lds();
    __hip_atomic_store((LAS unsigned*)a, v, __ATOMIC_RELAXED, __HIP_MEMORY_SCOPE_WORKGROUP);
}

static_assert(WIN >= 5 && WIN <= 8, "window");
constexpr int FJ = (3 + WIN) % 4;
constexpr int window_n(int j) {
    int c = WIN - 1;
    for (int q = j - WIN + 1; q < j; ++q) c += (q + 8) % 4 == FJ;
    return c;
}

__device__ __forceinline__ unsigned scan_incl(unsigned v) {
    v += __builtin_amdgcn_update_dpp(0u, v, 0x111, 0xf, 0xf, false);
    v += __builtin_amdgcn_update_dpp(0u, v, 0x112, 0xf, 0xf, false);
    v += __builtin_amdgcn_update_dpp(0u, v, 0x114, 0xf, 0xf, false);
    v += __builtin_amdgcn_update_dpp(0u, v, 0x118, 0xf, 0xf, false);
    v += __builtin_amdgcn_update_dpp(0u, v, 0x142, 0xa, 0xf, false);
    v += __builtin_amdgcn_update_dpp(0u, v, 0x143, 0xc, 0xf, false);
    return v;
}
__device__ __forceinline__ void symbol_entries(unsigned x, unsigned& lo, unsigned& sel, unsigned& two) {
    lo = sel = two = 0;
    unsigned r = 4;
#pragma unroll
    for (unsigned i = 0; i < 4; ++i) {
        const unsigned y = (x >> (2 * i)) & 3u;
        lo |= ((0x40182000u >> (8 * y)) & 0xffu) << (8 * i);
        sel |= (y ? i : r) << (8 * i);
        r += y == 0;
        two |= ((0x48000810u >> (8 * y)) & 0xffu) << (8 * i);
    }
}
__host__ __device__ inline void encode_table(int e0, unsigned* t) {
    unsigned a[16];
#pragma unroll
    for (int e = 0; e < 16; ++e) {
        const int q = e - e0;
        a[e] = (q >= 0 && q <= 2) ? (unsigned)((q + 1) << 3) : (q == -1 || q == -2) ? (unsigned)(4 - q) : q == 3 ? 7u
             : (q <= -3 && q >= -5) ? (unsigned)(0x84 | ((-3 - q) << 5)) : q == 4 ? 0xe4u : 0x0cu;
    }
    auto sg = [](unsigned v) { return (v & 0x80u) ? 0xffu : 0u; };
#pragma unroll
    for (int i = 5; i < 8; ++i) { a[i] ^= 0xffu; a[8 + i] ^= 0xffu; }
    a[3] ^= sg(a[15]); a[2] ^= sg(a[13]); a[11] ^= sg(a[7]); a[10] ^= sg(a[5]);
    a[1] ^= sg(a[11]); a[9] ^= sg(a[3]);
    a[0] ^= sg(a[9]); a[8] ^= sg(a[1]);
#pragma unroll
    for (int i = 0; i < 4; ++i) t[i] = a[4 * i] | (a[4 * i + 1] << 8) | (a[4 * i + 2] << 16) | (a[4 * i + 3] << 24);
}
__device__ __forceinline__ unsigned encode_unit(const unsigned (&d)[16], const u32x4& te, unsigned char* scr,
                                                int lane, u32x4 (&o)[4]) {
    *(u32x4*)(scr + lane * 16) = u32x4{0u, 0u, 0u, 0u};
    unsigned es = 0;
#pragma unroll
    for (int i = 0; i < 16; i += 5) es = __builtin_amdgcn_sad_u8(d[i] & 0x78787878u, 0u, es);
    int e0 = ((int)__builtin_amdgcn_readlane(scan_incl(es), 63) - 1638) >> 13;
    e0 = e0 < 0 ? 0 : (e0 > 13 ? 13 : e0);
    const unsigned t0 = __builtin_amdgcn_readlane(te[0], e0), t1 = __builtin_amdgcn_readlane(te[1], e0);
    const unsigned t2 = __builtin_amdgcn_readlane(te[2], e0), t3 = __builtin_amdgcn_readlane(te[3], e0);
    unsigned c[16], bad = 0;
    unsigned s1[4], n1[4], s2[4], n2[4];
#pragma unroll
    for (int h = 0; h < 4; ++h) {
        unsigned v[4], n[4], w[4], k2[4];
#pragma unroll
        for (int k = 0; k < 4; ++k) {
            const unsigned x = d[4 * h + k];
            const unsigned e = (x >> 3) & 0x0f0f0f0fu;
            const unsigned inf = __builtin_amdgcn_perm(t1, t0, e) ^ __builtin_amdgcn_perm(t3, t2, e ^ 0x08080808u);
            c[4 * h + k] = (x & 0x87878787u) | (inf & 0x18181818u);
            bad |= inf & (inf >> 1);
            const unsigned f1 = inf & 0x04040404u, f2 = inf & 0x80808080u;
            v[k] = __builtin_amdgcn_udot4(inf & 0x03030303u, __builtin_amdgcn_perm(0u, 0x40100401u, f1 * 0x404040u), 0u, false);
            w[k] = __builtin_amdgcn_udot4((inf >> 5) & 0x03030303u, __builtin_amdgcn_perm(0u, 0x40100401u, f2 * 0x20202u), 0u, false);
            n[k] = __builtin_popcount(f1) + (k ? n[k - 1] : 0u);
            k2[k] = __builtin_popcount(f2) + (k ? k2[k - 1] : 0u);
        }
        s1[h] = v[0] | (v[1] << (2 * n[0])) | (v[2] << (2 * n[1])) | (v[3] << (2 * n[2]));
        s2[h] = w[0] | (w[1] << (2 * k2[0])) | (w[2] << (2 * k2[1])) | (w[3] << (2 * k2[2]));
        n1[h] = 2 * n[3]; n2[h] = 2 * k2[3];
    }
#pragma unroll
    for (int q = 0; q < 4; ++q) {
        const unsigned z = c[4 * q + 3];
        const unsigned p0 = c[4 * q] | ((z << 5) & 0x60606060u);
        const unsigned p1 = c[4 * q + 1] | ((z << 3) & 0x60606060u);
        const unsigned p2 = c[4 * q + 2] | ((z << 1) & 0x20202020u) | ((z >> 1) & 0x40404040u);
        o[(3 * q) >> 2][(3 * q) & 3] = p0; o[(3 * q + 1) >> 2][(3 * q + 1) & 3] = p1; o[(3 * q + 2) >> 2][(3 * q + 2) & 3] = p2;
    }
    const unsigned m1 = n1[0] + n1[1], m2 = n2[0] + n2[1] + n2[2] + n2[3];
    const unsigned tot = (m1 + n1[2] + n1[3]) | (m2 << 16);
    const unsigned inc = scan_incl(tot), sum = __builtin_amdgcn_readlane(inc, 63);
    const unsigned at2 = 16u + (sum & 0xffffu), bits = at2 + (sum >> 16);
    if (__builtin_amdgcn_ballot_w64((bad & 0x04040404u) || m2 > 64u) != 0 || at2 > 16u + 64u * 48u || bits > 8192u - 128u) return 0u;
    auto put = [&](u64 str, unsigned ob) {
        unsigned* wp = (unsigned*)(scr + (ob >> 5) * 4);
        const u64 a = (u64)(unsigned)str << (ob & 31u), b = (u64)(unsigned)(str >> 32) << (ob & 31u);
        __hip_atomic_fetch_or(wp, (unsigned)a, __ATOMIC_RELAXED, __HIP_MEMORY_SCOPE_WORKGROUP);
        __hip_atomic_fetch_or(wp + 1, (unsigned)(a >> 32) | (unsigned)b, __ATOMIC_RELAXED, __HIP_MEMORY_SCOPE_WORKGROUP);
        __hip_atomic_fetch_or(wp + 2, (unsigned)(b >> 32), __ATOMIC_RELAXED, __HIP_MEMORY_SCOPE_WORKGROUP);
    };
    const unsigned ex = inc - tot, o1 = 16u + (ex & 0xffffu);
    put(s1[0] | ((u64)s1[1] << n1[0]), o1);
    put(s1[2] | ((u64)s1[3] << n1[2]), o1 + m1);
    put(s2[0] | ((u64)s2[1] << n2[0]) | ((u64)s2[2] << ((n2[0] + n2[1]) & 63u)) | ((u64)s2[3] << ((m2 - n2[3]) & 63u)), at2 + (ex >> 16));
    if (lane == 0) __hip_atomic_fetch_or((unsigned*)scr, at2, __ATOMIC_RELAXED, __HIP_MEMORY_SCOPE_WORKGROUP);
    asm volatile("" ::: "memory");
    {
        const unsigned g0 = 16u + lane * 48u;
        const unsigned* gp = (const unsigned*)(scr + ((g0 >> 3) & ~3u));
        u64 g = (gp[0] | ((u64)gp[1] << 32)) >> (g0 & 16u);
        const int left = (int)at2 - (int)g0;
        g |= ~0ull << (left <= 0 ? 0 : (left < 48 ? left : 48));
        if (__builtin_amdgcn_ballot_w64(__builtin_popcountll(~(g | (g >> 1)) & 0x5555555555555555ull) > 15) != 0) return 0u;
    }
    asm volatile("s_waitcnt lgkmcnt(0)" ::: "memory");
    o[3] = *(const u32x4*)(scr + lane * 16);
    const unsigned nl = ((bits + 255u) >> 8) * 2u;
    return (nl << 4) | (unsigned)e0;
}
__device__ __forceinline__ void decode_unit(const unsigned char* slot, unsigned char* ex, int lane, unsigned e0, unsigned (&d)[16]) {
    const unsigned g0 = 2u + lane * 6u;
    const unsigned* gp = (const unsigned*)(slot + 3072 + (g0 & ~3u));
    const unsigned ga = gp[0], gb = gp[1], at2 = *(const unsigned*)(slot + 3072) & 0xffffu;
    unsigned p[12];
#pragma unroll
    for (int q = 0; q < 3; ++q) {
        const u32x4 v = *(const u32x4*)(slot + q * 1024 + lane * 16);
        p[4 * q] = v[0]; p[4 * q + 1] = v[1]; p[4 * q + 2] = v[2]; p[4 * q + 3] = v[3];
    }
    const u64 g = (ga | ((u64)gb << 32)) >> ((g0 & 2u) * 8);
    const unsigned gl = (unsigned)g, gh = (unsigned)(g >> 32);
    u32x2 ta[6];
#pragma unroll
    for (int i = 0; i < 6; ++i) {
        const unsigned w = i < 4 ? gl : gh, sh = 8 * (i & 3);
        ta[i] = *(const LAS u32x2*)(size_t)((sh ? w >> (sh - 3) : w << 3) & 0x7f8u);
    }
    const unsigned zl = (~(gl | (gl >> 1)) & 0x55555555u) * 3u, zh = (~(gh | (gh >> 1)) & 0x5555u) * 3u;
    unsigned z[7];
    z[0] = 0; z[1] = __builtin_popcount(zl & 0xffu); z[2] = __builtin_popcount(zl & 0xffffu); z[3] = __builtin_popcount(zl & 0xffffffu);
    z[4] = __builtin_popcount(zl); z[5] = __builtin_popcount(zh & 0xffu) + z[4]; z[6] = __builtin_popcount(zh) + z[4];
#pragma unroll
    for (int q = 0; q < 4; ++q) {
        d[4 * q] = p[3 * q] & 0x9f9f9f9fu; d[4 * q + 1] = p[3 * q + 1] & 0x9f9f9f9fu; d[4 * q + 2] = p[3 * q + 2] & 0x9f9f9f9fu;
        d[4 * q + 3] = ((p[3 * q] >> 5) & 0x03030303u) | ((p[3 * q + 1] >> 3) & 0x0c0c0c0cu) |
                       ((p[3 * q + 2] >> 1) & 0x10101010u) | ((p[3 * q + 2] << 1) & 0x80808080u);
    }
    const unsigned o2 = at2 + scan_incl(z[6]) - z[6] - 2u;
    const unsigned* wp = (const unsigned*)(slot + 3072 + ((o2 >> 5) & 255u) * 4);
    const unsigned w0 = wp[0], w1 = wp[1];
    __builtin_amdgcn_sched_barrier(0);
    unsigned f[16], c[17];
    c[0] = 0;
#pragma unroll
    for (int i = 0; i < 16; ++i) {
        f[i] = ~(d[i] | (d[i] >> 1)) & 0x08080808u;
        c[i + 1] = __builtin_popcount(f[i]) + c[i];
    }
    __builtin_amdgcn_sched_barrier(0);
    const unsigned win = __builtin_amdgcn_alignbit(w1, w0, o2);
    unsigned tb[6];
#pragma unroll
    for (int i = 0; i < 6; ++i) tb[i] = *(const LAS unsigned*)(size_t)(2048u + ((win >> (z[i] & 31u)) & 0x3fcu));
    __builtin_amdgcn_sched_barrier(0);
    const unsigned base = (unsigned)(size_t)(LAS unsigned char*)ex + scan_incl(c[16]) - c[16];
    unsigned at[16], al[16];
#pragma unroll
    for (int i = 0; i < 16; ++i) { at[i] = base + c[i]; al[i] = at[i] & ~3u; }
    __builtin_amdgcn_sched_barrier(0);
#pragma unroll
    for (int i = 0; i < 6; i += 2)
        *(u32x2*)(ex + lane * 24 + i * 4) = u32x2{__builtin_amdgcn_perm(tb[i], ta[i][0], ta[i][1]),
                                                  __builtin_amdgcn_perm(tb[i + 1], ta[i + 1][0], ta[i + 1][1])};
    asm volatile("" ::: "memory");
    unsigned ya[16], yb[16];
#pragma unroll
    for (int i = 0; i < 16; ++i) {
        const LAS unsigned* yp = (const LAS unsigned*)(size_t)al[i];
        ya[i] = yp[0]; yb[i] = yp[1];
    }
    __builtin_amdgcn_sched_barrier(0);
#pragma unroll
    for (int i = 0; i < 16; ++i) f[i] = __umul24(f[i], 0x202020u) + (f[i] >> 1);
    __builtin_amdgcn_sched_barrier(0);
    const unsigned bias = (e0 - 5u) * 0x08080808u;
#pragma unroll
    for (int i = 0; i < 16; ++i)
        d[i] = d[i] + __builtin_amdgcn_perm(__builtin_amdgcn_alignbyte(yb[i], ya[i], at[i]), 0x20202020u, f[i]) + bias;
}

__global__ void __launch_bounds__(THREADS, 3) ag_gemm(Params p) {
    extern __shared__ __attribute__((aligned(16))) unsigned char smem[];
    const int lane = threadIdx.x & 63;
    const int wave = __builtin_amdgcn_readfirstlane(threadIdx.x >> 6);
    const unsigned j = 1 + blockIdx.x % (NR - 1);
    unsigned char* const myws = p.ws[0];
    unsigned char* const pws = p.ws[j];
    const unsigned char* const xp = p.x;
    const unsigned rank = p.rank;
    asm volatile("" :: "s"(myws), "s"(pws), "s"(xp), "s"(rank));
    const unsigned lds0 = (unsigned)(size_t)(LAS unsigned char*)smem;
    unsigned* const cnt = (unsigned*)(myws + WS_CNT + (size_t)(blockIdx.x * 16 + wave) * 32);
    const unsigned sv = lane * 16;

    if (__builtin_expect(blockIdx.x < GEMM0, 1)) {
        constexpr unsigned POS_RUSH = 1u << 29, POS_UNIT = 1u << 30, POS_END = 1u << 31;
        const unsigned pk = blockIdx.x / (NR - 1);
        const unsigned lv = (lane & 15) * KD + (lane >> 4) * 16;
        unsigned char* const w_pos = smem + RING * 4096;
        unsigned char* const w_meta = w_pos + 4096;
        unsigned char* const w_cons = w_meta + NU * 4;
        unsigned char* const w_done = w_cons + 4;
        unsigned* const tick = (unsigned*)(myws + WS_TICKET + j * 128);
        const bool feeder = wave >= 1 && wave <= NFEED, packer = wave >= NFEED + 2;
        unsigned* const ctr = feeder ? tick : cnt;
        const unsigned n0 = packer ? NRAW + wave - (NFEED + 2) : wave;
        unsigned u = unit_at(pk + PPP * n0);
        unsigned cv = 0;
        if (!packer) draw(cv, ctr, 0u, 1u);
        if (wave < NRAW || packer) {
            const unsigned char* sb = xp + (size_t)(u % NB16) * 16 * KD + (u / NB16) * CHK;
#pragma unroll
            for (int q = 0; q < 4; ++q) glds16(sb + q * 64, lv, lds0 + n0 * 4096 + q * 1024);
        } else if (wave > NFEED) {
#pragma unroll
            for (int q = 0; q < 7; ++q) *(LAS u32x4*)(w_pos + q * 1024 + sv) = u32x4{0u, 0u, 0u, 0u};
            wait_lds();
        }
        u32x4 te = {0u, 0u, 0u, 0u};
        if (packer) te = p.tab[lane & 15];
        const u64 tstart = __builtin_amdgcn_s_memrealtime();
        __builtin_amdgcn_s_barrier();
        unsigned t = take<0>(cv);
        if (j >= (unsigned)NR) return;
        if (__builtin_expect(packer, 0)) {
            unsigned xd[16];
            unsigned char* slot = smem + n0 * 4096;
#pragma unroll
            for (int q = 0; q < 4; ++q) {
                const u32x4 v = *(const u32x4*)(slot + q * 1024 + sv);
                xd[4 * q] = v[0]; xd[4 * q + 1] = v[1]; xd[4 * q + 2] = v[2]; xd[4 * q + 3] = v[3];
            }
            u32x4 o[4];
            const unsigned m = encode_unit(xd, te, slot + 3072, lane, o);
            if (m) { *(u32x4*)(slot + sv) = o[0]; *(u32x4*)(slot + 1024 + sv) = o[1]; *(u32x4*)(slot + 2048 + sv) = o[2]; }
            else *(u32x4*)(slot + 3072 + sv) = u32x4{xd[12], xd[13], xd[14], xd[15]};
            lds_set(w_pos + n0 * 4, POS_UNIT | (u << 12) | (m ? (64u - (m >> 4)) | ((m & 15u) << 6) : INFO_RAW << 6));
            return;
        }
        if (feeder) {
            unsigned n = wave, m = 0, rush = 0;
            while (true) {
                if (n < NRAW || n >= NPOS0)
                    lds_set(w_pos + n * 4, POS_UNIT | rush | (u << 12) |
                                           (m ? (64u - (m >> 4)) | ((m & 15u) << 6) : INFO_RAW << 6));
                n = n < NPOS0 ? NPOS0 + n - 1 : n + NFEED;
                const unsigned o = NFIX + t % NU;
                if (o >= (unsigned)NU) { lds_set(w_pos + n * 4, POS_END); return; }
                while (n >= __builtin_amdgcn_readfirstlane(lds_get(w_cons)) + DEPTH) __builtin_amdgcn_s_sleep(1);
                u = unit_at(o);
                rush = o >= NU - RUSH && n * PPP + 4 < o ? POS_RUSH : 0u;
                m = __builtin_amdgcn_readfirstlane(lds_get(w_meta + o * 4));
                if (m == 0 && o >= ENC_LO && o < ENC_HI) {
                    const unsigned e = __builtin_amdgcn_readfirstlane(__hip_atomic_load(
                        (unsigned*)(myws + WS_META + (size_t)o * MSTR), __ATOMIC_RELAXED, __HIP_MEMORY_SCOPE_SYSTEM));
                    if ((e >> 12) == ((t / NU + 1) & 0xfffffu)) m = e & 0xfffu;
                    asm volatile("" ::: "memory");
                }
                if ((m >> 4) < 1u || (m >> 4) > 64u) m = 0;
                unsigned tv = 0;
                draw(tv, ctr, 0u, 1u);
                const unsigned char* sb = m ? myws + WS_ENC + (size_t)u * 4096
                                            : xp + (size_t)(u % NB16) * 16 * KD + (u / NB16) * CHK;
                const unsigned vo = m ? sv : lv, st = m ? 1024 : 64;
#pragma unroll
                for (int q = 0; q < 4; ++q) glds16_sys(sb + q * st, vo, lds0 + (n % RING) * 4096 + q * 1024);
                t = take<0>(tv);
            }
        }
        const unsigned old = t, call = old + 1;
        if (__builtin_expect(wave == 0, 1)) {
            unsigned char* const dst = pws + WS_SLOT + (size_t)((old & 1) * NR + rank) * SLOT_BYTES;
            unsigned char* const fl = pws + WS_FLAG + (size_t)((old & 1) * NR + rank) * (NU * FS);
            unsigned char* const idle = pws + WS_IDLE + (rank * 64 + blockIdx.x) * 128;
            __builtin_amdgcn_s_setprio(3);
            unsigned n = 0, live = (1u << NFEED) - 1u, dsc = POS_UNIT | (u << 12) | (INFO_RAW << 6);
            unsigned d1 = 0, d2 = 0;
            auto flag = [&](unsigned dd) {
                const unsigned h = __builtin_amdgcn_readfirstlane(dd), uu = (h >> 12) & 0x1ffu;
                put_flag(h ? fl + ((uu % NB16) * NCH + uu / NB16) * FS : idle, lane * 8, (h >> 6) & 15u, call);
            };
            auto look = [&]() { return __builtin_amdgcn_readfirstlane(lds_get(w_pos + n * 4)); };
            auto acquire = [&]() {
                while (live) {
                    if ((live >> (n % NFEED)) & 1u) {
                        dsc = look();
                        for (int k = 0; k < 6 && !dsc; ++k) dsc = look();
                        if (!dsc) {
                            wait_vm<5>(); flag(d2); d2 = 0;
                            dsc = look();
                            if (!dsc) {
                                wait_vm<1>(); flag(d1); d1 = 0;
                                do dsc = look(); while (!dsc);
                            }
                        }
                        if (dsc & POS_UNIT) return true;
                        live &= ~(1u << (n % NFEED));
                    }
                    n += 1;
                    lds_set(w_cons, n);
                }
                return false;
            };
            u32x4 r[4], q[4];
            auto grab = [&](u32x4 (&z)[4]) {
#pragma unroll
                for (int k = 0; k < 4; ++k) z[k] = *(const u32x4*)(smem + (n % RING) * 4096 + k * 1024 + sv);
            };
            auto send = [&](u32x4 (&a)[4], u32x4 (&z)[4]) {
                const unsigned vo = (dsc & 0x1ff000u) | sv, d0 = dsc;
                n += 1;
                unsigned w = lds_get(w_pos + n * 4);
                const bool pace = !(dsc & POS_RUSH);
                if (FJ == 0) { wait_vm<window_n(0)>(); flag(d2); } else if (pace) wait_vm<window_n(0)>();
                store_sys<0>(a[0], dst, vo);
                if (FJ == 1) { wait_vm<window_n(1)>(); flag(d2); } else if (pace) wait_vm<window_n(1)>();
                store_sys<1024>(a[1], dst, vo);
                asm volatile("" : "+v"(w));
                dsc = __builtin_amdgcn_readfirstlane(w);
                bool ok = (dsc & POS_UNIT) != 0;
                if (ok) grab(z);
                if (FJ == 2) { wait_vm<window_n(2)>(); flag(d2); } else if (pace) wait_vm<window_n(2)>();
                store_sys<2048>(a[2], dst, vo);
                if (FJ == 3) { wait_vm<window_n(3)>(); flag(d2); }
                store_sys_short<3072>(a[3], dst, vo, d0);
                d2 = d1; d1 = d0;
                if (!ok) {
                    ok = acquire();
                    if (ok) grab(z);
                }
                lds_set(w_cons, n + 1);
                return ok;
            };
            grab(r); lds_set(w_cons, 1u);
            while (send(r, q) && send(q, r)) {}
            wait_vm<5>(); flag(d2);
            wait_vm<1>(); flag(d1);
            lds_set(w_done, 1u);
            if (lane == 0) __hip_atomic_fetch_add(tick, NPOS0 - NFEED, __ATOMIC_RELAXED, __HIP_MEMORY_SCOPE_AGENT);
            return;
        }
        constexpr int NQ = (ENC_HI - ENC_LO + 63) / 64;
        const __amdgpu_buffer_rsrc_t mr = __builtin_amdgcn_make_buffer_rsrc(myws + WS_META, 0, ENC_HI * MSTR, 0x00020000);
        while (__builtin_amdgcn_s_memrealtime() - tstart < 450) __builtin_amdgcn_s_sleep(4);
        while (true) {
            unsigned e[NQ];
#pragma unroll
            for (int q = 0; q < NQ; ++q) e[q] = __builtin_amdgcn_raw_buffer_load_b32(mr, (ENC_LO + q * 64 + lane) * MSTR, 0, 17);
            bool ok = true;
#pragma unroll
            for (int q = 0; q < NQ; ++q) {
                const unsigned o = ENC_LO + q * 64 + lane;
                if ((e[q] >> 12) == (call & 0xfffffu) && o < (unsigned)ENC_HI) *(LAS unsigned*)(w_meta + o * 4) = (e[q] & 0xfffu) ? (e[q] & 0xfffu) : 1u;
                else if (o < (unsigned)ENC_HI) ok = false;
            }
            if (__builtin_amdgcn_ballot_w64(!ok) == 0 || __builtin_amdgcn_readfirstlane(lds_get(w_done))) break;
            __builtin_amdgcn_s_sleep(16);
        }
        return;
    }

    if (lds0 != 0) return;
    const int g = blockIdx.x - GEMM0;
    const int cb = g & 3, rg = g >> 2;
    static_assert(7 * NB16 == 4 * NRW + 24 && NB16 == 26 + 6, "block deal");
    const int nloc = rg < 24 ? 0 : (rg < 44 ? 1 : 2), S = rg < 44 ? 5 : 6;
    auto block_of = [&](unsigned k, unsigned& src, unsigned& b) {
        const unsigned t = rg + (k - nloc) * NRW;
        const bool own = k < (unsigned)nloc;
        src = own ? 0u : 1u + t % 7u;
        b = own ? (k ? rg - 18 : rg - 24) : t / 7u;
    };
    unsigned char* const w_item = smem + LDS_W;
    unsigned char* const w_land = w_item + 96;
    unsigned char* const w_got = w_item + 104;
    unsigned char* const w_tab = w_item + 108;
    unsigned char* const w_next = w_item + 112;
    unsigned char* const w_used = w_item + 128;
    unsigned char* const w_read = w_item + 144;
    if (threadIdx.x < 64) *(LAS unsigned*)(w_item + threadIdx.x * 4) = 0u;
    wait_lds();
    __builtin_amdgcn_s_barrier();

    if (wave == 4 + NLOAD) {
        const unsigned wv0 = (lane >> 3) * NL + cb * 128 + (((lane & 7) ^ ((lane >> 4) & 3)) << 4);
        const unsigned wv1 = wv0 ^ 64;
        while (__builtin_amdgcn_readfirstlane(lds_get(w_got)) < (unsigned)NLOAD) __builtin_amdgcn_s_sleep(2);
#pragma unroll 1
        for (int c = 0; c < NCH; ++c) {
            while (__builtin_amdgcn_ballot_w64(lds_get(w_read + (lane & 3) * 4) + 1 < (unsigned)c) != 0) __builtin_amdgcn_s_sleep(2);
            const unsigned char* sb = p.w + (size_t)c * (CHK * NL);
            const unsigned ld = lds0 + LDS_S + (c & 1) * STAGE;
#pragma unroll
            for (int i = 0; i < 32; ++i) glds16(sb + i * 8 * NL, (i & 2) ? wv1 : wv0, ld + i * 1024);
            wait_vm<0>();
            lds_set(w_land, c + 1);
        }
        return;
    }
    if (wave >= 4) {
        const int d = wave - 4;
        unsigned cv = 0;
        draw(cv, cnt, 0u, 1u);
        const unsigned eo = ENC_LO + (unsigned)(d * NGEMM + g), eu = unit_at(eo) % NU;
        const bool enc = eo < (unsigned)ENC_HI;
        unsigned xd[16];
        const u32x4 te = p.tab[lane & 15];
        if (enc) {
            const unsigned char* xs = xp + (size_t)((eu % NB16) * 16 + (lane & 15)) * KD + (eu / NB16) * CHK + (lane >> 4) * 16;
#pragma unroll
            for (int q = 0; q < 4; ++q) {
                const u32x4 v = *(const u32x4*)(xs + q * 64);
                xd[4 * q] = v[0]; xd[4 * q + 1] = v[1]; xd[4 * q + 2] = v[2]; xd[4 * q + 3] = v[3];
            }
        }
        const unsigned old = take<0>(cv), call = old + 1;
        if (lane == 0) __hip_atomic_fetch_add((LAS unsigned*)w_got, 1u, __ATOMIC_RELAXED, __HIP_MEMORY_SCOPE_WORKGROUP);
        if (enc) {
            u32x4 o[4];
            const unsigned m = encode_unit(xd, te, smem + LDS_A + d * 4096, lane, o);
            if (m) {
                unsigned char* eb = myws + WS_ENC + (size_t)eu * 4096;
                store_sys<0>(o[0], eb, sv); store_sys<1024>(o[1], eb, sv);
                store_sys<2048>(o[2], eb, sv); store_sys<3072>(o[3], eb, sv);
            }
            wait_vm<0>();
            put_word(myws + WS_META + (size_t)eo * MSTR, 0u, (call << 12) | m);
        }
        while (__builtin_amdgcn_readfirstlane(lds_get(w_tab)) < 4u) __builtin_amdgcn_s_sleep(2);
        unsigned* tmo = (unsigned*)(myws + WS_TMO);
        const unsigned par = (old & 1) * NR;
        const unsigned lv = (lane & 15) * KD + (lane >> 4) * 16;
        const unsigned nitem = S * NCH;
        unsigned dead = 0;
        unsigned o_unit, o_flag;
        {
            unsigned src, b;
            block_of(lane < S ? lane : 0, src, b);
            const unsigned s = par + (rank + src) % NR;
            o_unit = src ? (unsigned)WS_SLOT + s * (unsigned)SLOT_BYTES + b * 4096 : b * 16 * KD;
            o_flag = (unsigned)WS_FLAG + (s * NB16 + b) * (NCH * FS);
        }
#pragma unroll 1
        for (unsigned i = d; i < nitem;) {
            const unsigned sc = S == SL ? i / (2 * SL) : i / (2 * SL - 2), r = i - sc * 2 * S;
            const unsigned k = r >> 1, c = 2 * sc + (r & 1), sl = i % ARING;
            const bool own = k < (unsigned)nloc;
            unsigned info = INFO_RAW;
            while (__builtin_amdgcn_ballot_w64(lds_get(w_used + (lane & 3) * 4) + ARING <= i) != 0) __builtin_amdgcn_s_sleep(2);
            if (!own && !dead) {
                const u64* fp = (const u64*)(myws + __builtin_amdgcn_readlane(o_flag, k) + c * FS);
                const u64 t0 = __builtin_amdgcn_s_memrealtime();
                unsigned it = 0;
                while (true) {
                    const u64 fv = __hip_atomic_load(fp, __ATOMIC_RELAXED, __HIP_MEMORY_SCOPE_SYSTEM);
                    info = __builtin_amdgcn_readfirstlane((unsigned)fv);
                    if ((int)(__builtin_amdgcn_readfirstlane((unsigned)(fv >> 32)) - call) >= 0) break;
                    if ((++it & 255) == 0) {
                        const unsigned tv = __builtin_amdgcn_readfirstlane(
                            __hip_atomic_load(tmo, __ATOMIC_RELAXED, __HIP_MEMORY_SCOPE_AGENT));
                        if (tv == call || __builtin_amdgcn_s_memrealtime() - t0 > DEADLINE_TICKS) {
                            __hip_atomic_store(tmo, call, __ATOMIC_RELAXED, __HIP_MEMORY_SCOPE_AGENT);
                            dead = 0x80000000u;
                            break;
                        }
                    }
                    __builtin_amdgcn_s_sleep(0);
                }
                asm volatile("" ::: "memory");
            }
            if (!dead) {
                const unsigned ou = __builtin_amdgcn_readlane(o_unit, k);
                const unsigned char* sb = own ? xp + ou + c * CHK : myws + ou + c * (NB16 * 4096);
                const unsigned vo = own ? lv : sv, st = own ? 64 : 1024;
#pragma unroll
                for (int q = 0; q < 4; ++q) glds16_sys(sb + q * st, vo, lds0 + LDS_A + sl * 4096 + q * 1024);
            }
            if (c >= NCH - 2 && __builtin_amdgcn_readfirstlane(
                    __hip_atomic_load(tmo, __ATOMIC_RELAXED, __HIP_MEMORY_SCOPE_AGENT)) == call) dead = 0x80000000u;
            wait_vm<0>();
            if (c + WARM < NCH && !own)
                glds4_sys(myws + __builtin_amdgcn_readlane(o_unit, k) + (c + WARM) * (NB16 * 4096), cb * 1024 + (lane & 7) * 128,
                          lds0 + LDS_P + d * 256);
            if (info <= 13u && !dead) {
                unsigned char* unit = smem + LDS_A + sl * 4096;
                unsigned v[16];
                decode_unit(unit, smem + LDS_X + d * 1600, lane, info, v);
#pragma unroll
                for (int q = 0; q < 4; ++q)
                    *(u32x4*)(unit + q * 1024 + sv) = u32x4{v[4 * q], v[4 * q + 1], v[4 * q + 2], v[4 * q + 3]};
            }
            lds_set(w_item + sl * 4, (i + 1) | dead);
            unsigned nx = 0;
            if (lane == 0) nx = __hip_atomic_fetch_add((LAS unsigned*)w_next, 1u, __ATOMIC_RELAXED, __HIP_MEMORY_SCOPE_WORKGROUP);
            i = NLOAD + __builtin_amdgcn_readfirstlane(nx);
        }
        return;
    }

    {
        unsigned lo, sel, two;
        symbol_entries(threadIdx.x, lo, sel, two);
        *(LAS u32x2*)(smem + LDS_T + threadIdx.x * 8) = u32x2{lo, sel};
        *(LAS unsigned*)(smem + LDS_T + 2048 + threadIdx.x * 4) = two;
    }
    wait_lds();
    if (lane == 0) __hip_atomic_fetch_add((LAS unsigned*)w_tab, 1u, __ATOMIC_RELAXED, __HIP_MEMORY_SCOPE_WORKGROUP);
    __builtin_amdgcn_s_setprio(2);
    const float scale = p.sx[0] * p.sw[0];
    const __amdgpu_buffer_rsrc_t orsrc = __builtin_amdgcn_make_buffer_rsrc((void*)p.out, 0, NR * MS * NL * 4, 0x00020000);
    const int G = lane >> 4, i16 = lane & 15;
    const int ldsb = ((16 * G + (i16 >> 1)) * 128 + (((i16 >> 2) & 3) | ((G & 1) << 2)) * 16 + 8 * (i16 & 1)) ^ (wave << 5);
    const int ocol = ((i16 & 7) * NL + cb * 128 + wave * 32 + (i16 >> 3) * 16 + G * 4) * 4;
    unsigned ctl = 0;
    auto wait_ctl = [&](int l, unsigned v) {
        unsigned x = __builtin_amdgcn_readlane(ctl, l);
        while ((x & 0x7fffffffu) <= v) {
            ctl = lds_get(w_item + (lane & 31) * 4);
            x = __builtin_amdgcn_readlane(ctl, l);
            if ((x & 0x7fffffffu) <= v) __builtin_amdgcn_s_sleep(1);
        }
        return x;
    };
    i32x8 wf[2][2], wl[2][2];
    auto read_w = [&](i32x8 (&z)[2][2], int slot) {
#pragma unroll
        for (int ks = 0; ks < 2; ++ks)
#pragma unroll
            for (int n = 0; n < 2; ++n) {
                const unsigned char* lp = smem + LDS_S + slot * STAGE + ks * 16384 + (ldsb ^ (n << 4));
                i32x2 t0 = __builtin_amdgcn_ds_read_tr8_b64_v2i32((LAS i32x2*)(lp));
                i32x2 t1 = __builtin_amdgcn_ds_read_tr8_b64_v2i32((LAS i32x2*)(lp + 1024));
                i32x2 t2 = __builtin_amdgcn_ds_read_tr8_b64_v2i32((LAS i32x2*)(lp + 8192));
                i32x2 t3 = __builtin_amdgcn_ds_read_tr8_b64_v2i32((LAS i32x2*)(lp + 8192 + 1024));
                z[ks][n] = i32x8{t0[0], t0[1], t1[0], t1[1], t2[0], t2[1], t3[0], t3[1]};
            }
    };
    f32x4 acc[SL][2];
#pragma unroll
    for (int k = 0; k < SL; ++k) acc[k][0] = acc[k][1] = f32x4{0.f, 0.f, 0.f, 0.f};
    asm volatile("" :: "v"(scale) : "memory");
    unsigned i = 0, as = 0, bad = 0;
    unsigned rowoff;
    {
        unsigned src, b;
        block_of(lane < S ? lane : 0, src, b);
        rowoff = (((rank + src) % NR) * MS + b * 16) * (NL * 4);
        asm volatile("" : "+v"(rowoff));
        const unsigned z = __builtin_amdgcn_raw_buffer_load_b32(orsrc, rowoff + cb * 512, 0, 17);
        asm volatile("" :: "v"(z));
    }
    u32x4 a[2][4];
    auto read_a = [&](u32x4 (&z)[4], unsigned v) {
        bad |= v >> 31;
        const unsigned char* unit = smem + LDS_A + as * 4096 + sv;
#pragma unroll
        for (int q = 0; q < 4; ++q) z[q] = *(const u32x4*)(unit + q * 1024);
        ++i; as = as + 1 == ARING ? 0 : as + 1;
    };
    auto mul = [&](f32x4 (&ac)[2], const i32x8 (&w)[2][2], const u32x4 (&z)[4]) {
        const i32x8 xa0 = {(int)z[0][0], (int)z[0][1], (int)z[0][2], (int)z[0][3], (int)z[1][0], (int)z[1][1], (int)z[1][2], (int)z[1][3]};
        const i32x8 xa1 = {(int)z[2][0], (int)z[2][1], (int)z[2][2], (int)z[2][3], (int)z[3][0], (int)z[3][1], (int)z[3][2], (int)z[3][3]};
        ac[0] = __builtin_amdgcn_mfma_scale_f32_16x16x128_f8f6f4(w[0][0], xa0, ac[0], 0, 0, 0, 0, 0, 0);
        ac[1] = __builtin_amdgcn_mfma_scale_f32_16x16x128_f8f6f4(w[0][1], xa0, ac[1], 0, 0, 0, 0, 0, 0);
        ac[0] = __builtin_amdgcn_mfma_scale_f32_16x16x128_f8f6f4(w[1][0], xa1, ac[0], 0, 0, 0, 0, 0, 0);
        ac[1] = __builtin_amdgcn_mfma_scale_f32_16x16x128_f8f6f4(w[1][1], xa1, ac[1], 0, 0, 0, 0, 0, 0);
    };
#pragma unroll 1
    for (int c = 0; c < NCH; c += 2) {
        wait_ctl(24, c + 1);
        read_w(wf, 0);
        read_w(wl, 1);
        lds_set(w_read + wave * 4, c + 2);
#pragma unroll
        for (int k = 0; k < SL; ++k) {
            if (k < SL - 1 || S == SL) {
                read_a(a[0], wait_ctl(as, i));
                read_a(a[1], wait_ctl(as, i));
                __builtin_amdgcn_sched_barrier(0);
                mul(acc[k], wf, a[0]);
                mul(acc[k], wl, a[1]);
                __builtin_amdgcn_sched_barrier(0);
                if (c < NCH - 2) lds_set(w_used + wave * 4, i);
                else {
                    int o1 = ocol + __builtin_amdgcn_readlane(rowoff, k), o2 = o1 + 8 * NL * 4;
                    asm volatile("" : "+v"(o1), "+v"(o2));
                    const float sc = bad ? __builtin_nanf("") : scale;
                    const u32x4 lo = __builtin_bit_cast(u32x4, acc[k][0] * sc), hi = __builtin_bit_cast(u32x4, acc[k][1] * sc);
                    u32x4 x, y;
#pragma unroll
                    for (int e = 0; e < 4; ++e) {
                        x[e] = __builtin_amdgcn_update_dpp(lo[e], hi[e], 0x128, 0xf, 0xc, false);
                        y[e] = __builtin_amdgcn_update_dpp(hi[e], lo[e], 0x128, 0xf, 0x3, false);
                    }
                    __builtin_amdgcn_raw_buffer_store_b128(x, orsrc, o1, 0, 17);
                    __builtin_amdgcn_raw_buffer_store_b128(y, orsrc, o2, 0, 17);
                }
            }
        }
    }
}

extern "C" void kernel_launch(int rank, int nranks, ncclComm_t comm,
                              hipStream_t stream,
                              void* const* d_in, const int* in_sizes, int n_in,
                              void* d_out, int out_size,
                              const int* in_dtypes, int out_dtype,
                              void* const* d_out_all,
                              void* const* d_ws_all, size_t ws_bytes) {
    (void)comm; (void)in_dtypes; (void)out_dtype; (void)d_out_all;
    if (nranks != NR || (unsigned)rank >= (unsigned)NR || n_in != 4 || in_sizes[0] != MS * KD ||
        in_sizes[1] != KD * NL || out_size != NR * MS * NL || ws_bytes < WS_NEED) {
        fprintf(stderr, "unexpected problem shape\n");
        return;
    }
    Params p;
    for (int j = 0; j < NR; ++j) p.ws[j] = (unsigned char*)d_ws_all[(rank + j) % NR];
    p.x = (const unsigned char*)d_in[0];
    p.w = (const unsigned char*)d_in[1];
    p.sx = (const float*)d_in[2];
    p.sw = (const float*)d_in[3];
    p.out = (float*)d_out;
    p.rank = rank;
    p.pad = 0;
    for (int e0 = 0; e0 < 16; ++e0) {
        unsigned t[4];
        encode_table(e0 < 13 ? e0 : 13, t);
        p.tab[e0] = u32x4{t[0], t[1], t[2], t[3]};
    }
    HIP_CHECK(hipFuncSetAttribute((const void*)ag_gemm, hipFuncAttributeMaxDynamicSharedMemorySize, LDS_BYTES));
    hipLaunchKernelGGL(ag_gemm, dim3(GRIDX), dim3(THREADS), LDS_BYTES, stream, p);
    HIP_CHECK(hipGetLastError());
}
```
